```python
import numpy as np

kernel_name = "gpu_gemm_ar_m4096_k4096_n4096_f32_silu"


ABSMAX_RTOL = 0.001

COMPUTE_DTYPE = "float32"
EPILOGUE = "silu"
IN_QUANT = None
OUT_QUANT = None
IN_DTYPES = {"x": "float32", "w_mat": "float32"}
WORLD_SIZE = 8

def setup_inputs():

    x = np.random.randn(4096, 4096).astype(np.float32)
    w_mat = (np.random.randn(4096, 4096) / np.sqrt(4096)).astype(np.float32)
    return {"x": x, "w_mat": w_mat}

def _epilogue(y):
    return (y / (1.0 + np.exp(-np.clip(y, -60.0, 60.0)))).astype(np.float32)

def reference(x, w_mat):
    y = (x @ w_mat).astype(np.float32)
    return _epilogue(y)
```

```cpp
#include <hip/hip_runtime.h>
#include <rccl/rccl.h>
#include <cstdio>

#define HIP_CHECK(expr) do { hipError_t _e = (expr); if (_e != hipSuccess) { \
    fprintf(stderr, "HIP error %s at %s:%d\n", hipGetErrorString(_e), __FILE__, __LINE__); \
    return; } } while (0)

#define LD_AUX 17

namespace {

constexpr int NR = 8;
constexpr int MM = 4096, NN = 4096, KSH = 512;
constexpr int NSTEP = 64;
constexpr int NCOMM = 32;
constexpr int NWG = 256;
constexpr int TM = 320, TN = 256;
constexpr int NGRP = 21;

constexpr size_t OFF_DONE = 0;
constexpr size_t OFF_TMO = 128;
constexpr size_t OFF_LC = 4096;
constexpr size_t OFF_FLAG = 16384;
constexpr size_t OFF_DBG = 65536;
constexpr size_t CTRL_BYTES = 1u << 20;
constexpr size_t CH_BYTES = 65536;
constexpr size_t OP_BYTES = (size_t)NSTEP * NR * CH_BYTES;
constexpr size_t WS_NEED = CTRL_BYTES + 4 * OP_BYTES;
static_assert(OFF_LC + NGRP * 128 <= OFF_FLAG, "layout");
static_assert(OFF_FLAG + NGRP * NR * 128 <= OFF_DBG, "layout");

constexpr int A_STAGE = 8 * TM * 16;
constexpr int B_STAGE = 8 * TN * 16;
constexpr int STAGE = A_STAGE + B_STAGE;
constexpr int LDS_BYTES = 2 * STAGE;

constexpr unsigned long long DEADLINE_TICKS = 50000000ull;

typedef unsigned u32x4 __attribute__((ext_vector_type(4)));
typedef float f32x4 __attribute__((ext_vector_type(4)));
typedef _Float16 h16x8 __attribute__((ext_vector_type(8)));
typedef _Float16 h16x2 __attribute__((ext_vector_type(2)));
typedef __attribute__((address_space(1))) unsigned gu32;
typedef __attribute__((address_space(1))) char gchar;

struct Params {
    const float* x;
    const float* w;
    float* out;
    char* ws[NR];
    int rank;
};

__device__ __forceinline__ int grp_of(int j) {
    return j < 52 ? (j >> 2) : (j < 60 ? 13 + ((j - 52) >> 1) : 17 + (j - 60));
}
__device__ __forceinline__ bool last_of_grp(int j) {
    return j < 52 ? ((j & 3) == 3) : (j < 60 ? (j & 1) == 1 : true);
}
__device__ __forceinline__ int grp_first(int g) {
    return g < 13 ? g * 4 : (g < 17 ? 52 + (g - 13) * 2 : 60 + (g - 17));
}

template <class T>
__device__ __forceinline__ T* uni(T* p) {
    unsigned long long v = (unsigned long long)p;
    unsigned lo = __builtin_amdgcn_readfirstlane((unsigned)v);
    unsigned hi = __builtin_amdgcn_readfirstlane((unsigned)(v >> 32));
    return (T*)(((unsigned long long)hi << 32) | lo);
}
__device__ __forceinline__ unsigned pk16(float a, float b) {
    h16x2 v;
    v.x = (_Float16)a;
    v.y = (_Float16)b;
    return __builtin_bit_cast(unsigned, v);
}
__device__ __forceinline__ float silu1(float y) {
    float c = __builtin_amdgcn_fmed3f(y, -60.0f, 60.0f);
    float e = __builtin_amdgcn_exp2f(c * -1.44269504088896340736f);
    return y * __builtin_amdgcn_rcpf(1.0f + e);
}
__device__ __forceinline__ unsigned ld_sys(const unsigned* p) {
    return __hip_atomic_load((gu32*)p, __ATOMIC_RELAXED, __HIP_MEMORY_SCOPE_SYSTEM);
}
__device__ __forceinline__ unsigned ld_agent(const unsigned* p) {
    return __hip_atomic_load((gu32*)p, __ATOMIC_RELAXED, __HIP_MEMORY_SCOPE_AGENT);
}
__device__ __forceinline__ void st_agent(unsigned* p, unsigned v) {
    __hip_atomic_store((gu32*)p, v, __ATOMIC_RELAXED, __HIP_MEMORY_SCOPE_AGENT);
}
__device__ __forceinline__ void st_sys(unsigned* p, unsigned v) {
    __hip_atomic_store((gu32*)p, v, __ATOMIC_RELAXED, __HIP_MEMORY_SCOPE_SYSTEM);
}
__device__ __forceinline__ void wait_flags(const unsigned* flag, unsigned call, unsigned* tmo) {
    unsigned long long t0 = __builtin_amdgcn_s_memrealtime();
    while (true) {
        unsigned v = ld_sys(flag);
        if (__all(v >= call)) break;
        if (__builtin_amdgcn_readfirstlane(ld_agent(tmo)) == call ||
            __builtin_amdgcn_s_memrealtime() - t0 > DEADLINE_TICKS) {
            st_agent(tmo, call);
            break;
        }
        __builtin_amdgcn_s_sleep(2);
    }
}

#define STAMP(idx) do { } while (0)

__device__ __forceinline__ void pusher(const Params& p, int piece, int phase, int lane, size_t stage_off,
                                       unsigned* cnt, char* ring) {
    const char* base[NR];
#pragma unroll
    for (int i = 0; i < NR; ++i) {
        int q = (p.rank + 1 + i + piece) & 7;
        base[i] = uni(p.ws[q] + stage_off);
    }
    const char* srcx = (const char*)(p.x + (size_t)piece * 64 * KSH) + lane * (KSH * 4);
    const char* srcw = (const char*)(p.w + piece * 64) + (lane >> 4) * (NN * 4) + (lane & 15) * 16;
    auto issue_load = [&](int j, int slot) {
        char* l = ring + slot * 2048;
        const bool isw = (j + phase) & 1;
        const char* g0 = isw ? srcw + (size_t)j * (8 * NN * 4) : srcx + j * 32;
        const char* g1 = g0 + (isw ? 4 * NN * 4 : 16);
        __builtin_amdgcn_global_load_lds((const gu32*)g0, (unsigned*)l, 16, 0, 0);
        __builtin_amdgcn_global_load_lds((const gu32*)g1, (unsigned*)(l + 1024), 16, 0, 0);
    };
    const unsigned cnt_addr = (unsigned)(size_t)(__attribute__((address_space(3))) unsigned*)cnt;
    auto signal = [&](int j) {
        if (j >= 0 && last_of_grp(j) && lane == 0)
            asm volatile("ds_add_u32 %0, %1" :: "v"(cnt_addr + grp_of(j) * 4), "v"(1u) : "memory");
    };
    const unsigned vbase = (unsigned)(p.rank * CH_BYTES + piece * 1024 + lane * 16);
    auto push = [&](int j, int slot) {
        const char* l = ring + slot * 2048;
        const bool isw = (j + phase) & 1;
        float s[8];
        if (isw) {
#pragma unroll
            for (int q = 0; q < 8; ++q) s[q] = *(const float*)(l + q * 256 + lane * 4);
        } else {
            *(f32x4*)&s[0] = *(const f32x4*)(l + lane * 16);
            *(f32x4*)&s[4] = *(const f32x4*)(l + 1024 + lane * 16);
        }
        u32x4 d;
        d.x = pk16(s[0], s[1]); d.y = pk16(s[2], s[3]); d.z = pk16(s[4], s[5]); d.w = pk16(s[6], s[7]);
        unsigned vo = vbase + (unsigned)j * (unsigned)(NR * CH_BYTES) + (isw ? (unsigned)OP_BYTES : 0u);
#pragma unroll
        for (int q = 0; q < NR; ++q)
            asm volatile("global_store_dwordx4 %0, %1, %2 sc0 sc1\n\ts_nop 1" :: "v"(vo), "v"(d), "s"(base[q]) : "memory");
    };
#define PUSH_WAIT(N) asm volatile("s_waitcnt vmcnt(" #N ")" ::: "memory")
    asm volatile("s_nop 4" ::: "memory");
    issue_load(0, 0);
    issue_load(1, 1);
    issue_load(2, 2); PUSH_WAIT(4);  push(0, 0);
    issue_load(3, 0); PUSH_WAIT(12); push(1, 1);
    issue_load(4, 1); PUSH_WAIT(20); push(2, 2);
#pragma unroll 1
    for (int j0 = 3; j0 < 60; j0 += 3) {
        issue_load(j0 + 2, 2); PUSH_WAIT(20); signal(j0 - 3); push(j0, 0);
        issue_load(j0 + 3, 0); PUSH_WAIT(20); signal(j0 - 2); push(j0 + 1, 1);
        issue_load(j0 + 4, 1); PUSH_WAIT(20); signal(j0 - 1); push(j0 + 2, 2);
    }
    issue_load(62, 2); PUSH_WAIT(20); signal(57); push(60, 0);
    issue_load(63, 0); PUSH_WAIT(20); signal(58); push(61, 1);
    PUSH_WAIT(18); signal(59); push(62, 2);
    PUSH_WAIT(16); signal(60); push(63, 0);
    PUSH_WAIT(16); signal(61);
    PUSH_WAIT(8);  signal(62);
    PUSH_WAIT(0);  signal(63);
#undef PUSH_WAIT
}

__device__ __forceinline__ void signaller(const Params& p, int which, int lane, unsigned call, unsigned* cnt) {
    unsigned* tmo = (unsigned*)(p.ws[p.rank] + OFF_TMO);
    for (int g = which; g < NGRP; g += 2) {
        unsigned long long t0 = __builtin_amdgcn_s_memrealtime();
        bool ok = true;
        while (__hip_atomic_load(&cnt[g], __ATOMIC_RELAXED, __HIP_MEMORY_SCOPE_WORKGROUP) < 4u) {
            if (__builtin_amdgcn_s_memrealtime() - t0 > DEADLINE_TICKS) { ok = false; break; }
            __builtin_amdgcn_s_sleep(1);
        }
        if (!ok) {
            st_agent(tmo, call);
            break;
        }
        unsigned old = 0;
        if (lane == 0)
            old = __hip_atomic_fetch_add((gu32*)(p.ws[p.rank] + OFF_LC + g * 128), 1u, __ATOMIC_RELAXED, __HIP_MEMORY_SCOPE_AGENT);
        old = __builtin_amdgcn_readfirstlane(old);
        STAMP(2 + g);
        if (((old + 1) & (NCOMM - 1)) == 0 && lane < NR) {
            unsigned* f = (unsigned*)(p.ws[lane] + OFF_FLAG + (g * NR + p.rank) * 128);
            st_sys(f, call);
        }
    }
}

__device__ __forceinline__ void gemm(const Params& p, int tm, int tn, char* smem, unsigned call, size_t stage_off) {
    const int tid = threadIdx.x, lane = tid & 63;
    const int wave = __builtin_amdgcn_readfirstlane(tid >> 6);
    const int wr = wave >> 2, wc = wave & 3;
    const int row0 = tm * TM, col0 = tn * TN;
    char* ws = uni(p.ws[p.rank]);
    unsigned* tmo = (unsigned*)(ws + OFF_TMO);
    const unsigned* myflag = (const unsigned*)(ws + OFF_FLAG + wave * 128);
    const __amdgpu_buffer_rsrc_t rs =
        __builtin_amdgcn_make_buffer_rsrc(ws + stage_off, 0, (int)(2 * OP_BYTES), 0x00020000);
    const int vx = wave * (int)CH_BYTES + row0 * 16 + lane * 16;
    const int vw = (int)OP_BYTES + wave * (int)CH_BYTES + col0 * 16 + lane * 16;
    const bool full = row0 + TM <= MM;

    auto stage_in = [&](int j) {
        __attribute__((address_space(3))) char* la =
            (__attribute__((address_space(3))) char*)(smem + (j & 1) * STAGE + wave * (TM * 16));
        __attribute__((address_space(3))) char* lb =
            (__attribute__((address_space(3))) char*)(smem + (j & 1) * STAGE + A_STAGE + wave * (TN * 16));
        const int so = j * (int)(NR * CH_BYTES);
        __builtin_amdgcn_raw_ptr_buffer_load_lds(rs, la, 16, vx, so, 0, LD_AUX);
        __builtin_amdgcn_raw_ptr_buffer_load_lds(rs, la, 16, vx, so, 1024, LD_AUX);
        __builtin_amdgcn_raw_ptr_buffer_load_lds(rs, la, 16, vx, so, 2048, LD_AUX);
        __builtin_amdgcn_raw_ptr_buffer_load_lds(rs, la, 16, vx, so, 3072, LD_AUX);
        if (full)
            __builtin_amdgcn_raw_ptr_buffer_load_lds(rs, la + 4096, 16, vx + 4096, so, 0, LD_AUX);
        __builtin_amdgcn_raw_ptr_buffer_load_lds(rs, lb, 16, vw, so, 0, LD_AUX);
        __builtin_amdgcn_raw_ptr_buffer_load_lds(rs, lb, 16, vw, so, 1024, LD_AUX);
        __builtin_amdgcn_raw_ptr_buffer_load_lds(rs, lb, 16, vw, so, 2048, LD_AUX);
        __builtin_amdgcn_raw_ptr_buffer_load_lds(rs, lb, 16, vw, so, 3072, LD_AUX);
    };

    f32x4 acc[10][4];
#pragma unroll
    for (int m = 0; m < 10; ++m)
#pragma unroll
        for (int n = 0; n < 4; ++n) acc[m][n] = f32x4{0.f, 0.f, 0.f, 0.f};

    const int fr = lane & 15, fq = lane >> 4;
    const int xoff = fq * (TM * 16) + (wr * 160 + fr) * 16;
    const int woff = A_STAGE + fq * (TN * 16) + (wc * 64 + fr) * 16;

    int ready = 0;
    wait_flags(myflag, call, tmo);
    ready = 1;
    stage_in(0);
    asm volatile("s_waitcnt vmcnt(0)" ::: "memory");
    __builtin_amdgcn_s_barrier();
    if (wave == 0) STAMP(1);

    for (int j = 0; j < NSTEP; ++j) {
        const bool more = j + 1 < NSTEP;
        const bool pre = more && grp_of(j + 1) < ready;
        if (pre) stage_in(j + 1);
        const int gnext = j + 2 < NSTEP ? grp_of(j + 2) : 0;
        const bool peek = pre && gnext >= ready;
        unsigned peekv = 0;
        if (peek) peekv = ld_sys(myflag + gnext * (NR * 128 / 4));
        const char* sx = smem + (j & 1) * STAGE + xoff;
        const char* sw = smem + (j & 1) * STAGE + woff;
        auto ldx = [&](int t) { return *(const h16x8*)(sx + (t / 10) * 4 * (TM * 16) + (t % 10) * 256); };
        auto ldw = [&](int s, int n) { return *(const h16x8*)(sw + s * 4 * (TN * 16) + n * 256); };
        h16x8 wf[2][4], xf[4];
        __builtin_amdgcn_sched_barrier(0);
        wf[0][0] = ldw(0, 0); xf[0] = ldx(0);
        wf[0][1] = ldw(0, 1); wf[0][2] = ldw(0, 2); wf[0][3] = ldw(0, 3);
        xf[1] = ldx(1); xf[2] = ldx(2);
#pragma unroll
        for (int t = 0; t < 20; ++t) {
            __builtin_amdgcn_sched_barrier(0);
            acc[t % 10][0] = __builtin_amdgcn_mfma_f32_16x16x32_f16(wf[t / 10][0], xf[t & 3], acc[t % 10][0], 0, 0, 0);
            __builtin_amdgcn_sched_barrier(0);
            if (t + 3 < 20) xf[(t + 3) & 3] = ldx(t + 3);
            if (t >= 2 && t < 6) wf[1][t - 2] = ldw(1, t - 2);
            __builtin_amdgcn_sched_barrier(0);
#pragma unroll
            for (int n = 1; n < 4; ++n)
                acc[t % 10][n] = __builtin_amdgcn_mfma_f32_16x16x32_f16(wf[t / 10][n], xf[t & 3], acc[t % 10][n], 0, 0, 0);
        }
        __builtin_amdgcn_sched_barrier(0);
        const unsigned pv = __builtin_amdgcn_readfirstlane(peekv);
        if (peek && pv >= call) ready = gnext + 1;
        if (more && !pre) {
            const int g = grp_of(j + 1);
            wait_flags(myflag + g * (NR * 128 / 4), call, tmo);
            ready = g + 1;
            stage_in(j + 1);
        }
        asm volatile("s_waitcnt vmcnt(0) lgkmcnt(0)" ::: "memory");
        __builtin_amdgcn_s_barrier();
    }

    if (wave == 0) STAMP(30);
    const bool bad = __builtin_amdgcn_readfirstlane(ld_agent(tmo)) == call;
    const int mlim = full ? 10 : (MM - row0 - wr * 160 + 15) / 16;
    float* o = p.out + (size_t)(row0 + wr * 160 + fr) * NN + col0 + wc * 64 + fq * 4;
#pragma unroll
    for (int m = 0; m < 10; ++m) {
        if (m < mlim) {
#pragma unroll
            for (int n = 0; n < 4; ++n) {
                f32x4 v = acc[m][n], y;
                y.x = silu1(v.x); y.y = silu1(v.y); y.z = silu1(v.z); y.w = silu1(v.w);
                if (bad) y = f32x4{__builtin_nanf(""), __builtin_nanf(""), __builtin_nanf(""), __builtin_nanf("")};
                *(f32x4*)(o + (size_t)m * 16 * NN + n * 16) = y;
            }
        }
    }
}

__global__ __launch_bounds__(512, 2) void fused_gemm_allgather_silu(Params p) {
    extern __shared__ __attribute__((aligned(16))) char smem[];
    const int tid = threadIdx.x, lane = tid & 63;
    const int wave = __builtin_amdgcn_readfirstlane(tid >> 6);
    unsigned* done = (unsigned*)(p.ws[p.rank] + OFF_DONE);
    const unsigned call = __builtin_amdgcn_readfirstlane(ld_agent(done)) / NWG + 1;
    const size_t stage_off = CTRL_BYTES + (size_t)(call & 1) * 2 * OP_BYTES;
    const int xcd = blockIdx.x & 7, slot = blockIdx.x >> 3;
    if (wave == 0) STAMP(0);
    if (slot < 4) {
        const int cb = slot * 8 + xcd;
        unsigned* cnt = (unsigned*)smem;
        if (tid < 32) cnt[tid] = 0;
        __syncthreads();
        if (wave < 4) pusher(p, cb * 2 + (wave & 1), wave >> 1, lane, stage_off, cnt, smem + 1024 + wave * 6144);
        else if (wave < 6) signaller(p, wave - 4, lane, call, cnt);
    } else if (xcd < 4) {
        const int i = slot - 4;
        gemm(p, i % 7, xcd * 4 + i / 7, smem, call, stage_off);
    } else if (slot >= 8) {
        const int i = slot - 8;
        gemm(p, 7 + i % 6, (xcd - 4) * 4 + i / 6, smem, call, stage_off);
    }
    __syncthreads();
    if (tid == 0) __hip_atomic_fetch_add((gu32*)done, 1u, __ATOMIC_RELAXED, __HIP_MEMORY_SCOPE_AGENT);
}

}

extern "C" void kernel_launch(int rank, int nranks, ncclComm_t comm,
                              hipStream_t stream,
                              void* const* d_in, const int* in_sizes, int n_in,
                              void* d_out, int out_size,
                              const int* in_dtypes, int out_dtype,
                              void* const* d_out_all,
                              void* const* d_ws_all, size_t ws_bytes) {
    (void)comm; (void)in_dtypes; (void)out_dtype; (void)d_out_all;
    if (nranks != NR || (unsigned)rank >= (unsigned)NR || n_in < 2 || ws_bytes < WS_NEED ||
        in_sizes[0] != MM * KSH || in_sizes[1] != KSH * NN || out_size != MM * NN) {
        fprintf(stderr, "unexpected problem shape\n");
        return;
    }
    static bool attr_set = false;
    if (!attr_set) {
        HIP_CHECK(hipFuncSetAttribute((const void*)fused_gemm_allgather_silu,
                                      hipFuncAttributeMaxDynamicSharedMemorySize, LDS_BYTES));
        attr_set = true;
    }
    Params p;
    p.x = (const float*)d_in[0];
    p.w = (const float*)d_in[1];
    p.out = (float*)d_out;
    for (int i = 0; i < NR; ++i) p.ws[i] = (char*)d_ws_all[i];
    p.rank = rank;
    fused_gemm_allgather_silu<<<NWG, 512, LDS_BYTES, stream>>>(p);
    HIP_CHECK(hipGetLastError());
}
```

```cpp
#include <hip/hip_runtime.h>
#include <rccl/rccl.h>
#include <cstdio>

#define HIP_CHECK(expr) do { hipError_t _e = (expr); if (_e != hipSuccess) { \
    fprintf(stderr, "HIP error %s at %s:%d\n", hipGetErrorString(_e), __FILE__, __LINE__); \
    return; } } while (0)

#define LD_AUX 17
#define EXP_SIGOFF 0
#define ST_AUX 16

namespace {

constexpr int NR = 8;
constexpr int MM = 4096, NN = 4096, KSH = 512;
constexpr int NSTEP = 64;
constexpr int NCOMM = 32;
constexpr int NWG = 256;
constexpr int TM = 320, TN = 256;
#define THROTTLE 8
#define ILV_E 1
#define ILV_L 0
#define LATE_T 4
#define PW 2

constexpr size_t OFF_DONE = 0;
constexpr size_t OFF_TMO = 128;
constexpr size_t OFF_LC = 4096;
constexpr size_t OFF_FLAG = 16384;
constexpr size_t OFF_DBG = 131072;
constexpr size_t CTRL_BYTES = 1u << 20;
constexpr size_t CH_BYTES = 65536;
constexpr size_t OP_BYTES = (size_t)NSTEP * NR * CH_BYTES;
constexpr size_t WS_NEED = CTRL_BYTES + 4 * OP_BYTES;
static_assert(OFF_LC + NSTEP * 128 <= OFF_FLAG, "layout");
static_assert(OFF_FLAG + NSTEP * NR * 128 <= OFF_DBG, "layout");

constexpr int A_STAGE = 8 * TM * 16;
constexpr int B_STAGE = 8 * TN * 16;
constexpr int STAGE = A_STAGE + B_STAGE;
constexpr int LDS_BYTES = 2 * STAGE;

constexpr unsigned long long DEADLINE_TICKS = 50000000ull;

typedef unsigned u32x4 __attribute__((ext_vector_type(4)));
typedef float f32x4 __attribute__((ext_vector_type(4)));
typedef _Float16 h16x8 __attribute__((ext_vector_type(8)));
typedef _Float16 h16x2 __attribute__((ext_vector_type(2)));
typedef __attribute__((address_space(1))) unsigned gu32;
typedef __attribute__((address_space(1))) char gchar;

struct Params {
    const float* x;
    const float* w;
    float* out;
    char* ws[NR];
    int rank;
};

template <class T>
__device__ __forceinline__ T* uni(T* p) {
    unsigned long long v = (unsigned long long)p;
    unsigned lo = __builtin_amdgcn_readfirstlane((unsigned)v);
    unsigned hi = __builtin_amdgcn_readfirstlane((unsigned)(v >> 32));
    return (T*)(((unsigned long long)hi << 32) | lo);
}
__device__ __forceinline__ unsigned pk16(float a, float b) {
    h16x2 v;
    v.x = (_Float16)a;
    v.y = (_Float16)b;
    return __builtin_bit_cast(unsigned, v);
}
__device__ __forceinline__ f32x4 silu4(f32x4 v) {
    f32x4 c, e, d;
#pragma unroll
    for (int i = 0; i < 4; ++i) c[i] = __builtin_amdgcn_fmed3f(v[i], -60.0f, 60.0f);
    c = c * -1.44269504088896340736f;
#pragma unroll
    for (int i = 0; i < 4; ++i) e[i] = __builtin_amdgcn_exp2f(c[i]);
    d = e + 1.0f;
#pragma unroll
    for (int i = 0; i < 4; ++i) d[i] = __builtin_amdgcn_rcpf(d[i]);
    return v * d;
}
__device__ __forceinline__ unsigned ld_sys(const unsigned* p) {
    return __hip_atomic_load((gu32*)p, __ATOMIC_RELAXED, __HIP_MEMORY_SCOPE_SYSTEM);
}
__device__ __forceinline__ unsigned ld_agent(const unsigned* p) {
    return __hip_atomic_load((gu32*)p, __ATOMIC_RELAXED, __HIP_MEMORY_SCOPE_AGENT);
}
__device__ __forceinline__ void st_agent(unsigned* p, unsigned v) {
    __hip_atomic_store((gu32*)p, v, __ATOMIC_RELAXED, __HIP_MEMORY_SCOPE_AGENT);
}
__device__ __forceinline__ void st_sys(unsigned* p, unsigned v) {
    __hip_atomic_store((gu32*)p, v, __ATOMIC_RELAXED, __HIP_MEMORY_SCOPE_SYSTEM);
}
__device__ __forceinline__ unsigned flag_peek(const unsigned* flag0, int g0, int lane) {
    int g = g0 + (lane & 7);
    g = g < NSTEP ? g : NSTEP - 1;
    return ld_sys(flag0 + g * (NR * 128 / 4));
}
__device__ __forceinline__ int flag_count(unsigned v, unsigned call) {
    unsigned m = (unsigned)__builtin_amdgcn_ballot_w64(v >= call) & 0xffu;
    return __builtin_ctz(~m);
}
__device__ __forceinline__ int wait_step(const unsigned* flag0, int g0, int lane, unsigned call, unsigned* tmo) {
    int c = flag_count(flag_peek(flag0, g0, lane), call);
    if (c > 0) return g0 + c;
    unsigned long long t0 = __builtin_amdgcn_s_memrealtime();
    for (unsigned it = 1;; ++it) {
        c = flag_count(flag_peek(flag0, g0, lane), call);
        if (c > 0) return g0 + c;
        if ((it & 15) == 0) {
            if (__builtin_amdgcn_readfirstlane(ld_agent(tmo)) == call ||
                __builtin_amdgcn_s_memrealtime() - t0 > DEADLINE_TICKS) {
                st_agent(tmo, call);
                return NSTEP + 8;
            }
        }
        __builtin_amdgcn_s_sleep(1);
    }
}

#define STAMP(idx) do { } while (0)

__device__ __forceinline__ void pusher(const Params& p, int piece, int phase, int lane, size_t stage_off,
                                       unsigned* cnt, char* ring) {
    const char* base[NR];
#pragma unroll
    for (int i = 0; i < NR; ++i) {
        int q = (p.rank + 1 + i + piece) & 7;
        base[i] = uni(p.ws[q] + stage_off);
    }
    const char* srcx = (const char*)(p.x + (size_t)piece * 64 * KSH) + lane * (KSH * 4);
    const char* srcw = (const char*)(p.w + piece * 64) + (lane >> 4) * (NN * 4) + (lane & 15) * 16;
    auto issue_load = [&](int j, int slot) {
        char* l = ring + slot * 2048;
        const bool isw = (j + phase) & 1;
        const char* g0 = isw ? srcw + (size_t)j * (8 * NN * 4) : srcx + j * 32;
        const char* g1 = g0 + (isw ? 4 * NN * 4 : 16);
        __builtin_amdgcn_global_load_lds((const gu32*)g0, (unsigned*)l, 16, 0, 0);
        __builtin_amdgcn_global_load_lds((const gu32*)g1, (unsigned*)(l + 1024), 16, 0, 0);
    };
    const unsigned cnt_addr = (unsigned)(size_t)(__attribute__((address_space(3))) unsigned*)cnt;
    auto signal = [&](int j) {
        if (lane == 0)
            asm volatile("ds_add_u32 %0, %1" :: "v"(cnt_addr + j * 4), "v"(1u) : "memory");
    };
    const unsigned vbase = (unsigned)(p.rank * CH_BYTES + piece * 1024 + lane * 16);
    auto push = [&](int j, int slot) {
        const char* l = ring + slot * 2048;
        const bool isw = (j + phase) & 1;
        float s[8];
        if (isw) {
#pragma unroll
            for (int q = 0; q < 8; ++q) s[q] = *(const float*)(l + q * 256 + lane * 4);
        } else {
            *(f32x4*)&s[0] = *(const f32x4*)(l + lane * 16);
            *(f32x4*)&s[4] = *(const f32x4*)(l + 1024 + lane * 16);
        }
        u32x4 d;
        d.x = pk16(s[0], s[1]); d.y = pk16(s[2], s[3]); d.z = pk16(s[4], s[5]); d.w = pk16(s[6], s[7]);
        unsigned vo = vbase + (unsigned)j * (unsigned)(NR * CH_BYTES) + (isw ? (unsigned)OP_BYTES : 0u);
#pragma unroll
        for (int q = 0; q < NR; ++q)
            asm volatile("global_store_dwordx4 %0, %1, %2 sc0 sc1\n\ts_nop 1" :: "v"(vo), "v"(d), "s"(base[q]) : "memory");
    };
#define PUSH_WAIT(N) asm volatile("s_waitcnt vmcnt(" #N ")" ::: "memory")
#define PUSH_WAITV(N) PUSH_WAIT(N)
    auto throttle = [&](int j) {
        if (j < THROTTLE) return;
        const unsigned need = j - THROTTLE + 1;
        unsigned v;
        asm volatile("ds_read_b32 %0, %1\n\ts_waitcnt lgkmcnt(0)" : "=v"(v) : "v"(cnt_addr + NSTEP * 4) : "memory");
        if (__builtin_amdgcn_readfirstlane(v) >= need) return;
        unsigned long long t0 = __builtin_amdgcn_s_memrealtime();
        do {
            __builtin_amdgcn_s_sleep(2);
            asm volatile("ds_read_b32 %0, %1\n\ts_waitcnt lgkmcnt(0)" : "=v"(v) : "v"(cnt_addr + NSTEP * 4) : "memory");
        } while (__builtin_amdgcn_readfirstlane(v) < need &&
                 __builtin_amdgcn_s_memrealtime() - t0 < DEADLINE_TICKS);
    };
    asm volatile("s_nop 4" ::: "memory");
    issue_load(0, 0);
    issue_load(1, 1);
    issue_load(2, 2); PUSH_WAIT(4);  push(0, 0);
    issue_load(3, 0); PUSH_WAIT(12); push(1, 1);
#pragma unroll 1
    for (int j0 = 2; j0 < 62; j0 += 3) {
        issue_load(j0 + 2, 1); PUSH_WAITV(PW); signal(j0 - 2); throttle(j0); push(j0, 2);
        issue_load(j0 + 3, 2); PUSH_WAITV(PW); signal(j0 - 1); throttle(j0 + 1); push(j0 + 1, 0);
        issue_load(j0 + 4, 0); PUSH_WAITV(PW); signal(j0); throttle(j0 + 2); push(j0 + 2, 1);
    }
    PUSH_WAIT(10); signal(60); throttle(62); push(62, 2);
    PUSH_WAIT(8);  signal(61); throttle(63); push(63, 0);
    PUSH_WAIT(8);  signal(62);
    PUSH_WAIT(0);  signal(63);
#undef PUSH_WAITV
#undef PUSH_WAIT
}

constexpr int NSIG = 3;
__device__ __forceinline__ void signaller(const Params& p, int which, int lane, unsigned call, unsigned* cnt) {
    unsigned* tmo = (unsigned*)(p.ws[p.rank] + OFF_TMO);
    for (int g = which; g < NSTEP; g += NSIG) {
        unsigned long long t0 = __builtin_amdgcn_s_memrealtime();
        bool ok = true;
        while (__hip_atomic_load(&cnt[g], __ATOMIC_RELAXED, __HIP_MEMORY_SCOPE_WORKGROUP) < 4u) {
            if (__builtin_amdgcn_s_memrealtime() - t0 > DEADLINE_TICKS) { ok = false; break; }
            __builtin_amdgcn_s_sleep(1);
        }
        if (!ok) {
            st_agent(tmo, call);
            break;
        }
        unsigned old = 0;
        if (lane == 0)
            old = __hip_atomic_fetch_add((gu32*)(p.ws[p.rank] + OFF_LC + g * 128), 1u, __ATOMIC_RELAXED, __HIP_MEMORY_SCOPE_AGENT);
        old = __builtin_amdgcn_readfirstlane(old);
        STAMP(2 + g);
        if (((old + 1) & (NCOMM - 1)) == 0 && lane < NR) {
            unsigned* f = (unsigned*)(p.ws[lane] + OFF_FLAG + (g * NR + p.rank) * 128);
            st_sys(f, call);
        }
    }
}
__device__ __forceinline__ void releaser(const Params& p, int lane, unsigned call, unsigned* cnt) {
    unsigned* tmo = (unsigned*)(p.ws[p.rank] + OFF_TMO);
    const unsigned* flag0 = (const unsigned*)(p.ws[p.rank] + OFF_FLAG + p.rank * 128);
    int have = 0;
    while (have < NSTEP - THROTTLE) {
        have = wait_step(flag0, have, lane, call, tmo);
        if (lane == 0) __hip_atomic_store(&cnt[NSTEP], (unsigned)have, __ATOMIC_RELAXED, __HIP_MEMORY_SCOPE_WORKGROUP);
    }
}

__device__ __forceinline__ void gemm(const Params& p, int tm, int tn, char* smem, unsigned call, size_t stage_off) {
    const int tid = threadIdx.x, lane = tid & 63;
    const int wave = __builtin_amdgcn_readfirstlane(tid >> 6);
    const int wr = wave >> 2, wc = wave & 3;
    const int row0 = tm * TM, col0 = tn * TN;
    char* ws = uni(p.ws[p.rank]);
    unsigned* tmo = (unsigned*)(ws + OFF_TMO);
    const unsigned* myflag = (const unsigned*)(ws + OFF_FLAG + wave * 128);
    const __amdgpu_buffer_rsrc_t rs =
        __builtin_amdgcn_make_buffer_rsrc(ws + stage_off, 0, (int)(2 * OP_BYTES), 0x00020000);
    const int vx = wave * (int)CH_BYTES + row0 * 16 + lane * 16;
    const int vw = (int)OP_BYTES + wave * (int)CH_BYTES + col0 * 16 + lane * 16;
    const bool full = row0 + TM <= MM;

    auto stage_in = [&](int j) {
        __attribute__((address_space(3))) char* la =
            (__attribute__((address_space(3))) char*)(smem + (j & 1) * STAGE + wave * (TM * 16));
        __attribute__((address_space(3))) char* lb =
            (__attribute__((address_space(3))) char*)(smem + (j & 1) * STAGE + A_STAGE + wave * (TN * 16));
        const int so = j * (int)(NR * CH_BYTES);
        __builtin_amdgcn_raw_ptr_buffer_load_lds(rs, la, 16, vx, so, 0, LD_AUX);
        __builtin_amdgcn_raw_ptr_buffer_load_lds(rs, la, 16, vx, so, 1024, LD_AUX);
        __builtin_amdgcn_raw_ptr_buffer_load_lds(rs, la, 16, vx, so, 2048, LD_AUX);
        __builtin_amdgcn_raw_ptr_buffer_load_lds(rs, la, 16, vx, so, 3072, LD_AUX);
        if (full)
            __builtin_amdgcn_raw_ptr_buffer_load_lds(rs, la + 4096, 16, vx + 4096, so, 0, LD_AUX);
        __builtin_amdgcn_raw_ptr_buffer_load_lds(rs, lb, 16, vw, so, 0, LD_AUX);
        __builtin_amdgcn_raw_ptr_buffer_load_lds(rs, lb, 16, vw, so, 1024, LD_AUX);
        __builtin_amdgcn_raw_ptr_buffer_load_lds(rs, lb, 16, vw, so, 2048, LD_AUX);
        __builtin_amdgcn_raw_ptr_buffer_load_lds(rs, lb, 16, vw, so, 3072, LD_AUX);
    };

    auto stage_one = [&](int j, int i) {
        __attribute__((address_space(3))) char* la =
            (__attribute__((address_space(3))) char*)(smem + (j & 1) * STAGE + wave * (TM * 16));
        __attribute__((address_space(3))) char* lb =
            (__attribute__((address_space(3))) char*)(smem + (j & 1) * STAGE + A_STAGE + wave * (TN * 16));
        const int so = j * (int)(NR * CH_BYTES);
        if (i < 4) __builtin_amdgcn_raw_ptr_buffer_load_lds(rs, la + i * 1024, 16, vx + i * 1024, so, 0, LD_AUX);
        else if (i == 4) { if (full) __builtin_amdgcn_raw_ptr_buffer_load_lds(rs, la + 4096, 16, vx + 4096, so, 0, LD_AUX); }
        else __builtin_amdgcn_raw_ptr_buffer_load_lds(rs, lb + (i - 5) * 1024, 16, vw + (i - 5) * 1024, so, 0, LD_AUX);
    };

    f32x4 acc[10][4];
#pragma unroll
    for (int m = 0; m < 10; ++m)
#pragma unroll
        for (int n = 0; n < 4; ++n) acc[m][n] = f32x4{0.f, 0.f, 0.f, 0.f};

    const int fr = lane & 15, fq = lane >> 4;
    const int xoff = fq * (TM * 16) + (wr * 160 + fr) * 16;
    const int woff = A_STAGE + fq * (TN * 16) + (wc * 64 + fr) * 16;

    int ready;
    ready = wait_step(myflag, 0, lane, call, tmo);
    stage_in(0);
    asm volatile("s_waitcnt vmcnt(0)" ::: "memory");
    __builtin_amdgcn_s_barrier();
    if (wave == 0) STAMP(1);

    for (int j = 0; j < NSTEP; ++j) {
        const bool more = j + 1 < NSTEP;
        const bool pre = more && j + 1 < ready;
        const bool peek = ready < NSTEP && ready < j + 4;
        const int ready0 = ready;
        unsigned peekv = 0;
        if (peek) peekv = flag_peek(myflag, ready0, lane);
        bool late = false;
        const bool preE = pre && wr == 0, preL = pre && wr != 0;
        if (ILV_L == 0 && preL) stage_in(j + 1);
        const char* sx = smem + (j & 1) * STAGE + xoff;
        const char* sw = smem + (j & 1) * STAGE + woff;
        auto ldx = [&](int t) { return *(const h16x8*)(sx + (t / 10) * 4 * (TM * 16) + (t % 10) * 256); };
        auto ldw = [&](int s, int n) { return *(const h16x8*)(sw + s * 4 * (TN * 16) + n * 256); };
        h16x8 wf[2][4], xf[4];
        __builtin_amdgcn_sched_barrier(0);
        wf[0][0] = ldw(0, 0); xf[0] = ldx(0);
        __builtin_amdgcn_sched_barrier(0);
        wf[0][1] = ldw(0, 1); wf[0][2] = ldw(0, 2); wf[0][3] = ldw(0, 3);
        xf[1] = ldx(1); xf[2] = ldx(2);
#pragma unroll
        for (int t = 0; t < 20; ++t) {
            __builtin_amdgcn_sched_barrier(0);
            acc[t % 10][0] = __builtin_amdgcn_mfma_f32_16x16x32_f16(wf[t / 10][0], xf[t & 3], acc[t % 10][0], 0, 0, 0);
            __builtin_amdgcn_sched_barrier(0);
            if (t + 3 < 20) xf[(t + 3) & 3] = ldx(t + 3);
            if (t >= 2 && t < 6) wf[1][t - 2] = ldw(1, t - 2);
#pragma unroll
            for (int i = 0; i < 9; ++i) {
                if (i / ILV_E == t && preE) stage_one(j + 1, i);
                if (ILV_L > 0 && i / (ILV_L > 0 ? ILV_L : 1) == t && preL) stage_one(j + 1, i);
            }
            if (t == LATE_T && more && !pre) {
                asm volatile("" : "+v"(peekv));
                ready = ready0 + flag_count(peekv, call);
                late = j + 1 < ready;
            }
            if (t >= LATE_T && t < LATE_T + 9 && late) stage_one(j + 1, t - LATE_T);
            __builtin_amdgcn_sched_barrier(0);
#pragma unroll
            for (int n = 1; n < 4; ++n)
                acc[t % 10][n] = __builtin_amdgcn_mfma_f32_16x16x32_f16(wf[t / 10][n], xf[t & 3], acc[t % 10][n], 0, 0, 0);
        }
        __builtin_amdgcn_sched_barrier(0);
        {
            const int c = flag_count(peekv, call);
            if (peek) ready = ready0 + c;
        }
        if (more && !pre && !late) {
            if (j + 1 >= ready) ready = wait_step(myflag, j + 1, lane, call, tmo);
            stage_in(j + 1);
        }
        asm volatile("s_waitcnt vmcnt(0) lgkmcnt(0)" ::: "memory");
        __builtin_amdgcn_s_barrier();
    }

    if (wave == 0) STAMP(68);
    const bool bad = __builtin_amdgcn_readfirstlane(ld_agent(tmo)) == call;
    const int mlim = full ? 10 : (MM - row0 - wr * 160 + 15) / 16;
    const __amdgpu_buffer_rsrc_t ro =
        __builtin_amdgcn_make_buffer_rsrc(p.out, 0, (int)((size_t)MM * NN * 4), 0x00020000);
    const int vo = ((row0 + wr * 160 + fr) * NN + col0 + wc * 64 + fq * 4) * 4;
    if (!bad) {
#pragma unroll
        for (int m = 0; m < 10; ++m) {
            if (m < mlim) {
#pragma unroll
                for (int n = 0; n < 4; ++n)
                    __builtin_amdgcn_raw_buffer_store_b128(__builtin_bit_cast(u32x4, silu4(acc[m][n])), ro,
                                                           vo + m * (16 * NN * 4) + n * 64, 0, ST_AUX);
            }
        }
    } else {
        const u32x4 nan4 = {0x7fc00000u, 0x7fc00000u, 0x7fc00000u, 0x7fc00000u};
        for (int m = 0; m < mlim; ++m)
            for (int n = 0; n < 4; ++n)
                __builtin_amdgcn_raw_buffer_store_b128(nan4, ro, vo + m * (16 * NN * 4) + n * 64, 0, 0);
    }
}

__global__ __launch_bounds__(512, 2) void fused_gemm_allgather_silu(Params p) {
    extern __shared__ __attribute__((aligned(16))) char smem[];
    const int tid = threadIdx.x, lane = tid & 63;
    const int wave = __builtin_amdgcn_readfirstlane(tid >> 6);
    unsigned* done = (unsigned*)(p.ws[p.rank] + OFF_DONE);
    const unsigned call = __builtin_amdgcn_readfirstlane(ld_agent(done)) / NWG + 1;
    const size_t stage_off = CTRL_BYTES + (size_t)(call & 1) * 2 * OP_BYTES;
    const int xcd = blockIdx.x & 7, slot = blockIdx.x >> 3;
    if (wave == 0) STAMP(0);
    if (slot < 4) {
        const int cb = slot * 8 + xcd;
        unsigned* cnt = (unsigned*)smem;
        if (tid < 128) cnt[tid] = 0;
        __syncthreads();
        if (wave < 4) pusher(p, cb * 2 + (wave & 1), wave >> 1, lane, stage_off, cnt, smem + 1024 + wave * 6144);
        else if (wave < 4 + NSIG + EXP_SIGOFF) signaller(p, wave - 4, lane, call, cnt);
        else if (wave < 8 + EXP_SIGOFF) releaser(p, lane, call, cnt);
    } else if (xcd < 4 || slot >= 8) {
        const int i = xcd < 4 ? slot - 4 : slot - 8;
        const int tm = xcd < 4 ? i % 7 : 7 + i % 6;
        const int tn = (xcd & 3) * 4 + (xcd < 4 ? i / 7 : i / 6);
        gemm(p, tm, tn, smem, call, stage_off);
    }
    __syncthreads();
    if (tid == 0) __hip_atomic_fetch_add((gu32*)done, 1u, __ATOMIC_RELAXED, __HIP_MEMORY_SCOPE_AGENT);
}

}

extern "C" void kernel_launch(int rank, int nranks, ncclComm_t comm,
                              hipStream_t stream,
                              void* const* d_in, const int* in_sizes, int n_in,
                              void* d_out, int out_size,
                              const int* in_dtypes, int out_dtype,
                              void* const* d_out_all,
                              void* const* d_ws_all, size_t ws_bytes) {
    (void)comm; (void)in_dtypes; (void)out_dtype; (void)d_out_all;
    if (nranks != NR || (unsigned)rank >= (unsigned)NR || n_in < 2 || ws_bytes < WS_NEED ||
        in_sizes[0] != MM * KSH || in_sizes[1] != KSH * NN || out_size != MM * NN) {
        fprintf(stderr, "unexpected problem shape\n");
        return;
    }
    static bool attr_set = false;
    if (!attr_set) {
        HIP_CHECK(hipFuncSetAttribute((const void*)fused_gemm_allgather_silu,
                                      hipFuncAttributeMaxDynamicSharedMemorySize, LDS_BYTES));
        attr_set = true;
    }
    Params p;
    p.x = (const float*)d_in[0];
    p.w = (const float*)d_in[1];
    p.out = (float*)d_out;
    for (int i = 0; i < NR; ++i) p.ws[i] = (char*)d_ws_all[i];
    p.rank = rank;
    fused_gemm_allgather_silu<<<NWG, 512, LDS_BYTES, stream>>>(p);
    HIP_CHECK(hipGetLastError());
}
```

```cpp
#include <hip/hip_runtime.h>
#include <rccl/rccl.h>
#include <cstdio>

#define HIP_CHECK(expr) do { hipError_t _e = (expr); if (_e != hipSuccess) { \
    fprintf(stderr, "HIP error %s at %s:%d\n", hipGetErrorString(_e), __FILE__, __LINE__); \
    return; } } while (0)

#define LD_AUX 17
#define EXP_SIGOFF 0
#define ST_AUX 16

namespace {

constexpr int NR = 8;
constexpr int MM = 4096, NN = 4096, KSH = 512;
constexpr int NSTEP = 64;
constexpr int NCOMM = 32;
constexpr int NSTAGE = 16;
constexpr int NWG = 256;
constexpr int TM = 320, TN = 256;
#define THROTTLE 6
#define ILV_E 1
#define ILV_L 0
#define LA 1
#define PSLEEP 4
#define PEEKW 3
#define LATE_T 4

constexpr size_t OFF_DONE = 0;
constexpr size_t OFF_TMO = 128;
constexpr size_t OFF_LC = 4096;
constexpr size_t OFF_LCL = 12288;
constexpr size_t OFF_FLAG = 16384;
constexpr size_t OFF_DBG = 131072;
constexpr size_t CTRL_BYTES = 1u << 20;
constexpr size_t CH_BYTES = 65536;
constexpr size_t OP_BYTES = (size_t)NSTEP * NR * CH_BYTES;
constexpr size_t WS_NEED = CTRL_BYTES + 4 * OP_BYTES;
static_assert(OFF_LC + NSTEP * 128 <= OFF_LCL && OFF_LCL + 8 * 128 <= OFF_FLAG, "layout");
static_assert(OFF_FLAG + NSTEP * NR * 128 <= OFF_DBG, "layout");

constexpr int A_STAGE = 8 * TM * 16;
constexpr int B_STAGE = 8 * TN * 16;
constexpr int STAGE = A_STAGE + B_STAGE;
constexpr int LDS_BYTES = 2 * STAGE;

constexpr unsigned long long DEADLINE_TICKS = 50000000ull;

typedef unsigned u32x4 __attribute__((ext_vector_type(4)));
typedef float f32x4 __attribute__((ext_vector_type(4)));
typedef _Float16 h16x8 __attribute__((ext_vector_type(8)));
typedef _Float16 h16x2 __attribute__((ext_vector_type(2)));
typedef __attribute__((address_space(1))) unsigned gu32;
typedef __attribute__((address_space(1))) char gchar;

struct Params {
    const float* x;
    const float* w;
    float* out;
    char* ws[NR];
    int rank;
};

template <class T>
__device__ __forceinline__ T* uni(T* p) {
    unsigned long long v = (unsigned long long)p;
    unsigned lo = __builtin_amdgcn_readfirstlane((unsigned)v);
    unsigned hi = __builtin_amdgcn_readfirstlane((unsigned)(v >> 32));
    return (T*)(((unsigned long long)hi << 32) | lo);
}
__device__ __forceinline__ unsigned pk16(float a, float b) {
    h16x2 v;
    v.x = (_Float16)a;
    v.y = (_Float16)b;
    return __builtin_bit_cast(unsigned, v);
}
typedef float f32x2 __attribute__((ext_vector_type(2)));
__device__ __forceinline__ f32x4 silu4(f32x4 v, f32x2 k) {
    const float lim = 60.0f * 1.44269504088896340736f;
    f32x2 a = f32x2{v.x, v.y} * k, b = f32x2{v.z, v.w} * k;
    a.x = __builtin_amdgcn_exp2f(__builtin_amdgcn_fmed3f(a.x, -lim, lim));
    a.y = __builtin_amdgcn_exp2f(__builtin_amdgcn_fmed3f(a.y, -lim, lim));
    b.x = __builtin_amdgcn_exp2f(__builtin_amdgcn_fmed3f(b.x, -lim, lim));
    b.y = __builtin_amdgcn_exp2f(__builtin_amdgcn_fmed3f(b.y, -lim, lim));
    a = a + 1.0f;
    b = b + 1.0f;
    a.x = __builtin_amdgcn_rcpf(a.x); a.y = __builtin_amdgcn_rcpf(a.y);
    b.x = __builtin_amdgcn_rcpf(b.x); b.y = __builtin_amdgcn_rcpf(b.y);
    a = a * f32x2{v.x, v.y};
    b = b * f32x2{v.z, v.w};
    return f32x4{a.x, a.y, b.x, b.y};
}
__device__ __forceinline__ unsigned ld_sys(const unsigned* p) {
    return __hip_atomic_load((gu32*)p, __ATOMIC_RELAXED, __HIP_MEMORY_SCOPE_SYSTEM);
}
__device__ __forceinline__ unsigned ld_agent(const unsigned* p) {
    return __hip_atomic_load((gu32*)p, __ATOMIC_RELAXED, __HIP_MEMORY_SCOPE_AGENT);
}
__device__ __forceinline__ void st_agent(unsigned* p, unsigned v) {
    __hip_atomic_store((gu32*)p, v, __ATOMIC_RELAXED, __HIP_MEMORY_SCOPE_AGENT);
}
__device__ __forceinline__ void st_sys(unsigned* p, unsigned v) {
    __hip_atomic_store((gu32*)p, v, __ATOMIC_RELAXED, __HIP_MEMORY_SCOPE_SYSTEM);
}
struct FlagTrack {
    const char* f0;
    unsigned call;
    __device__ __forceinline__ unsigned peek(int i0, int lane, unsigned& thr) const {
        int g = i0 + (lane & 7);
        g = g < NSTEP ? g : NSTEP - 1;
        thr = call;
        unsigned v = call - 1;
        if (lane < LA) v = ld_sys((const unsigned*)(f0 + g * (NR * 128)));
        return v;
    }
};
struct StepTrack {
    const char* ws;
    unsigned call;
    int wave, rank;
    __device__ __forceinline__ unsigned peek(int i0, int lane, unsigned& thr) const {
        int s = i0 + (lane & 7);
        s = s < NSTEP ? s : NSTEP - 1;
        const int q = 8 * (s - 8) + wave;
        const int j = (q * 9363) >> 16;
        const int src = (rank + 1 + q - 7 * j) & 7;
        const int off = s < 8 ? (int)OFF_LCL + s * 128 : (int)OFF_FLAG + (j * NR + src) * 128;
        thr = s < 8 ? call * NSTAGE : call;
        unsigned v = 0;
        if (lane < LA) v = ld_sys((const unsigned*)(ws + off));
        return v;
    }
};
__device__ __forceinline__ int flag_count(unsigned v, unsigned thr) {
    unsigned m = (unsigned)__builtin_amdgcn_ballot_w64((int)(v - thr) >= 0) & ((1u << LA) - 1);
    return __builtin_ctz(~m);
}
template <class T>
__device__ __forceinline__ int wait_step(const T& tr, int g0, int lane, unsigned* tmo) {
    unsigned thr;
    unsigned v = tr.peek(g0, lane, thr);
    int c = flag_count(v, thr);
    if (c > 0) return g0 + c;
    unsigned long long t0 = __builtin_amdgcn_s_memrealtime();
    for (unsigned it = 1;; ++it) {
        v = tr.peek(g0, lane, thr);
        c = flag_count(v, thr);
        if (c > 0) return g0 + c;
        if ((it & 15) == 0) {
            if (__builtin_amdgcn_readfirstlane(ld_agent(tmo)) == tr.call ||
                __builtin_amdgcn_s_memrealtime() - t0 > DEADLINE_TICKS) {
                st_agent(tmo, tr.call);
                return NSTEP + 8;
            }
        }
        __builtin_amdgcn_s_sleep(PSLEEP);
    }
}

#define STAMP(idx) do { } while (0)

__device__ __forceinline__ void pusher(const Params& p, int piece, int phase, int lane, size_t stage_off,
                                       unsigned* cnt, char* ring) {
    const char* base[NR - 1];
#pragma unroll
    for (int i = 0; i < NR - 1; ++i) {
        int q = (p.rank + 1 + (i + piece) % 7) & 7;
        base[i] = uni(p.ws[q] + stage_off);
    }
    const char* srcx = (const char*)(p.x + (size_t)piece * 64 * KSH) + lane * (KSH * 4);
    const char* srcw = (const char*)(p.w + piece * 64) + (lane >> 4) * (NN * 4) + (lane & 15) * 16;
    auto issue_load = [&](int j, int slot) {
        char* l = ring + slot * 2048;
        const bool isw = (j + phase) & 1;
        const char* g0 = isw ? srcw + (size_t)j * (8 * NN * 4) : srcx + j * 32;
        const char* g1 = g0 + (isw ? 4 * NN * 4 : 16);
        __builtin_amdgcn_global_load_lds((const gu32*)g0, (unsigned*)l, 16, 0, 0);
        __builtin_amdgcn_global_load_lds((const gu32*)g1, (unsigned*)(l + 1024), 16, 0, 0);
    };
    const unsigned cnt_addr = (unsigned)(size_t)(__attribute__((address_space(3))) unsigned*)cnt;
    auto signal = [&](int j) {
        if (lane == 0)
            asm volatile("ds_add_u32 %0, %1" :: "v"(cnt_addr + j * 4), "v"(1u) : "memory");
    };
    const unsigned vbase = (unsigned)(p.rank * CH_BYTES + piece * 1024 + lane * 16);
    auto push = [&](int j, int slot) {
        const char* l = ring + slot * 2048;
        const bool isw = (j + phase) & 1;
        float s[8];
        if (isw) {
#pragma unroll
            for (int q = 0; q < 8; ++q) s[q] = *(const float*)(l + q * 256 + lane * 4);
        } else {
            *(f32x4*)&s[0] = *(const f32x4*)(l + lane * 16);
            *(f32x4*)&s[4] = *(const f32x4*)(l + 1024 + lane * 16);
        }
        u32x4 d;
        d.x = pk16(s[0], s[1]); d.y = pk16(s[2], s[3]); d.z = pk16(s[4], s[5]); d.w = pk16(s[6], s[7]);
        unsigned vo = vbase + (unsigned)j * (unsigned)(NR * CH_BYTES) + (isw ? (unsigned)OP_BYTES : 0u);
#pragma unroll
        for (int q = 0; q < NR - 1; ++q)
            asm volatile("global_store_dwordx4 %0, %1, %2 sc0 sc1\n\ts_nop 1" :: "v"(vo), "v"(d), "s"(base[q]) : "memory");
    };
#define PUSH_WAIT(N) asm volatile("s_waitcnt vmcnt(" #N ")" ::: "memory")
    auto throttle = [&](int j) {
        if (j < THROTTLE) return;
        const unsigned need = j - THROTTLE + 1;
        unsigned v;
        asm volatile("ds_read_b32 %0, %1\n\ts_waitcnt lgkmcnt(0)" : "=v"(v) : "v"(cnt_addr + NSTEP * 4) : "memory");
        if (__builtin_amdgcn_readfirstlane(v) >= need) return;
        unsigned long long t0 = __builtin_amdgcn_s_memrealtime();
        do {
            __builtin_amdgcn_s_sleep(2);
            asm volatile("ds_read_b32 %0, %1\n\ts_waitcnt lgkmcnt(0)" : "=v"(v) : "v"(cnt_addr + NSTEP * 4) : "memory");
        } while (__builtin_amdgcn_readfirstlane(v) < need &&
                 __builtin_amdgcn_s_memrealtime() - t0 < DEADLINE_TICKS);
    };
    asm volatile("s_nop 4" ::: "memory");
    issue_load(0, 0);
    issue_load(1, 1);
    issue_load(2, 2); PUSH_WAIT(4); push(0, 0);
    issue_load(3, 0); PUSH_WAIT(2); signal(0); push(1, 1);
#pragma unroll 1
    for (int j0 = 2; j0 < 62; j0 += 3) {
        issue_load(j0 + 2, 1); PUSH_WAIT(2); signal(j0 - 1); throttle(j0); push(j0, 2);
        issue_load(j0 + 3, 2); PUSH_WAIT(2); signal(j0); throttle(j0 + 1); push(j0 + 1, 0);
        issue_load(j0 + 4, 0); PUSH_WAIT(2); signal(j0 + 1); throttle(j0 + 2); push(j0 + 2, 1);
    }
    PUSH_WAIT(0); signal(61); throttle(62); push(62, 2);
    PUSH_WAIT(0); signal(62); throttle(63); push(63, 0);
    PUSH_WAIT(0); signal(63);
#undef PUSH_WAIT
}

constexpr int NSIG = 3;
__device__ __forceinline__ void signaller(const Params& p, int which, int lane, unsigned call, unsigned* cnt) {
    unsigned* tmo = (unsigned*)(p.ws[p.rank] + OFF_TMO);
    for (int g = which; g < NSTEP; g += NSIG) {
        unsigned long long t0 = __builtin_amdgcn_s_memrealtime();
        bool ok = true;
        while (__hip_atomic_load(&cnt[g], __ATOMIC_RELAXED, __HIP_MEMORY_SCOPE_WORKGROUP) < 4u) {
            if (__builtin_amdgcn_s_memrealtime() - t0 > DEADLINE_TICKS) { ok = false; break; }
            __builtin_amdgcn_s_sleep(1);
        }
        if (!ok) {
            st_agent(tmo, call);
            break;
        }
        unsigned old = 0;
        if (lane == 0)
            old = __hip_atomic_fetch_add((gu32*)(p.ws[p.rank] + OFF_LC + g * 128), 1u, __ATOMIC_RELAXED, __HIP_MEMORY_SCOPE_AGENT);
        old = __builtin_amdgcn_readfirstlane(old);
        STAMP(2 + g);
        if (((old + 1) & (NCOMM - 1)) == 0 && lane < NR) {
            unsigned* f = (unsigned*)(p.ws[lane] + OFF_FLAG + (g * NR + p.rank) * 128);
            st_sys(f, call);
        }
    }
}
__device__ __forceinline__ void releaser(const Params& p, int lane, unsigned call, unsigned* cnt) {
    unsigned* tmo = (unsigned*)(p.ws[p.rank] + OFF_TMO);
    const FlagTrack tr{p.ws[p.rank] + OFF_FLAG + p.rank * 128, call};
    int have = 0;
    while (have < NSTEP - THROTTLE) {
        have = wait_step(tr, have, lane, tmo);
        if (lane == 0) __hip_atomic_store(&cnt[NSTEP], (unsigned)have, __ATOMIC_RELAXED, __HIP_MEMORY_SCOPE_WORKGROUP);
    }
}

__device__ __forceinline__ void stage_own(const Params& p, int g, int wave, int lane, size_t stage_off, char* l) {
    const int b = g & 7;
    const int v = (g >> 3) * 8 + wave;
    const int piece = v >> 1;
    const bool isw = v & 1;
    const __amdgpu_buffer_rsrc_t rd = __builtin_amdgcn_make_buffer_rsrc(
        p.ws[p.rank] + stage_off + (isw ? OP_BYTES : 0), 0, (int)OP_BYTES, 0x00020000);
    const int vo = p.rank * (int)CH_BYTES + piece * 1024 + lane * 16;
    const int r = lane >> 4, c = lane & 15;
    const char* src = isw ? (const char*)(p.w + (size_t)(b * 64 + r) * NN + piece * 64)
                          : (const char*)(p.x + (size_t)(piece * 64 + r) * KSH + b * 64);
    const int rstride = isw ? 4 * NN * 4 : 4 * KSH * 4;
#pragma unroll
    for (int i = 0; i < 16; ++i) {
        const int slot = isw ? c : (c ^ ((4 * i + r) & 15));
        __builtin_amdgcn_global_load_lds((const gu32*)(src + (size_t)i * rstride + slot * 16),
                                         (unsigned*)(l + i * 1024), 16, 0, 0);
    }
    asm volatile("s_waitcnt vmcnt(0)" ::: "memory");
    float cur[64];
    if (isw) {
#pragma unroll
        for (int k = 0; k < 64; ++k) cur[k] = *(const float*)(l + k * 256 + lane * 4);
    } else {
#pragma unroll
        for (int k = 0; k < 16; ++k) *(f32x4*)&cur[4 * k] = *(const f32x4*)(l + lane * 256 + ((k ^ (lane & 15)) * 16));
    }
#pragma unroll
    for (int o = 0; o < 8; ++o) {
        u32x4 d;
        d.x = pk16(cur[8 * o], cur[8 * o + 1]); d.y = pk16(cur[8 * o + 2], cur[8 * o + 3]);
        d.z = pk16(cur[8 * o + 4], cur[8 * o + 5]); d.w = pk16(cur[8 * o + 6], cur[8 * o + 7]);
        __builtin_amdgcn_raw_buffer_store_b128(d, rd, vo, (b * 8 + o) * (int)(NR * CH_BYTES), 17);
    }
    asm volatile("s_waitcnt vmcnt(0)" ::: "memory");
    __syncthreads();
    if (threadIdx.x == 0)
        __hip_atomic_fetch_add((gu32*)(p.ws[p.rank] + OFF_LCL + b * 128), 1u, __ATOMIC_RELAXED, __HIP_MEMORY_SCOPE_AGENT);
}

__device__ __forceinline__ void gemm(const Params& p, int tm, int tn, char* smem, unsigned call, size_t stage_off) {
    const int tid = threadIdx.x, lane = tid & 63;
    const int wave = __builtin_amdgcn_readfirstlane(tid >> 6);
    const int wr = wave >> 2, wc = wave & 3;
    const int row0 = tm * TM, col0 = tn * TN;
    char* ws = uni(p.ws[p.rank]);
    unsigned* tmo = (unsigned*)(ws + OFF_TMO);
    const StepTrack tr{ws, call, wave, p.rank};
    const __amdgpu_buffer_rsrc_t rs =
        __builtin_amdgcn_make_buffer_rsrc(ws + stage_off, 0, (int)(2 * OP_BYTES), 0x00020000);
    const int vx = row0 * 16 + lane * 16;
    const int vw = (int)OP_BYTES + col0 * 16 + lane * 16;
    const bool full = row0 + TM <= MM;

    auto plane = [&](int st) {
        const int q = 8 * (st - 8) + wave;
        const int jj = (q * 9363) >> 16;
        return st < 8 ? ((8 * st + wave) * NR + p.rank) * (int)CH_BYTES
                      : (jj * NR + ((p.rank + 1 + q - 7 * jj) & 7)) * (int)CH_BYTES;
    };
    auto stage_in = [&](int j) {
        __attribute__((address_space(3))) char* la =
            (__attribute__((address_space(3))) char*)(smem + (j & 1) * STAGE + wave * (TM * 16));
        __attribute__((address_space(3))) char* lb =
            (__attribute__((address_space(3))) char*)(smem + (j & 1) * STAGE + A_STAGE + wave * (TN * 16));
        const int so = plane(j);
        __builtin_amdgcn_raw_ptr_buffer_load_lds(rs, la, 16, vx, so, 0, LD_AUX);
        __builtin_amdgcn_raw_ptr_buffer_load_lds(rs, la, 16, vx, so, 1024, LD_AUX);
        __builtin_amdgcn_raw_ptr_buffer_load_lds(rs, la, 16, vx, so, 2048, LD_AUX);
        __builtin_amdgcn_raw_ptr_buffer_load_lds(rs, la, 16, vx, so, 3072, LD_AUX);
        if (full)
            __builtin_amdgcn_raw_ptr_buffer_load_lds(rs, la + 4096, 16, vx + 4096, so, 0, LD_AUX);
        __builtin_amdgcn_raw_ptr_buffer_load_lds(rs, lb, 16, vw, so, 0, LD_AUX);
        __builtin_amdgcn_raw_ptr_buffer_load_lds(rs, lb, 16, vw, so, 1024, LD_AUX);
        __builtin_amdgcn_raw_ptr_buffer_load_lds(rs, lb, 16, vw, so, 2048, LD_AUX);
        __builtin_amdgcn_raw_ptr_buffer_load_lds(rs, lb, 16, vw, so, 3072, LD_AUX);
    };

    auto stage_one = [&](int j, int i, int so) {
        __attribute__((address_space(3))) char* la =
            (__attribute__((address_space(3))) char*)(smem + (j & 1) * STAGE + wave * (TM * 16));
        __attribute__((address_space(3))) char* lb =
            (__attribute__((address_space(3))) char*)(smem + (j & 1) * STAGE + A_STAGE + wave * (TN * 16));
        if (i < 4) __builtin_amdgcn_raw_ptr_buffer_load_lds(rs, la + i * 1024, 16, vx + i * 1024, so, 0, LD_AUX);
        else if (i == 4) { if (full) __builtin_amdgcn_raw_ptr_buffer_load_lds(rs, la + 4096, 16, vx + 4096, so, 0, LD_AUX); }
        else __builtin_amdgcn_raw_ptr_buffer_load_lds(rs, lb + (i - 5) * 1024, 16, vw + (i - 5) * 1024, so, 0, LD_AUX);
    };

    f32x4 acc[10][4];
#pragma unroll
    for (int m = 0; m < 10; ++m)
#pragma unroll
        for (int n = 0; n < 4; ++n) acc[m][n] = f32x4{0.f, 0.f, 0.f, 0.f};

    const int fr = lane & 15, fq = lane >> 4;
    const int xoff = fq * (TM * 16) + (wr * 160 + fr) * 16;
    const int woff = A_STAGE + fq * (TN * 16) + (wc * 64 + fr) * 16;

    int ready;
    ready = wait_step(tr, 0, lane, tmo);
    stage_in(0);
    asm volatile("s_waitcnt vmcnt(0)" ::: "memory");
    __builtin_amdgcn_s_barrier();
    if (wave == 0) STAMP(1);

    for (int j = 0; j < NSTEP; ++j) {
        const bool more = j + 1 < NSTEP;
        const bool pre = more && j + 1 < ready;
        const bool peek = ready < NSTEP && ready < j + PEEKW;
        const int ready0 = ready;
        unsigned peekv = 0, peekt = 0;
        if (peek) peekv = tr.peek(ready0, lane, peekt);
        bool late = false;
        const int so1 = __builtin_amdgcn_readfirstlane(plane(j + 1));
        const bool preE = pre && wr == 0, preL = pre && wr != 0;
        if (ILV_L == 0 && preL) stage_in(j + 1);
        const char* sx = smem + (j & 1) * STAGE + xoff;
        const char* sw = smem + (j & 1) * STAGE + woff;
        auto ldx = [&](int t) { return *(const h16x8*)(sx + (t / 10) * 4 * (TM * 16) + (t % 10) * 256); };
        auto ldw = [&](int s, int n) { return *(const h16x8*)(sw + s * 4 * (TN * 16) + n * 256); };
        h16x8 wf[2][4], xf[4];
        __builtin_amdgcn_sched_barrier(0);
        wf[0][0] = ldw(0, 0); xf[0] = ldx(0);
        __builtin_amdgcn_sched_barrier(0);
        wf[0][1] = ldw(0, 1); wf[0][2] = ldw(0, 2); wf[0][3] = ldw(0, 3);
        xf[1] = ldx(1); xf[2] = ldx(2);
#pragma unroll
        for (int t = 0; t < 20; ++t) {
            __builtin_amdgcn_sched_barrier(0);
            acc[t % 10][0] = __builtin_amdgcn_mfma_f32_16x16x32_f16(wf[t / 10][0], xf[t & 3], acc[t % 10][0], 0, 0, 0);
            __builtin_amdgcn_sched_barrier(0);
            if (t + 3 < 20) xf[(t + 3) & 3] = ldx(t + 3);
            if (t >= 2 && t < 6) wf[1][t - 2] = ldw(1, t - 2);
#pragma unroll
            for (int i = 0; i < 9; ++i) {
                if (i / ILV_E == t && preE) stage_one(j + 1, i, so1);
                if (ILV_L > 0 && i / (ILV_L > 0 ? ILV_L : 1) == t && preL) stage_one(j + 1, i, so1);
            }
            if (t == LATE_T && more && !pre) {
                asm volatile("" : "+v"(peekv));
                ready = ready0 + flag_count(peekv, peekt);
                late = j + 1 < ready;
            }
            if (t >= LATE_T && t < LATE_T + 9 && late) stage_one(j + 1, t - LATE_T, so1);
            __builtin_amdgcn_sched_barrier(0);
#pragma unroll
            for (int n = 1; n < 4; ++n)
                acc[t % 10][n] = __builtin_amdgcn_mfma_f32_16x16x32_f16(wf[t / 10][n], xf[t & 3], acc[t % 10][n], 0, 0, 0);
        }
        __builtin_amdgcn_sched_barrier(0);
        {
            const int c = flag_count(peekv, peekt);
            if (peek) ready = ready0 + c;
        }
        if (more && !pre && !late) {
            if (j + 1 >= ready) ready = wait_step(tr, j + 1, lane, tmo);
            stage_in(j + 1);
        }
        asm volatile("s_waitcnt vmcnt(0) lgkmcnt(0)" ::: "memory");
        if (more) __builtin_amdgcn_s_barrier();
    }

    if (wave == 0) STAMP(68);
    const bool bad = __builtin_amdgcn_readfirstlane(ld_agent(tmo)) == call;
    const int mlim = full ? 10 : (MM - row0 - wr * 160 + 15) / 16;
    const __amdgpu_buffer_rsrc_t ro =
        __builtin_amdgcn_make_buffer_rsrc(p.out, 0, (int)((size_t)MM * NN * 4), 0x00020000);
    const int vo = ((row0 + wr * 160 + fr) * NN + col0 + wc * 64 + fq * 4) * 4;
    f32x2 kneg = {-1.44269504088896340736f, -1.44269504088896340736f};
    asm volatile("" : "+v"(kneg));
    if (!bad) {
#pragma unroll
        for (int m = 0; m < 10; ++m) {
            if (m < mlim) {
#pragma unroll
                for (int n = 0; n < 4; ++n)
                    __builtin_amdgcn_raw_buffer_store_b128(__builtin_bit_cast(u32x4, silu4(acc[m][n], kneg)), ro,
                                                           vo + m * (16 * NN * 4) + n * 64, 0, ST_AUX);
            }
        }
    } else {
        const u32x4 nan4 = {0x7fc00000u, 0x7fc00000u, 0x7fc00000u, 0x7fc00000u};
        for (int m = 0; m < mlim; ++m)
            for (int n = 0; n < 4; ++n)
                __builtin_amdgcn_raw_buffer_store_b128(nan4, ro, vo + m * (16 * NN * 4) + n * 64, 0, 0);
    }
}

__global__ __launch_bounds__(512, 2) void fused_gemm_allgather_silu(Params p) {
    extern __shared__ __attribute__((aligned(16))) char smem[];
    const int tid = threadIdx.x, lane = tid & 63;
    const int wave = __builtin_amdgcn_readfirstlane(tid >> 6);
    unsigned* done = (unsigned*)(p.ws[p.rank] + OFF_DONE);
    const unsigned call = __builtin_amdgcn_readfirstlane(ld_agent(done)) / NWG + 1;
    const size_t stage_off = CTRL_BYTES + (size_t)(call & 1) * 2 * OP_BYTES;
    const int xcd = blockIdx.x & 7, slot = blockIdx.x >> 3;
    if (wave == 0) STAMP(0);
    if (slot < 4) {
        const int cb = slot * 8 + xcd;
        unsigned* cnt = (unsigned*)smem;
        if (tid < 128) cnt[tid] = 0;
        __syncthreads();
        if (wave < 4) pusher(p, cb * 2 + (wave & 1), wave >> 1, lane, stage_off, cnt, smem + 1024 + wave * 6144);
        else if (wave < 4 + NSIG + EXP_SIGOFF) signaller(p, wave - 4, lane, call, cnt);
        else if (wave < 8 + EXP_SIGOFF) releaser(p, lane, call, cnt);
    } else if (slot < 30) {
        const int i = slot - 4, h = xcd >> 2;
        const int tm = i < 24 ? 7 * h + i % 6 : 6;
        const int tn = (xcd & 3) * 4 + (i < 24 ? i / 6 : 2 * h + i - 24);
        if (i < 16) stage_own(p, xcd * 16 + i, wave, lane, stage_off, smem + wave * 16384);
        gemm(p, tm, tn, smem, call, stage_off);
    }
    __syncthreads();
    if (tid == 0) __hip_atomic_fetch_add((gu32*)done, 1u, __ATOMIC_RELAXED, __HIP_MEMORY_SCOPE_AGENT);
}

}

extern "C" void kernel_launch(int rank, int nranks, ncclComm_t comm,
                              hipStream_t stream,
                              void* const* d_in, const int* in_sizes, int n_in,
                              void* d_out, int out_size,
                              const int* in_dtypes, int out_dtype,
                              void* const* d_out_all,
                              void* const* d_ws_all, size_t ws_bytes) {
    (void)comm; (void)in_dtypes; (void)out_dtype; (void)d_out_all;
    if (nranks != NR || (unsigned)rank >= (unsigned)NR || n_in < 2 || ws_bytes < WS_NEED ||
        in_sizes[0] != MM * KSH || in_sizes[1] != KSH * NN || out_size != MM * NN) {
        fprintf(stderr, "unexpected problem shape\n");
        return;
    }
    static bool attr_set = false;
    if (!attr_set) {
        HIP_CHECK(hipFuncSetAttribute((const void*)fused_gemm_allgather_silu,
                                      hipFuncAttributeMaxDynamicSharedMemorySize, LDS_BYTES));
        attr_set = true;
    }
    Params p;
    p.x = (const float*)d_in[0];
    p.w = (const float*)d_in[1];
    p.out = (float*)d_out;
    for (int i = 0; i < NR; ++i) p.ws[i] = (char*)d_ws_all[i];
    p.rank = rank;
    fused_gemm_allgather_silu<<<NWG, 512, LDS_BYTES, stream>>>(p);
    HIP_CHECK(hipGetLastError());
}
```

```cpp
#include <hip/hip_runtime.h>
#include <rccl/rccl.h>
#include <cstdio>

#define HIP_CHECK(expr) do { hipError_t _e = (expr); if (_e != hipSuccess) { \
    fprintf(stderr, "HIP error %s at %s:%d\n", hipGetErrorString(_e), __FILE__, __LINE__); \
    return; } } while (0)

#define LD_AUX 17
#define EXP_SIGOFF 0
#define ST_AUX 16

namespace {

constexpr int NR = 8;
constexpr int MM = 4096, NN = 4096, KSH = 512;
constexpr int NSTEP = 64;
constexpr int NCOMM = 32;
constexpr int NSTAGE = 16;
constexpr int NWG = 256;
constexpr int TM = 320, TN = 256;
#define THROTTLE 5
#define ILV_E 1
#define ILV_L 0
#define LA 1
#define PSLEEP 4
#define PEEKW 3
#define LATE2_T 12
#define LATE_T 4

constexpr size_t OFF_DONE = 0;
constexpr size_t OFF_TMO = 128;
constexpr size_t OFF_LC = 4096;
constexpr size_t OFF_LCL = 12288;
constexpr size_t OFF_FLAG = 16384;
constexpr size_t OFF_DBG = 131072;
constexpr size_t CTRL_BYTES = 1u << 20;
constexpr size_t CH_BYTES = 65536;
constexpr size_t OP_BYTES = (size_t)NSTEP * NR * CH_BYTES;
constexpr size_t WS_NEED = CTRL_BYTES + 4 * OP_BYTES;
static_assert(OFF_LC + NSTEP * 128 <= OFF_LCL && OFF_LCL + 8 * 128 <= OFF_FLAG, "layout");
static_assert(OFF_FLAG + NSTEP * NR * 128 <= OFF_DBG, "layout");

constexpr int A_STAGE = 8 * TM * 16;
constexpr int B_STAGE = 8 * TN * 16;
constexpr int STAGE = A_STAGE + B_STAGE;
constexpr int LDS_BYTES = 2 * STAGE;

constexpr unsigned long long DEADLINE_TICKS = 50000000ull;

typedef unsigned u32x4 __attribute__((ext_vector_type(4)));
typedef float f32x4 __attribute__((ext_vector_type(4)));
typedef _Float16 h16x8 __attribute__((ext_vector_type(8)));
typedef _Float16 h16x2 __attribute__((ext_vector_type(2)));
typedef __attribute__((address_space(1))) unsigned gu32;
typedef __attribute__((address_space(1))) char gchar;

struct Params {
    const float* x;
    const float* w;
    float* out;
    char* ws[NR];
    char* own;
    int rank;
};

template <class T>
__device__ __forceinline__ T* uni(T* p) {
    unsigned long long v = (unsigned long long)p;
    unsigned lo = __builtin_amdgcn_readfirstlane((unsigned)v);
    unsigned hi = __builtin_amdgcn_readfirstlane((unsigned)(v >> 32));
    return (T*)(((unsigned long long)hi << 32) | lo);
}
__device__ __forceinline__ unsigned pk16(float a, float b) {
    h16x2 v;
    v.x = (_Float16)a;
    v.y = (_Float16)b;
    return __builtin_bit_cast(unsigned, v);
}
typedef float f32x2 __attribute__((ext_vector_type(2)));
__device__ __forceinline__ f32x4 silu4(f32x4 v, f32x2 k) {
    const float lim = 60.0f * 1.44269504088896340736f;
    f32x2 a = f32x2{v.x, v.y} * k, b = f32x2{v.z, v.w} * k;
    a.x = __builtin_amdgcn_exp2f(__builtin_amdgcn_fmed3f(a.x, -lim, lim));
    a.y = __builtin_amdgcn_exp2f(__builtin_amdgcn_fmed3f(a.y, -lim, lim));
    b.x = __builtin_amdgcn_exp2f(__builtin_amdgcn_fmed3f(b.x, -lim, lim));
    b.y = __builtin_amdgcn_exp2f(__builtin_amdgcn_fmed3f(b.y, -lim, lim));
    a = a + 1.0f;
    b = b + 1.0f;
    a.x = __builtin_amdgcn_rcpf(a.x); a.y = __builtin_amdgcn_rcpf(a.y);
    b.x = __builtin_amdgcn_rcpf(b.x); b.y = __builtin_amdgcn_rcpf(b.y);
    a = a * f32x2{v.x, v.y};
    b = b * f32x2{v.z, v.w};
    return f32x4{a.x, a.y, b.x, b.y};
}
__device__ __forceinline__ unsigned ld_sys(const unsigned* p) {
    return __hip_atomic_load((gu32*)p, __ATOMIC_RELAXED, __HIP_MEMORY_SCOPE_SYSTEM);
}
__device__ __forceinline__ unsigned ld_agent(const unsigned* p) {
    return __hip_atomic_load((gu32*)p, __ATOMIC_RELAXED, __HIP_MEMORY_SCOPE_AGENT);
}
__device__ __forceinline__ void st_agent(unsigned* p, unsigned v) {
    __hip_atomic_store((gu32*)p, v, __ATOMIC_RELAXED, __HIP_MEMORY_SCOPE_AGENT);
}
__device__ __forceinline__ void st_sys(unsigned* p, unsigned v) {
    __hip_atomic_store((gu32*)p, v, __ATOMIC_RELAXED, __HIP_MEMORY_SCOPE_SYSTEM);
}
__device__ __forceinline__ unsigned get_call(const Params& p) {
    return __builtin_amdgcn_readfirstlane(ld_agent((const unsigned*)(p.own + OFF_DONE))) / NWG + 1;
}
__device__ __forceinline__ size_t stage_of(unsigned call) {
    return CTRL_BYTES + (size_t)(call & 1) * 2 * OP_BYTES;
}
struct FlagTrack {
    const char* f0;
    int stride;
    unsigned lim, call;
    __device__ __forceinline__ unsigned peek(int i0, int lane, unsigned& thr) const {
        int g = i0 + (lane & 7);
        g = g < NSTEP ? g : NSTEP - 1;
        thr = lim;
        unsigned v = lim - 1;
        if (lane < LA) v = ld_sys((const unsigned*)(f0 + g * stride));
        return v;
    }
};
struct StepTrack {
    const char* ws;
    unsigned call;
    int wave, rank;
    __device__ __forceinline__ unsigned peek(int i0, int lane, unsigned& thr) const {
        int s = i0 + (lane & 7);
        s = s < NSTEP ? s : NSTEP - 1;
        const int q = 8 * (s - 8) + wave;
        const int j = (q * 9363) >> 16;
        const int src = (rank + 1 + q - 7 * j) & 7;
        const int off = s < 8 ? (int)OFF_LCL + s * 128 : (int)OFF_FLAG + (j * NR + src) * 128;
        thr = s < 8 ? call * NSTAGE : call;
        unsigned v = 0;
        if (lane < LA) v = ld_sys((const unsigned*)(ws + off));
        return v;
    }
};
__device__ __forceinline__ int flag_count(unsigned v, unsigned thr) {
    unsigned m = (unsigned)__builtin_amdgcn_ballot_w64((int)(v - thr) >= 0) & ((1u << LA) - 1);
    return __builtin_ctz(~m);
}
template <class T>
__device__ __forceinline__ int wait_step(const T& tr, int g0, int lane, unsigned* tmo) {
    unsigned thr;
    unsigned v = tr.peek(g0, lane, thr);
    int c = flag_count(v, thr);
    if (c > 0) return g0 + c;
    unsigned long long t0 = __builtin_amdgcn_s_memrealtime();
    for (unsigned it = 1;; ++it) {
        v = tr.peek(g0, lane, thr);
        c = flag_count(v, thr);
        if (c > 0) return g0 + c;
        if ((it & 15) == 0) {
            if (__builtin_amdgcn_readfirstlane(ld_agent(tmo)) == tr.call ||
                __builtin_amdgcn_s_memrealtime() - t0 > DEADLINE_TICKS) {
                st_agent(tmo, tr.call);
                return NSTEP + 8;
            }
        }
        __builtin_amdgcn_s_sleep(PSLEEP);
    }
}

#define STAMP(idx) do { } while (0)

__device__ __forceinline__ void pusher(const Params& p, int piece, int phase, int lane,
                                       unsigned* cnt, char* ring) {
    const char* srcx = (const char*)(p.x + (size_t)piece * 64 * KSH) + lane * (KSH * 4);
    const char* srcw = (const char*)(p.w + piece * 64) + (lane >> 4) * (NN * 4) + (lane & 15) * 16;
    auto issue_load = [&](int j, int slot) {
        char* l = ring + slot * 2048;
        const bool isw = (j + phase) & 1;
        const char* g0 = isw ? srcw + (size_t)j * (8 * NN * 4) : srcx + j * 32;
        const char* g1 = g0 + (isw ? 4 * NN * 4 : 16);
        __builtin_amdgcn_global_load_lds((const gu32*)g0, (unsigned*)l, 16, 0, 0);
        __builtin_amdgcn_global_load_lds((const gu32*)g1, (unsigned*)(l + 1024), 16, 0, 0);
    };
    const unsigned cnt_addr = (unsigned)(size_t)(__attribute__((address_space(3))) unsigned*)cnt;
    auto signal = [&](int j) {
        if (lane == 0)
            asm volatile("ds_add_u32 %0, %1" :: "v"(cnt_addr + j * 4), "v"(1u) : "memory");
    };
    const char* base[NR - 1];
    const unsigned vbase = (unsigned)(p.rank * CH_BYTES + piece * 1024 + lane * 16);
    auto push = [&](int j, int slot) {
        const char* l = ring + slot * 2048;
        const bool isw = (j + phase) & 1;
        float s[8];
        if (isw) {
#pragma unroll
            for (int q = 0; q < 8; ++q) s[q] = *(const float*)(l + q * 256 + lane * 4);
        } else {
            *(f32x4*)&s[0] = *(const f32x4*)(l + lane * 16);
            *(f32x4*)&s[4] = *(const f32x4*)(l + 1024 + lane * 16);
        }
        u32x4 d;
        d.x = pk16(s[0], s[1]); d.y = pk16(s[2], s[3]); d.z = pk16(s[4], s[5]); d.w = pk16(s[6], s[7]);
        unsigned vo = vbase + (unsigned)j * (unsigned)(NR * CH_BYTES) + (isw ? (unsigned)OP_BYTES : 0u);
#pragma unroll
        for (int q = 0; q < NR - 1; ++q)
            asm volatile("global_store_dwordx4 %0, %1, %2 sc0 sc1\n\ts_nop 1" :: "v"(vo), "v"(d), "s"(base[q]) : "memory");
    };
#define PUSH_WAIT(N) asm volatile("s_waitcnt vmcnt(" #N ")" ::: "memory")
    auto throttle = [&](int j) {
        if (j < THROTTLE) return;
        const unsigned need = j - THROTTLE + 1;
        unsigned v;
        asm volatile("ds_read_b32 %0, %1\n\ts_waitcnt lgkmcnt(0)" : "=v"(v) : "v"(cnt_addr + NSTEP * 4) : "memory");
        if (__builtin_amdgcn_readfirstlane(v) >= need) return;
        unsigned long long t0 = __builtin_amdgcn_s_memrealtime();
        do {
            __builtin_amdgcn_s_sleep(2);
            asm volatile("ds_read_b32 %0, %1\n\ts_waitcnt lgkmcnt(0)" : "=v"(v) : "v"(cnt_addr + NSTEP * 4) : "memory");
        } while (__builtin_amdgcn_readfirstlane(v) < need &&
                 __builtin_amdgcn_s_memrealtime() - t0 < DEADLINE_TICKS);
    };
    issue_load(0, 0);
    issue_load(1, 1);
    issue_load(2, 2);
    unsigned long long wsv[NR];
#pragma unroll
    for (int k = 0; k < NR; ++k) {
        wsv[k] = (unsigned long long)p.ws[k];
        asm volatile("" : "+s"(wsv[k]));
    }
    char* peer[NR - 1];
#pragma unroll
    for (int i = 0; i < NR - 1; ++i) {
        const int q = (p.rank + 1 + (i + piece) % 7) & 7;
        unsigned long long r = wsv[0];
#pragma unroll
        for (int k = 1; k < NR; ++k) r = q == k ? wsv[k] : r;
        peer[i] = (char*)r;
    }
    const size_t stage_off = stage_of(get_call(p));
#pragma unroll
    for (int i = 0; i < NR - 1; ++i) {
        base[i] = uni(peer[i] + stage_off);
    }
    asm volatile("s_nop 4" ::: "memory");
    PUSH_WAIT(4); push(0, 0);
    issue_load(3, 0); PUSH_WAIT(2); signal(0); push(1, 1);
#pragma unroll 1
    for (int j0 = 2; j0 < 62; j0 += 3) {
        issue_load(j0 + 2, 1); PUSH_WAIT(2); signal(j0 - 1); throttle(j0); push(j0, 2);
        issue_load(j0 + 3, 2); PUSH_WAIT(2); signal(j0); throttle(j0 + 1); push(j0 + 1, 0);
        issue_load(j0 + 4, 0); PUSH_WAIT(2); signal(j0 + 1); throttle(j0 + 2); push(j0 + 2, 1);
    }
    PUSH_WAIT(0); signal(61); throttle(62); push(62, 2);
    PUSH_WAIT(0); signal(62); throttle(63); push(63, 0);
    PUSH_WAIT(0); signal(63);
#undef PUSH_WAIT
}

constexpr int NSIG = 3;
__device__ __forceinline__ void signaller(const Params& p, int which, int lane, unsigned call, unsigned* cnt) {
    unsigned* tmo = (unsigned*)(p.own + OFF_TMO);
    for (int g = which; g < NSTEP; g += NSIG) {
        unsigned long long t0 = __builtin_amdgcn_s_memrealtime();
        bool ok = true;
        while (__hip_atomic_load(&cnt[g], __ATOMIC_RELAXED, __HIP_MEMORY_SCOPE_WORKGROUP) < 4u) {
            if (__builtin_amdgcn_s_memrealtime() - t0 > DEADLINE_TICKS) { ok = false; break; }
            __builtin_amdgcn_s_sleep(1);
        }
        if (!ok) {
            st_agent(tmo, call);
            break;
        }
        unsigned old = 0;
        if (lane == 0)
            old = __hip_atomic_fetch_add((gu32*)(p.own + OFF_LC + g * 128), 1u, __ATOMIC_RELAXED, __HIP_MEMORY_SCOPE_AGENT);
        old = __builtin_amdgcn_readfirstlane(old);
        STAMP(2 + g);
        if (((old + 1) & (NCOMM - 1)) == 0 && lane < NR) {
            unsigned* f = (unsigned*)(p.ws[lane] + OFF_FLAG + (g * NR + p.rank) * 128);
            st_sys(f, call);
        }
    }
}
__device__ __forceinline__ void releaser(const Params& p, int lane, unsigned call, unsigned* cnt) {
    unsigned* tmo = (unsigned*)(p.own + OFF_TMO);
    const FlagTrack tr{p.own + OFF_FLAG + p.rank * 128, NR * 128, call, call};
    int have = 0;
    while (have < NSTEP - THROTTLE) {
        have = wait_step(tr, have, lane, tmo);
        if (lane == 0) __hip_atomic_store(&cnt[NSTEP], (unsigned)have, __ATOMIC_RELAXED, __HIP_MEMORY_SCOPE_WORKGROUP);
    }
}

__device__ __forceinline__ unsigned stage_own(const Params& p, int g, int wave, int lane, char* l) {
    const int b = g & 7;
    const int v = (g >> 3) * 8 + wave;
    const int piece = v >> 1;
    const bool isw = v & 1;
    const int vo = p.rank * (int)CH_BYTES + piece * 1024 + lane * 16;
    const int r = lane >> 4, c = lane & 15;
    const char* src = isw ? (const char*)(p.w + (size_t)(b * 64 + r) * NN + piece * 64)
                          : (const char*)(p.x + (size_t)(piece * 64 + r) * KSH + b * 64);
    const int rstride = isw ? 4 * NN * 4 : 4 * KSH * 4;
#pragma unroll
    for (int i = 0; i < 16; ++i) {
        const int slot = isw ? c : (c ^ ((4 * i + r) & 15));
        __builtin_amdgcn_global_load_lds((const gu32*)(src + (size_t)i * rstride + slot * 16),
                                         (unsigned*)(l + i * 1024), 16, 0, 0);
    }
    const unsigned call = get_call(p);
    const __amdgpu_buffer_rsrc_t rd = __builtin_amdgcn_make_buffer_rsrc(
        p.own + stage_of(call) + (isw ? OP_BYTES : 0), 0, (int)OP_BYTES, 0x00020000);
    asm volatile("s_waitcnt vmcnt(0)" ::: "memory");
    float cur[64];
    if (isw) {
#pragma unroll
        for (int k = 0; k < 64; ++k) cur[k] = *(const float*)(l + k * 256 + lane * 4);
    } else {
#pragma unroll
        for (int k = 0; k < 16; ++k) *(f32x4*)&cur[4 * k] = *(const f32x4*)(l + lane * 256 + ((k ^ (lane & 15)) * 16));
    }
#pragma unroll
    for (int o = 0; o < 8; ++o) {
        u32x4 d;
        d.x = pk16(cur[8 * o], cur[8 * o + 1]); d.y = pk16(cur[8 * o + 2], cur[8 * o + 3]);
        d.z = pk16(cur[8 * o + 4], cur[8 * o + 5]); d.w = pk16(cur[8 * o + 6], cur[8 * o + 7]);
        __builtin_amdgcn_raw_buffer_store_b128(d, rd, vo, (b * 8 + o) * (int)(NR * CH_BYTES), 17);
    }
    asm volatile("s_waitcnt vmcnt(0)" ::: "memory");
    __syncthreads();
    if (threadIdx.x == 0)
        __hip_atomic_fetch_add((gu32*)(p.own + OFF_LCL + b * 128), 1u, __ATOMIC_RELAXED, __HIP_MEMORY_SCOPE_AGENT);
    return call;
}

__device__ __forceinline__ void gemm(const Params& p, int tm, int tn, char* smem, unsigned call, size_t stage_off) {
    const int tid = threadIdx.x, lane = tid & 63;
    const int wave = __builtin_amdgcn_readfirstlane(tid >> 6);
    const int wr = wave >> 2, wc = wave & 3;
    const int row0 = tm * TM, col0 = tn * TN;
    char* ws = uni(p.own);
    unsigned* tmo = (unsigned*)(ws + OFF_TMO);
    const StepTrack tr{ws, call, wave, p.rank};
    const __amdgpu_buffer_rsrc_t rs =
        __builtin_amdgcn_make_buffer_rsrc(ws + stage_off, 0, (int)(2 * OP_BYTES), 0x00020000);
    const int vx = row0 * 16 + lane * 16;
    const int vw = (int)OP_BYTES + col0 * 16 + lane * 16;
    const bool full = row0 + TM <= MM;

    auto plane = [&](int st) {
        const int q = 8 * (st - 8) + wave;
        const int jj = (q * 9363) >> 16;
        return st < 8 ? ((8 * st + wave) * NR + p.rank) * (int)CH_BYTES
                      : (jj * NR + ((p.rank + 1 + q - 7 * jj) & 7)) * (int)CH_BYTES;
    };
    auto stage_in = [&](int j, int so) {
        __attribute__((address_space(3))) char* la =
            (__attribute__((address_space(3))) char*)(smem + (j & 1) * STAGE + wave * (TM * 16));
        __attribute__((address_space(3))) char* lb =
            (__attribute__((address_space(3))) char*)(smem + (j & 1) * STAGE + A_STAGE + wave * (TN * 16));
        __builtin_amdgcn_raw_ptr_buffer_load_lds(rs, la, 16, vx, so, 0, LD_AUX);
        __builtin_amdgcn_raw_ptr_buffer_load_lds(rs, la, 16, vx, so, 1024, LD_AUX);
        __builtin_amdgcn_raw_ptr_buffer_load_lds(rs, la, 16, vx, so, 2048, LD_AUX);
        __builtin_amdgcn_raw_ptr_buffer_load_lds(rs, la, 16, vx, so, 3072, LD_AUX);
        if (full)
            __builtin_amdgcn_raw_ptr_buffer_load_lds(rs, la + 4096, 16, vx + 4096, so, 0, LD_AUX);
        __builtin_amdgcn_raw_ptr_buffer_load_lds(rs, lb, 16, vw, so, 0, LD_AUX);
        __builtin_amdgcn_raw_ptr_buffer_load_lds(rs, lb, 16, vw, so, 1024, LD_AUX);
        __builtin_amdgcn_raw_ptr_buffer_load_lds(rs, lb, 16, vw, so, 2048, LD_AUX);
        __builtin_amdgcn_raw_ptr_buffer_load_lds(rs, lb, 16, vw, so, 3072, LD_AUX);
    };

    auto stage_one = [&](int j, int i, int so) {
        __attribute__((address_space(3))) char* la =
            (__attribute__((address_space(3))) char*)(smem + (j & 1) * STAGE + wave * (TM * 16));
        __attribute__((address_space(3))) char* lb =
            (__attribute__((address_space(3))) char*)(smem + (j & 1) * STAGE + A_STAGE + wave * (TN * 16));
        if (i < 4) __builtin_amdgcn_raw_ptr_buffer_load_lds(rs, la + i * 1024, 16, vx + i * 1024, so, 0, LD_AUX);
        else if (i == 4) { if (full) __builtin_amdgcn_raw_ptr_buffer_load_lds(rs, la + 4096, 16, vx + 4096, so, 0, LD_AUX); }
        else __builtin_amdgcn_raw_ptr_buffer_load_lds(rs, lb + (i - 5) * 1024, 16, vw + (i - 5) * 1024, so, 0, LD_AUX);
    };

    f32x4 acc[10][4];
#pragma unroll
    for (int m = 0; m < 10; ++m)
#pragma unroll
        for (int n = 0; n < 4; ++n) acc[m][n] = f32x4{0.f, 0.f, 0.f, 0.f};

    const int fr = lane & 15, fq = lane >> 4;
    const int xoff = fq * (TM * 16) + (wr * 160 + fr) * 16;
    const int woff = A_STAGE + fq * (TN * 16) + (wc * 64 + fr) * 16;

    int ready;
    ready = wait_step(tr, 0, lane, tmo);
    stage_in(0, plane(0));
    asm volatile("s_waitcnt vmcnt(0)" ::: "memory");
    __builtin_amdgcn_s_barrier();
    if (wave == 0) STAMP(1);

    for (int j = 0; j < NSTEP - 1; ++j) {
        constexpr bool more = true;
        const bool pre = more && j + 1 < ready;
        const bool peek = ready < NSTEP && ready < j + PEEKW;
        const int ready0 = ready;
        unsigned peekv = 0, peekt = 0, peek2;
        asm volatile("v_mov_b32 %0, 0" : "=v"(peek2));
        if (peek) peekv = tr.peek(ready0, lane, peekt);
        bool late = false;
        const int so1 = __builtin_amdgcn_readfirstlane(plane(j + 1));
        const bool preE = pre && wr == 0, preL = pre && wr != 0;
        if (ILV_L == 0 && preL) stage_in(j + 1, so1);
        const char* sx = smem + (j & 1) * STAGE + xoff;
        const char* sw = smem + (j & 1) * STAGE + woff;
        auto ldx = [&](int t) { return *(const h16x8*)(sx + (t / 10) * 4 * (TM * 16) + (t % 10) * 256); };
        auto ldw = [&](int s, int n) { return *(const h16x8*)(sw + s * 4 * (TN * 16) + n * 256); };
        h16x8 wf[2][4], xf[4];
        __builtin_amdgcn_sched_barrier(0);
        wf[0][0] = ldw(0, 0); xf[0] = ldx(0);
        __builtin_amdgcn_sched_barrier(0);
        wf[0][1] = ldw(0, 1); wf[0][2] = ldw(0, 2); wf[0][3] = ldw(0, 3);
        xf[1] = ldx(1); xf[2] = ldx(2);
#pragma unroll
        for (int t = 0; t < 20; ++t) {
            __builtin_amdgcn_sched_barrier(0);
            acc[t % 10][0] = __builtin_amdgcn_mfma_f32_16x16x32_f16(wf[t / 10][0], xf[t & 3], acc[t % 10][0], 0, 0, 0);
            __builtin_amdgcn_sched_barrier(0);
            if (t + 3 < 20) xf[(t + 3) & 3] = ldx(t + 3);
            if (t >= 2 && t < 6) wf[1][t - 2] = ldw(1, t - 2);
#pragma unroll
            for (int i = 0; i < 9; ++i) {
                if (i / ILV_E == t && preE) stage_one(j + 1, i, so1);
                if (ILV_L > 0 && i / (ILV_L > 0 ? ILV_L : 1) == t && preL) stage_one(j + 1, i, so1);
            }
            if (t == LATE_T && more && !pre) {
                asm volatile("" : "+v"(peekv));
                ready = ready0 + flag_count(peekv, peekt);
                late = j + 1 < ready;
                if (late) stage_in(j + 1, so1);
            }
            if (t == LATE_T + 1 && more && !pre && !late) {
                unsigned th2;
                peek2 = tr.peek(ready0, lane, th2);
            }
            if (t == LATE2_T && more && !pre && !late) {
                asm volatile("" : "+v"(peek2));
                ready = ready0 + flag_count(peek2, peekt);
                late = j + 1 < ready;
                if (late) stage_in(j + 1, so1);
            }
            __builtin_amdgcn_sched_barrier(0);
#pragma unroll
            for (int n = 1; n < 4; ++n)
                acc[t % 10][n] = __builtin_amdgcn_mfma_f32_16x16x32_f16(wf[t / 10][n], xf[t & 3], acc[t % 10][n], 0, 0, 0);
        }
        __builtin_amdgcn_sched_barrier(0);
        {
            const int c = flag_count(peekv, peekt);
            if (peek && ready0 + c > ready) ready = ready0 + c;
        }
        if (more && !pre && !late) {
            if (j + 1 >= ready) ready = wait_step(tr, j + 1, lane, tmo);
            stage_in(j + 1, so1);
        }
        asm volatile("s_waitcnt vmcnt(0) lgkmcnt(0)" ::: "memory");
        __builtin_amdgcn_s_barrier();
    }

    if (wave == 0) STAMP(2 + NSTEP - 1);
    const unsigned tmo_v = ld_agent(tmo);
    const int mlim = full ? 10 : (MM - row0 - wr * 160 + 15) / 16;
    const __amdgpu_buffer_rsrc_t ro =
        __builtin_amdgcn_make_buffer_rsrc(p.out, 0, (int)((size_t)MM * NN * 4), 0x00020000);
    const int vo = ((row0 + wr * 160 + fr) * NN + col0 + wc * 64 + fq * 4) * 4;
    f32x2 kneg = {-1.44269504088896340736f, -1.44269504088896340736f};
    asm volatile("" : "+v"(kneg));
    bool bad = false;
    {
        const char* sx = smem + ((NSTEP - 1) & 1) * STAGE + xoff;
        const char* sw = smem + ((NSTEP - 1) & 1) * STAGE + woff;
        auto ldx = [&](int s, int m) { return *(const h16x8*)(sx + s * 4 * (TM * 16) + m * 256); };
        auto ldw = [&](int s, int n) { return *(const h16x8*)(sw + s * 4 * (TN * 16) + n * 256); };
        auto store_group = [&](int m) {
            if (!bad && m < mlim) {
#pragma unroll
                for (int n = 0; n < 4; ++n)
                    __builtin_amdgcn_raw_buffer_store_b128(__builtin_bit_cast(u32x4, silu4(acc[m][n], kneg)), ro,
                                                           vo + m * (16 * NN * 4) + n * 64, 0, ST_AUX);
            }
        };
        h16x8 wf[2][4], xf[2][2];
        __builtin_amdgcn_sched_barrier(0);
        wf[0][0] = ldw(0, 0); xf[0][0] = ldx(0, 0);
        __builtin_amdgcn_sched_barrier(0);
        wf[0][1] = ldw(0, 1); wf[0][2] = ldw(0, 2); wf[0][3] = ldw(0, 3);
        xf[0][1] = ldx(1, 0);
        wf[1][0] = ldw(1, 0); wf[1][1] = ldw(1, 1); wf[1][2] = ldw(1, 2); wf[1][3] = ldw(1, 3);
#pragma unroll
        for (int m = 0; m < 10; ++m) {
            __builtin_amdgcn_sched_barrier(0);
            if (m + 1 < 10) { xf[(m + 1) & 1][0] = ldx(0, m + 1); xf[(m + 1) & 1][1] = ldx(1, m + 1); }
            __builtin_amdgcn_sched_barrier(0);
#pragma unroll
            for (int s = 0; s < 2; ++s)
#pragma unroll
                for (int n = 0; n < 4; ++n)
                    acc[m][n] = __builtin_amdgcn_mfma_f32_16x16x32_f16(wf[s][n], xf[m & 1][s], acc[m][n], 0, 0, 0);
            __builtin_amdgcn_sched_barrier(0);
            if (m == 1) bad = __builtin_amdgcn_readfirstlane(tmo_v) == call;
            if (m >= 1) store_group(m - 1);
        }
        __builtin_amdgcn_sched_barrier(0);
        if (wave == 0) STAMP(68);
        store_group(9);
    }
    if (bad) {
        const u32x4 nan4 = {0x7fc00000u, 0x7fc00000u, 0x7fc00000u, 0x7fc00000u};
        for (int m = 0; m < mlim; ++m)
            for (int n = 0; n < 4; ++n)
                __builtin_amdgcn_raw_buffer_store_b128(nan4, ro, vo + m * (16 * NN * 4) + n * 64, 0, 0);
    }
}

__global__ __launch_bounds__(512, 2) void fused_gemm_allgather_silu(Params p) {
    extern __shared__ __attribute__((aligned(16))) char smem[];
    const int tid = threadIdx.x, lane = tid & 63;
    const int wave = __builtin_amdgcn_readfirstlane(tid >> 6);
    unsigned* done = (unsigned*)(p.own + OFF_DONE);
    const int xcd = blockIdx.x & 7, slot = blockIdx.x >> 3;
    if (wave == 0) STAMP(0);
    if (slot < 4) {
        const int cb = slot * 8 + xcd;
        unsigned* cnt = (unsigned*)smem;
        if (tid < 128) cnt[tid] = 0;
        __syncthreads();
        if (wave < 4) pusher(p, cb * 2 + (wave & 1), wave >> 1, lane, cnt, smem + 1024 + wave * 6144);
        else if (wave < 4 + NSIG + EXP_SIGOFF) signaller(p, wave - 4, lane, get_call(p), cnt);
        else if (wave < 8 + EXP_SIGOFF) releaser(p, lane, get_call(p), cnt);
    } else if (slot < 30) {
        const int i = slot - 4, h = xcd >> 2;
        const int tm = i < 24 ? 7 * h + i % 6 : 6;
        const int tn = (xcd & 3) * 4 + (i < 24 ? i / 6 : 2 * h + i - 24);
        const unsigned call = i < 16 ? stage_own(p, xcd * 16 + i, wave, lane, smem + wave * 16384) : get_call(p);
        gemm(p, tm, tn, smem, call, stage_of(call));
    }
    __syncthreads();
    if (tid == 0) __hip_atomic_fetch_add((gu32*)done, 1u, __ATOMIC_RELAXED, __HIP_MEMORY_SCOPE_AGENT);
}

}

extern "C" void kernel_launch(int rank, int nranks, ncclComm_t comm,
                              hipStream_t stream,
                              void* const* d_in, const int* in_sizes, int n_in,
                              void* d_out, int out_size,
                              const int* in_dtypes, int out_dtype,
                              void* const* d_out_all,
                              void* const* d_ws_all, size_t ws_bytes) {
    (void)comm; (void)in_dtypes; (void)out_dtype; (void)d_out_all;
    if (nranks != NR || (unsigned)rank >= (unsigned)NR || n_in < 2 || ws_bytes < WS_NEED ||
        in_sizes[0] != MM * KSH || in_sizes[1] != KSH * NN || out_size != MM * NN) {
        fprintf(stderr, "unexpected problem shape\n");
        return;
    }
    static bool attr_set = false;
    if (!attr_set) {
        HIP_CHECK(hipFuncSetAttribute((const void*)fused_gemm_allgather_silu,
                                      hipFuncAttributeMaxDynamicSharedMemorySize, LDS_BYTES));
        attr_set = true;
    }
    Params p;
    p.x = (const float*)d_in[0];
    p.w = (const float*)d_in[1];
    p.out = (float*)d_out;
    for (int i = 0; i < NR; ++i) p.ws[i] = (char*)d_ws_all[i];
    p.own = p.ws[rank];
    p.rank = rank;
    fused_gemm_allgather_silu<<<NWG, 512, LDS_BYTES, stream>>>(p);
    HIP_CHECK(hipGetLastError());
}
```

```cpp
#include <hip/hip_runtime.h>
#include <rccl/rccl.h>
#include <cstdio>

#define HIP_CHECK(expr) do { hipError_t _e = (expr); if (_e != hipSuccess) { \
    fprintf(stderr, "HIP error %s at %s:%d\n", hipGetErrorString(_e), __FILE__, __LINE__); \
    return; } } while (0)

#define LD_AUX 17
#define EXP_SIGOFF 0
#define ST_AUX 16

namespace {

constexpr int NR = 8;
constexpr int MM = 4096, NN = 4096, KSH = 512;
constexpr int NSTEP = 64;
constexpr int NCOMM = 32;
constexpr int NSTAGE = 16;
constexpr int NWG = 256;
constexpr int TM = 320, TN = 256;
#define THROTTLE 4
#define ILV_E 1
#define ILV_L 0
#define LA 1
#define PSLEEP 4
#define PEEKW 3
#define LATE2_T 12
#define LATE_T 4

constexpr size_t OFF_DONE = 0;
constexpr size_t OFF_TMO = 128;
constexpr size_t OFF_LC = 4096;
constexpr size_t OFF_LCL = 12288;
constexpr size_t OFF_FLAG = 16384;
constexpr size_t OFF_DBG = 131072;
constexpr size_t CTRL_BYTES = 1u << 20;
constexpr size_t CH_BYTES = 65536;
constexpr size_t OP_BYTES = (size_t)NSTEP * NR * CH_BYTES;
constexpr size_t WS_NEED = CTRL_BYTES + 4 * OP_BYTES;
static_assert(OFF_LC + NSTEP * 128 <= OFF_LCL && OFF_LCL + 8 * 128 <= OFF_FLAG, "layout");
static_assert(OFF_FLAG + NSTEP * NR * 128 <= OFF_DBG, "layout");

constexpr int A_STAGE = 8 * TM * 16;
constexpr int B_STAGE = 8 * TN * 16;
constexpr int STAGE = A_STAGE + B_STAGE;
constexpr int LDS_BYTES = 2 * STAGE;

constexpr unsigned long long DEADLINE_TICKS = 50000000ull;

typedef unsigned u32x4 __attribute__((ext_vector_type(4)));
typedef float f32x4 __attribute__((ext_vector_type(4)));
typedef _Float16 h16x8 __attribute__((ext_vector_type(8)));
typedef _Float16 h16x2 __attribute__((ext_vector_type(2)));
typedef __attribute__((address_space(1))) unsigned gu32;
typedef __attribute__((address_space(1))) char gchar;

struct Params {
    const float* x;
    const float* w;
    float* out;
    char* ws[NR];
    char* own;
    int rank;
};

template <class T>
__device__ __forceinline__ T* uni(T* p) {
    unsigned long long v = (unsigned long long)p;
    unsigned lo = __builtin_amdgcn_readfirstlane((unsigned)v);
    unsigned hi = __builtin_amdgcn_readfirstlane((unsigned)(v >> 32));
    return (T*)(((unsigned long long)hi << 32) | lo);
}
__device__ __forceinline__ unsigned pk16(float a, float b) {
    h16x2 v;
    v.x = (_Float16)a;
    v.y = (_Float16)b;
    return __builtin_bit_cast(unsigned, v);
}
typedef float f32x2 __attribute__((ext_vector_type(2)));
__device__ __forceinline__ f32x4 silu4(f32x4 v, f32x2 k) {
    const float lim = 60.0f * 1.44269504088896340736f;
    f32x2 a = f32x2{v.x, v.y} * k, b = f32x2{v.z, v.w} * k;
    a.x = __builtin_amdgcn_exp2f(__builtin_amdgcn_fmed3f(a.x, -lim, lim));
    a.y = __builtin_amdgcn_exp2f(__builtin_amdgcn_fmed3f(a.y, -lim, lim));
    b.x = __builtin_amdgcn_exp2f(__builtin_amdgcn_fmed3f(b.x, -lim, lim));
    b.y = __builtin_amdgcn_exp2f(__builtin_amdgcn_fmed3f(b.y, -lim, lim));
    a = a + 1.0f;
    b = b + 1.0f;
    a.x = __builtin_amdgcn_rcpf(a.x); a.y = __builtin_amdgcn_rcpf(a.y);
    b.x = __builtin_amdgcn_rcpf(b.x); b.y = __builtin_amdgcn_rcpf(b.y);
    a = a * f32x2{v.x, v.y};
    b = b * f32x2{v.z, v.w};
    return f32x4{a.x, a.y, b.x, b.y};
}
__device__ __forceinline__ unsigned ld_sys(const unsigned* p) {
    return __hip_atomic_load((gu32*)p, __ATOMIC_RELAXED, __HIP_MEMORY_SCOPE_SYSTEM);
}
__device__ __forceinline__ unsigned ld_agent(const unsigned* p) {
    return __hip_atomic_load((gu32*)p, __ATOMIC_RELAXED, __HIP_MEMORY_SCOPE_AGENT);
}
__device__ __forceinline__ void st_agent(unsigned* p, unsigned v) {
    __hip_atomic_store((gu32*)p, v, __ATOMIC_RELAXED, __HIP_MEMORY_SCOPE_AGENT);
}
__device__ __forceinline__ void st_sys(unsigned* p, unsigned v) {
    __hip_atomic_store((gu32*)p, v, __ATOMIC_RELAXED, __HIP_MEMORY_SCOPE_SYSTEM);
}
__device__ __forceinline__ unsigned get_call(const Params& p) {
    return __builtin_amdgcn_readfirstlane(ld_agent((const unsigned*)(p.own + OFF_DONE))) / NWG + 1;
}
__device__ __forceinline__ size_t stage_of(unsigned call) {
    return CTRL_BYTES + (size_t)(call & 1) * 2 * OP_BYTES;
}
struct FlagTrack {
    const char* f0;
    int stride;
    unsigned lim, call;
    __device__ __forceinline__ unsigned peek(int i0, int lane, unsigned& thr) const {
        int g = i0 + (lane & 7);
        g = g < NSTEP ? g : NSTEP - 1;
        thr = lim;
        unsigned v = lim - 1;
        if (lane < LA) v = ld_sys((const unsigned*)(f0 + g * stride));
        return v;
    }
};
struct StepTrack {
    const char* ws;
    unsigned call;
    int wave, rank;
    __device__ __forceinline__ unsigned peek(int i0, int lane, unsigned& thr) const {
        int s = i0 + (lane & 7);
        s = s < NSTEP ? s : NSTEP - 1;
        const int q = 8 * (s - 8) + wave;
        const int j = (q * 9363) >> 16;
        const int src = (rank + 1 + q - 7 * j) & 7;
        const int off = s < 8 ? (int)OFF_LCL + s * 128 : (int)OFF_FLAG + (j * NR + src) * 128;
        thr = s < 8 ? call * NSTAGE : call;
        unsigned v = 0;
        if (lane < LA) v = ld_sys((const unsigned*)(ws + off));
        return v;
    }
};
__device__ __forceinline__ int flag_count(unsigned v, unsigned thr) {
    unsigned m = (unsigned)__builtin_amdgcn_ballot_w64((int)(v - thr) >= 0) & ((1u << LA) - 1);
    return __builtin_ctz(~m);
}
template <class T>
__device__ __forceinline__ int wait_step(const T& tr, int g0, int lane, unsigned* tmo) {
    unsigned thr;
    unsigned v = tr.peek(g0, lane, thr);
    int c = flag_count(v, thr);
    if (c > 0) return g0 + c;
    unsigned long long t0 = __builtin_amdgcn_s_memrealtime();
    for (unsigned it = 1;; ++it) {
        v = tr.peek(g0, lane, thr);
        c = flag_count(v, thr);
        if (c > 0) return g0 + c;
        if ((it & 15) == 0) {
            if (__builtin_amdgcn_readfirstlane(ld_agent(tmo)) == tr.call ||
                __builtin_amdgcn_s_memrealtime() - t0 > DEADLINE_TICKS) {
                st_agent(tmo, tr.call);
                return NSTEP + 8;
            }
        }
        __builtin_amdgcn_s_sleep(PSLEEP);
    }
}

#define STAMP(idx) do { } while (0)

__device__ __forceinline__ void pusher(const Params& p, int piece, int phase, int lane,
                                       unsigned* cnt, char* ring) {
    const char* srcx = (const char*)(p.x + (size_t)piece * 64 * KSH) + lane * (KSH * 4);
    const char* srcw = (const char*)(p.w + piece * 64) + (lane >> 4) * (NN * 4) + (lane & 15) * 16;
    auto issue_load = [&](int j, int slot) {
        char* l = ring + slot * 2048;
        const bool isw = (j + phase) & 1;
        const char* g0 = isw ? srcw + (size_t)j * (8 * NN * 4) : srcx + j * 32;
        const char* g1 = g0 + (isw ? 4 * NN * 4 : 16);
        __builtin_amdgcn_global_load_lds((const gu32*)g0, (unsigned*)l, 16, 0, 0);
        __builtin_amdgcn_global_load_lds((const gu32*)g1, (unsigned*)(l + 1024), 16, 0, 0);
    };
    const unsigned cnt_addr = (unsigned)(size_t)(__attribute__((address_space(3))) unsigned*)cnt;
    const char* lc = uni(p.own + OFF_LC);
    auto signal = [&](int j) {
        if (lane == 0)
            asm volatile("global_atomic_add %0, %1, %2" :: "v"(j * 128), "v"(1u), "s"(lc) : "memory");
    };
    const char* base[NR - 1];
    const unsigned vbase = (unsigned)(p.rank * CH_BYTES + piece * 1024 + lane * 16);
    auto push = [&](int j, int slot) {
        const char* l = ring + slot * 2048;
        const bool isw = (j + phase) & 1;
        float s[8];
        if (isw) {
#pragma unroll
            for (int q = 0; q < 8; ++q) s[q] = *(const float*)(l + q * 256 + lane * 4);
        } else {
            *(f32x4*)&s[0] = *(const f32x4*)(l + lane * 16);
            *(f32x4*)&s[4] = *(const f32x4*)(l + 1024 + lane * 16);
        }
        u32x4 d;
        d.x = pk16(s[0], s[1]); d.y = pk16(s[2], s[3]); d.z = pk16(s[4], s[5]); d.w = pk16(s[6], s[7]);
        unsigned vo = vbase + (unsigned)j * (unsigned)(NR * CH_BYTES) + (isw ? (unsigned)OP_BYTES : 0u);
#pragma unroll
        for (int q = 0; q < NR - 1; ++q)
            asm volatile("global_store_dwordx4 %0, %1, %2 sc0 sc1\n\ts_nop 1" :: "v"(vo), "v"(d), "s"(base[q]) : "memory");
    };
#define PUSH_WAIT(N) asm volatile("s_waitcnt vmcnt(" #N ")" ::: "memory")
    auto throttle = [&](int j) {
        if (j < THROTTLE) return;
        const unsigned need = j - THROTTLE + 1;
        unsigned v;
        asm volatile("ds_read_b32 %0, %1\n\ts_waitcnt lgkmcnt(0)" : "=v"(v) : "v"(cnt_addr) : "memory");
        if (__builtin_amdgcn_readfirstlane(v) >= need) return;
        unsigned long long t0 = __builtin_amdgcn_s_memrealtime();
        do {
            __builtin_amdgcn_s_sleep(2);
            asm volatile("ds_read_b32 %0, %1\n\ts_waitcnt lgkmcnt(0)" : "=v"(v) : "v"(cnt_addr) : "memory");
        } while (__builtin_amdgcn_readfirstlane(v) < need &&
                 __builtin_amdgcn_s_memrealtime() - t0 < DEADLINE_TICKS);
    };
    issue_load(0, 0);
    issue_load(1, 1);
    issue_load(2, 2);
    unsigned long long wsv[NR];
#pragma unroll
    for (int k = 0; k < NR; ++k) {
        wsv[k] = (unsigned long long)p.ws[k];
        asm volatile("" : "+s"(wsv[k]));
    }
    char* peer[NR - 1];
#pragma unroll
    for (int i = 0; i < NR - 1; ++i) {
        const int q = (p.rank + 1 + (i + piece) % 7) & 7;
        unsigned long long r = wsv[0];
#pragma unroll
        for (int k = 1; k < NR; ++k) r = q == k ? wsv[k] : r;
        peer[i] = (char*)r;
    }
    const size_t stage_off = stage_of(get_call(p));
#pragma unroll
    for (int i = 0; i < NR - 1; ++i) {
        base[i] = uni(peer[i] + stage_off);
    }
    asm volatile("s_nop 4" ::: "memory");
    PUSH_WAIT(4); push(0, 0);
    issue_load(3, 0); PUSH_WAIT(2); signal(0); push(1, 1);
#pragma unroll 1
    for (int j0 = 2; j0 < 62; j0 += 3) {
        issue_load(j0 + 2, 1); PUSH_WAIT(2); signal(j0 - 1); throttle(j0); push(j0, 2);
        issue_load(j0 + 3, 2); PUSH_WAIT(2); signal(j0); throttle(j0 + 1); push(j0 + 1, 0);
        issue_load(j0 + 4, 0); PUSH_WAIT(2); signal(j0 + 1); throttle(j0 + 2); push(j0 + 2, 1);
    }
    PUSH_WAIT(0); signal(61); throttle(62); push(62, 2);
    PUSH_WAIT(0); signal(62); throttle(63); push(63, 0);
    PUSH_WAIT(0); signal(63);
#undef PUSH_WAIT
}

__device__ __forceinline__ void sender(const Params& p, int lane, unsigned call) {
    unsigned* tmo = (unsigned*)(p.own + OFF_TMO);
    const FlagTrack tr{p.own + OFF_LC, 128, call * (4 * NCOMM), call};
    char* dst = p.ws[lane & 7] + OFF_FLAG + p.rank * 128;
    int sent = 0;
    while (sent < NSTEP) {
        int have = wait_step(tr, sent, lane, tmo);
        if (have > NSTEP) break;
        for (; sent < have; ++sent)
            if (lane < NR) st_sys((unsigned*)(dst + sent * (NR * 128)), call);
    }
}
__device__ __forceinline__ void releaser(const Params& p, int lane, unsigned call, unsigned* cnt) {
    unsigned* tmo = (unsigned*)(p.own + OFF_TMO);
    const FlagTrack tr{p.own + OFF_FLAG + p.rank * 128, NR * 128, call, call};
    int have = 0;
    while (have < NSTEP - THROTTLE) {
        have = wait_step(tr, have, lane, tmo);
        if (lane == 0) __hip_atomic_store(cnt, (unsigned)have, __ATOMIC_RELAXED, __HIP_MEMORY_SCOPE_WORKGROUP);
    }
}

__device__ __forceinline__ unsigned stage_own(const Params& p, int g, int wave, int lane, char* l) {
    const int b = g & 7;
    const int v = (g >> 3) * 8 + wave;
    const int piece = v >> 1;
    const bool isw = v & 1;
    const int vo = p.rank * (int)CH_BYTES + piece * 1024 + lane * 16;
    const int r = lane >> 4, c = lane & 15;
    const char* src = isw ? (const char*)(p.w + (size_t)(b * 64 + r) * NN + piece * 64)
                          : (const char*)(p.x + (size_t)(piece * 64 + r) * KSH + b * 64);
    const int rstride = isw ? 4 * NN * 4 : 4 * KSH * 4;
#pragma unroll
    for (int i = 0; i < 16; ++i) {
        const int slot = isw ? c : (c ^ ((4 * i + r) & 15));
        __builtin_amdgcn_global_load_lds((const gu32*)(src + (size_t)i * rstride + slot * 16),
                                         (unsigned*)(l + i * 1024), 16, 0, 0);
    }
    const unsigned call = get_call(p);
    const __amdgpu_buffer_rsrc_t rd = __builtin_amdgcn_make_buffer_rsrc(
        p.own + stage_of(call) + (isw ? OP_BYTES : 0), 0, (int)OP_BYTES, 0x00020000);
    asm volatile("s_waitcnt vmcnt(0)" ::: "memory");
    float cur[64];
    if (isw) {
#pragma unroll
        for (int k = 0; k < 64; ++k) cur[k] = *(const float*)(l + k * 256 + lane * 4);
    } else {
#pragma unroll
        for (int k = 0; k < 16; ++k) *(f32x4*)&cur[4 * k] = *(const f32x4*)(l + lane * 256 + ((k ^ (lane & 15)) * 16));
    }
#pragma unroll
    for (int o = 0; o < 8; ++o) {
        u32x4 d;
        d.x = pk16(cur[8 * o], cur[8 * o + 1]); d.y = pk16(cur[8 * o + 2], cur[8 * o + 3]);
        d.z = pk16(cur[8 * o + 4], cur[8 * o + 5]); d.w = pk16(cur[8 * o + 6], cur[8 * o + 7]);
        __builtin_amdgcn_raw_buffer_store_b128(d, rd, vo, (b * 8 + o) * (int)(NR * CH_BYTES), 17);
    }
    asm volatile("s_waitcnt vmcnt(0)" ::: "memory");
    __syncthreads();
    if (threadIdx.x == 0)
        __hip_atomic_fetch_add((gu32*)(p.own + OFF_LCL + b * 128), 1u, __ATOMIC_RELAXED, __HIP_MEMORY_SCOPE_AGENT);
    return call;
}

__device__ __forceinline__ void gemm(const Params& p, int tm, int tn, char* smem, unsigned call, size_t stage_off) {
    const int tid = threadIdx.x, lane = tid & 63;
    const int wave = __builtin_amdgcn_readfirstlane(tid >> 6);
    const int wr = wave >> 2, wc = wave & 3;
    const int row0 = tm * TM, col0 = tn * TN;
    char* ws = uni(p.own);
    unsigned* tmo = (unsigned*)(ws + OFF_TMO);
    const StepTrack tr{ws, call, wave, p.rank};
    const __amdgpu_buffer_rsrc_t rs =
        __builtin_amdgcn_make_buffer_rsrc(ws + stage_off, 0, (int)(2 * OP_BYTES), 0x00020000);
    const int vx = row0 * 16 + lane * 16;
    const int vw = (int)OP_BYTES + col0 * 16 + lane * 16;
    const bool full = row0 + TM <= MM;

    auto plane = [&](int st) {
        const int q = 8 * (st - 8) + wave;
        const int jj = (q * 9363) >> 16;
        return st < 8 ? ((8 * st + wave) * NR + p.rank) * (int)CH_BYTES
                      : (jj * NR + ((p.rank + 1 + q - 7 * jj) & 7)) * (int)CH_BYTES;
    };
    auto stage_in = [&](int j, int so) {
        __attribute__((address_space(3))) char* la =
            (__attribute__((address_space(3))) char*)(smem + (j & 1) * STAGE + wave * (TM * 16));
        __attribute__((address_space(3))) char* lb =
            (__attribute__((address_space(3))) char*)(smem + (j & 1) * STAGE + A_STAGE + wave * (TN * 16));
        __builtin_amdgcn_raw_ptr_buffer_load_lds(rs, la, 16, vx, so, 0, LD_AUX);
        __builtin_amdgcn_raw_ptr_buffer_load_lds(rs, la, 16, vx, so, 1024, LD_AUX);
        __builtin_amdgcn_raw_ptr_buffer_load_lds(rs, la, 16, vx, so, 2048, LD_AUX);
        __builtin_amdgcn_raw_ptr_buffer_load_lds(rs, la, 16, vx, so, 3072, LD_AUX);
        if (full)
            __builtin_amdgcn_raw_ptr_buffer_load_lds(rs, la + 4096, 16, vx + 4096, so, 0, LD_AUX);
        __builtin_amdgcn_raw_ptr_buffer_load_lds(rs, lb, 16, vw, so, 0, LD_AUX);
        __builtin_amdgcn_raw_ptr_buffer_load_lds(rs, lb, 16, vw, so, 1024, LD_AUX);
        __builtin_amdgcn_raw_ptr_buffer_load_lds(rs, lb, 16, vw, so, 2048, LD_AUX);
        __builtin_amdgcn_raw_ptr_buffer_load_lds(rs, lb, 16, vw, so, 3072, LD_AUX);
    };

    auto stage_one = [&](int j, int i, int so) {
        __attribute__((address_space(3))) char* la =
            (__attribute__((address_space(3))) char*)(smem + (j & 1) * STAGE + wave * (TM * 16));
        __attribute__((address_space(3))) char* lb =
            (__attribute__((address_space(3))) char*)(smem + (j & 1) * STAGE + A_STAGE + wave * (TN * 16));
        if (i < 4) __builtin_amdgcn_raw_ptr_buffer_load_lds(rs, la + i * 1024, 16, vx + i * 1024, so, 0, LD_AUX);
        else if (i == 4) { if (full) __builtin_amdgcn_raw_ptr_buffer_load_lds(rs, la + 4096, 16, vx + 4096, so, 0, LD_AUX); }
        else __builtin_amdgcn_raw_ptr_buffer_load_lds(rs, lb + (i - 5) * 1024, 16, vw + (i - 5) * 1024, so, 0, LD_AUX);
    };

    f32x4 acc[10][4];
#pragma unroll
    for (int m = 0; m < 10; ++m)
#pragma unroll
        for (int n = 0; n < 4; ++n) acc[m][n] = f32x4{0.f, 0.f, 0.f, 0.f};

    const int fr = lane & 15, fq = lane >> 4;
    const int xoff = fq * (TM * 16) + (wr * 160 + fr) * 16;
    const int woff = A_STAGE + fq * (TN * 16) + (wc * 64 + fr) * 16;

    int ready;
    ready = wait_step(tr, 0, lane, tmo);
    stage_in(0, plane(0));
    asm volatile("s_waitcnt vmcnt(0)" ::: "memory");
    __builtin_amdgcn_s_barrier();
    if (wave == 0) STAMP(1);

    for (int j = 0; j < NSTEP - 1; ++j) {
        constexpr bool more = true;
        const bool pre = more && j + 1 < ready;
        const bool peek = ready < NSTEP && ready < j + PEEKW;
        const int ready0 = ready;
        unsigned peekv = 0, peekt = 0, peek2;
        asm volatile("v_mov_b32 %0, 0" : "=v"(peek2));
        if (peek) peekv = tr.peek(ready0, lane, peekt);
        bool late = false;
        const int so1 = __builtin_amdgcn_readfirstlane(plane(j + 1));
        const bool preE = pre && wr == 0, preL = pre && wr != 0;
        if (ILV_L == 0 && preL) stage_in(j + 1, so1);
        const char* sx = smem + (j & 1) * STAGE + xoff;
        const char* sw = smem + (j & 1) * STAGE + woff;
        auto ldx = [&](int t) { return *(const h16x8*)(sx + (t / 10) * 4 * (TM * 16) + (t % 10) * 256); };
        auto ldw = [&](int s, int n) { return *(const h16x8*)(sw + s * 4 * (TN * 16) + n * 256); };
        h16x8 wf[2][4], xf[4];
        __builtin_amdgcn_sched_barrier(0);
        wf[0][0] = ldw(0, 0); xf[0] = ldx(0);
        __builtin_amdgcn_sched_barrier(0);
        wf[0][1] = ldw(0, 1); wf[0][2] = ldw(0, 2); wf[0][3] = ldw(0, 3);
        xf[1] = ldx(1); xf[2] = ldx(2);
#pragma unroll
        for (int t = 0; t < 20; ++t) {
            __builtin_amdgcn_sched_barrier(0);
            acc[t % 10][0] = __builtin_amdgcn_mfma_f32_16x16x32_f16(wf[t / 10][0], xf[t & 3], acc[t % 10][0], 0, 0, 0);
            __builtin_amdgcn_sched_barrier(0);
            if (t + 3 < 20) xf[(t + 3) & 3] = ldx(t + 3);
            if (t >= 2 && t < 6) wf[1][t - 2] = ldw(1, t - 2);
#pragma unroll
            for (int i = 0; i < 9; ++i) {
                if (i / ILV_E == t && preE) stage_one(j + 1, i, so1);
                if (ILV_L > 0 && i / (ILV_L > 0 ? ILV_L : 1) == t && preL) stage_one(j + 1, i, so1);
            }
            if (t == LATE_T && more && !pre) {
                asm volatile("" : "+v"(peekv));
                ready = ready0 + flag_count(peekv, peekt);
                late = j + 1 < ready;
                if (late) stage_in(j + 1, so1);
            }
            if (t == LATE_T + 1 && more && !pre && !late) {
                unsigned th2;
                peek2 = tr.peek(ready0, lane, th2);
            }
            if (t == LATE2_T && more && !pre && !late) {
                asm volatile("" : "+v"(peek2));
                ready = ready0 + flag_count(peek2, peekt);
                late = j + 1 < ready;
                if (late) stage_in(j + 1, so1);
            }
            __builtin_amdgcn_sched_barrier(0);
#pragma unroll
            for (int n = 1; n < 4; ++n)
                acc[t % 10][n] = __builtin_amdgcn_mfma_f32_16x16x32_f16(wf[t / 10][n], xf[t & 3], acc[t % 10][n], 0, 0, 0);
        }
        __builtin_amdgcn_sched_barrier(0);
        {
            const int c = flag_count(peekv, peekt);
            if (peek && ready0 + c > ready) ready = ready0 + c;
        }
        if (more && !pre && !late) {
            if (j + 1 >= ready) ready = wait_step(tr, j + 1, lane, tmo);
            stage_in(j + 1, so1);
        }
        asm volatile("s_waitcnt vmcnt(0) lgkmcnt(0)" ::: "memory");
        __builtin_amdgcn_s_barrier();
    }

    if (wave == 0) STAMP(2 + NSTEP - 1);
    const unsigned tmo_v = ld_agent(tmo);
    const int mlim = full ? 10 : (MM - row0 - wr * 160 + 15) / 16;
    const __amdgpu_buffer_rsrc_t ro =
        __builtin_amdgcn_make_buffer_rsrc(p.out, 0, (int)((size_t)MM * NN * 4), 0x00020000);
    const int vo = ((row0 + wr * 160 + fr) * NN + col0 + wc * 64 + fq * 4) * 4;
    f32x2 kneg = {-1.44269504088896340736f, -1.44269504088896340736f};
    asm volatile("" : "+v"(kneg));
    bool bad = false;
    {
        const char* sx = smem + ((NSTEP - 1) & 1) * STAGE + xoff;
        const char* sw = smem + ((NSTEP - 1) & 1) * STAGE + woff;
        auto ldx = [&](int s, int m) { return *(const h16x8*)(sx + s * 4 * (TM * 16) + m * 256); };
        auto ldw = [&](int s, int n) { return *(const h16x8*)(sw + s * 4 * (TN * 16) + n * 256); };
        auto store_group = [&](int m) {
            if (!bad && m < mlim) {
#pragma unroll
                for (int n = 0; n < 4; ++n)
                    __builtin_amdgcn_raw_buffer_store_b128(__builtin_bit_cast(u32x4, silu4(acc[m][n], kneg)), ro,
                                                           vo + m * (16 * NN * 4) + n * 64, 0, ST_AUX);
            }
        };
        h16x8 wf[2][4], xf[2][2];
        __builtin_amdgcn_sched_barrier(0);
        wf[0][0] = ldw(0, 0); xf[0][0] = ldx(0, 0);
        __builtin_amdgcn_sched_barrier(0);
        wf[0][1] = ldw(0, 1); wf[0][2] = ldw(0, 2); wf[0][3] = ldw(0, 3);
        xf[0][1] = ldx(1, 0);
        wf[1][0] = ldw(1, 0); wf[1][1] = ldw(1, 1); wf[1][2] = ldw(1, 2); wf[1][3] = ldw(1, 3);
#pragma unroll
        for (int m = 0; m < 10; ++m) {
            __builtin_amdgcn_sched_barrier(0);
            if (m + 1 < 10) { xf[(m + 1) & 1][0] = ldx(0, m + 1); xf[(m + 1) & 1][1] = ldx(1, m + 1); }
            __builtin_amdgcn_sched_barrier(0);
#pragma unroll
            for (int s = 0; s < 2; ++s)
#pragma unroll
                for (int n = 0; n < 4; ++n)
                    acc[m][n] = __builtin_amdgcn_mfma_f32_16x16x32_f16(wf[s][n], xf[m & 1][s], acc[m][n], 0, 0, 0);
            __builtin_amdgcn_sched_barrier(0);
            if (m == 1) bad = __builtin_amdgcn_readfirstlane(tmo_v) == call;
            if (m >= 1) store_group(m - 1);
        }
        __builtin_amdgcn_sched_barrier(0);
        if (wave == 0) STAMP(68);
        store_group(9);
    }
    if (bad) {
        const u32x4 nan4 = {0x7fc00000u, 0x7fc00000u, 0x7fc00000u, 0x7fc00000u};
        for (int m = 0; m < mlim; ++m)
            for (int n = 0; n < 4; ++n)
                __builtin_amdgcn_raw_buffer_store_b128(nan4, ro, vo + m * (16 * NN * 4) + n * 64, 0, 0);
    }
}

__global__ __launch_bounds__(512, 2) void fused_gemm_allgather_silu(Params p) {
    extern __shared__ __attribute__((aligned(16))) char smem[];
    const int tid = threadIdx.x, lane = tid & 63;
    const int wave = __builtin_amdgcn_readfirstlane(tid >> 6);
    unsigned* done = (unsigned*)(p.own + OFF_DONE);
    const int xcd = blockIdx.x & 7, slot = blockIdx.x >> 3;
    if (wave == 0) STAMP(0);
    if (slot < 4) {
        const int cb = slot * 8 + xcd;
        unsigned* cnt = (unsigned*)smem;
        if (tid == 0) *cnt = 0;
        __syncthreads();
        if (wave < 4) pusher(p, cb * 2 + (wave & 1), wave >> 1, lane, cnt, smem + 1024 + wave * 6144);
        else if (wave == 4 + EXP_SIGOFF) releaser(p, lane, get_call(p), cnt);
    } else if (slot < 30) {
        const int i = slot - 4, h = xcd >> 2;
        const int tm = i < 24 ? 7 * h + i % 6 : 6;
        const int tn = (xcd & 3) * 4 + (i < 24 ? i / 6 : 2 * h + i - 24);
        const unsigned call = i < 16 ? stage_own(p, xcd * 16 + i, wave, lane, smem + wave * 16384) : get_call(p);
        gemm(p, tm, tn, smem, call, stage_of(call));
    }
    else if (blockIdx.x == 241) {
        if (wave == 0) sender(p, lane, get_call(p));
    }
    __syncthreads();
    if (tid == 0) __hip_atomic_fetch_add((gu32*)done, 1u, __ATOMIC_RELAXED, __HIP_MEMORY_SCOPE_AGENT);
}

}

extern "C" void kernel_launch(int rank, int nranks, ncclComm_t comm,
                              hipStream_t stream,
                              void* const* d_in, const int* in_sizes, int n_in,
                              void* d_out, int out_size,
                              const int* in_dtypes, int out_dtype,
                              void* const* d_out_all,
                              void* const* d_ws_all, size_t ws_bytes) {
    (void)comm; (void)in_dtypes; (void)out_dtype; (void)d_out_all;
    if (nranks != NR || (unsigned)rank >= (unsigned)NR || n_in < 2 || ws_bytes < WS_NEED ||
        in_sizes[0] != MM * KSH || in_sizes[1] != KSH * NN || out_size != MM * NN) {
        fprintf(stderr, "unexpected problem shape\n");
        return;
    }
    static bool attr_set = false;
    if (!attr_set) {
        HIP_CHECK(hipFuncSetAttribute((const void*)fused_gemm_allgather_silu,
                                      hipFuncAttributeMaxDynamicSharedMemorySize, LDS_BYTES));
        attr_set = true;
    }
    Params p;
    p.x = (const float*)d_in[0];
    p.w = (const float*)d_in[1];
    p.out = (float*)d_out;
    for (int i = 0; i < NR; ++i) p.ws[i] = (char*)d_ws_all[i];
    p.own = p.ws[rank];
    p.rank = rank;
    fused_gemm_allgather_silu<<<NWG, 512, LDS_BYTES, stream>>>(p);
    HIP_CHECK(hipGetLastError());
}
```

```cpp
#include <hip/hip_runtime.h>
#include <rccl/rccl.h>
#include <cstdio>

#define HIP_CHECK(expr) do { hipError_t _e = (expr); if (_e != hipSuccess) { \
    fprintf(stderr, "HIP error %s at %s:%d\n", hipGetErrorString(_e), __FILE__, __LINE__); \
    return; } } while (0)

#define LD_AUX 17
#define ST_AUX 16

namespace {

constexpr int NR = 8;
constexpr int MM = 4096, NN = 4096, KSH = 512;
constexpr int NSTEP = 64;
constexpr int NCOMM = 32;
constexpr int NSTAGE = 16;
constexpr int NWG = 256;
constexpr int TM = 320, TN = 256;
#define THROTTLE 4
#define ILV_E 1
#define ILV_L 0
#define LA 1
#define PSLEEP 4
#define PEEKW 3
#define LATE2_T 12
#define LATE_T 4

constexpr size_t OFF_DONE = 0;
constexpr size_t OFF_TMO = 128;
constexpr size_t OFF_LC = 4096;
constexpr size_t OFF_LCL = 12288;
constexpr size_t OFF_FLAG = 16384;
constexpr size_t CTRL_BYTES = 1u << 20;
constexpr size_t CH_BYTES = 65536;
constexpr size_t OP_BYTES = (size_t)NSTEP * NR * CH_BYTES;
constexpr size_t WS_NEED = CTRL_BYTES + 4 * OP_BYTES;
static_assert(OFF_LC + NSTEP * 128 <= OFF_LCL && OFF_LCL + 8 * 128 <= OFF_FLAG, "layout");
static_assert(OFF_FLAG + NSTEP * NR * 128 <= CTRL_BYTES, "layout");

constexpr int A_STAGE = 8 * TM * 16;
constexpr int B_STAGE = 8 * TN * 16;
constexpr int STAGE = A_STAGE + B_STAGE;
constexpr int LDS_BYTES = 2 * STAGE;

constexpr unsigned long long DEADLINE_TICKS = 50000000ull;

typedef unsigned u32x4 __attribute__((ext_vector_type(4)));
typedef float f32x4 __attribute__((ext_vector_type(4)));
typedef _Float16 h16x8 __attribute__((ext_vector_type(8)));
typedef _Float16 h16x2 __attribute__((ext_vector_type(2)));
typedef __attribute__((address_space(1))) unsigned gu32;

struct Params {
    const float* x;
    const float* w;
    float* out;
    char* ws[NR];
    char* own;
    int rank;
};

template <class T>
__device__ __forceinline__ T* uni(T* p) {
    unsigned long long v = (unsigned long long)p;
    unsigned lo = __builtin_amdgcn_readfirstlane((unsigned)v);
    unsigned hi = __builtin_amdgcn_readfirstlane((unsigned)(v >> 32));
    return (T*)(((unsigned long long)hi << 32) | lo);
}
__device__ __forceinline__ unsigned pk16(float a, float b) {
    h16x2 v;
    v.x = (_Float16)a;
    v.y = (_Float16)b;
    return __builtin_bit_cast(unsigned, v);
}
typedef float f32x2 __attribute__((ext_vector_type(2)));
__device__ __forceinline__ f32x4 silu4(f32x4 v, f32x2 k) {
    const float lim = 60.0f * 1.44269504088896340736f;
    f32x2 a = f32x2{v.x, v.y} * k, b = f32x2{v.z, v.w} * k;
    a.x = __builtin_amdgcn_exp2f(__builtin_amdgcn_fmed3f(a.x, -lim, lim));
    a.y = __builtin_amdgcn_exp2f(__builtin_amdgcn_fmed3f(a.y, -lim, lim));
    b.x = __builtin_amdgcn_exp2f(__builtin_amdgcn_fmed3f(b.x, -lim, lim));
    b.y = __builtin_amdgcn_exp2f(__builtin_amdgcn_fmed3f(b.y, -lim, lim));
    a = a + 1.0f;
    b = b + 1.0f;
    a.x = __builtin_amdgcn_rcpf(a.x); a.y = __builtin_amdgcn_rcpf(a.y);
    b.x = __builtin_amdgcn_rcpf(b.x); b.y = __builtin_amdgcn_rcpf(b.y);
    a = a * f32x2{v.x, v.y};
    b = b * f32x2{v.z, v.w};
    return f32x4{a.x, a.y, b.x, b.y};
}
__device__ __forceinline__ unsigned ld_sys(const unsigned* p) {
    return __hip_atomic_load((gu32*)p, __ATOMIC_RELAXED, __HIP_MEMORY_SCOPE_SYSTEM);
}
__device__ __forceinline__ unsigned ld_agent(const unsigned* p) {
    return __hip_atomic_load((gu32*)p, __ATOMIC_RELAXED, __HIP_MEMORY_SCOPE_AGENT);
}
__device__ __forceinline__ void st_agent(unsigned* p, unsigned v) {
    __hip_atomic_store((gu32*)p, v, __ATOMIC_RELAXED, __HIP_MEMORY_SCOPE_AGENT);
}
__device__ __forceinline__ void st_sys(unsigned* p, unsigned v) {
    __hip_atomic_store((gu32*)p, v, __ATOMIC_RELAXED, __HIP_MEMORY_SCOPE_SYSTEM);
}
__device__ __forceinline__ unsigned get_call(const Params& p) {
    return __builtin_amdgcn_readfirstlane(ld_agent((const unsigned*)(p.own + OFF_DONE))) / NWG + 1;
}
__device__ __forceinline__ size_t stage_of(unsigned call) {
    return CTRL_BYTES + (size_t)(call & 1) * 2 * OP_BYTES;
}
struct FlagTrack {
    const char* f0;
    int stride;
    unsigned lim, call;
    __device__ __forceinline__ unsigned peek(int i0, int lane, unsigned& thr) const {
        int g = i0 + (lane & 7);
        g = g < NSTEP ? g : NSTEP - 1;
        thr = lim;
        unsigned v = lim - 1;
        if (lane < LA) v = ld_sys((const unsigned*)(f0 + g * stride));
        return v;
    }
};
struct StepTrack {
    const char* ws;
    unsigned call;
    int wave, rank;
    __device__ __forceinline__ unsigned peek(int i0, int lane, unsigned& thr) const {
        int s = i0 + (lane & 7);
        s = s < NSTEP ? s : NSTEP - 1;
        const int q = 8 * (s - 8) + wave;
        const int j = (q * 9363) >> 16;
        const int src = (rank + 1 + q - 7 * j) & 7;
        const int off = s < 8 ? (int)OFF_LCL + s * 128 : (int)OFF_FLAG + (j * NR + src) * 128;
        thr = s < 8 ? call * NSTAGE : call;
        unsigned v = 0;
        if (lane < LA) v = ld_sys((const unsigned*)(ws + off));
        return v;
    }
};
__device__ __forceinline__ int flag_count(unsigned v, unsigned thr) {
    unsigned m = (unsigned)__builtin_amdgcn_ballot_w64((int)(v - thr) >= 0) & ((1u << LA) - 1);
    return __builtin_ctz(~m);
}
template <class T>
__device__ __forceinline__ int wait_step(const T& tr, int g0, int lane, unsigned* tmo) {
    unsigned thr;
    unsigned v = tr.peek(g0, lane, thr);
    int c = flag_count(v, thr);
    if (c > 0) return g0 + c;
    unsigned long long t0 = __builtin_amdgcn_s_memrealtime();
    for (unsigned it = 1;; ++it) {
        v = tr.peek(g0, lane, thr);
        c = flag_count(v, thr);
        if (c > 0) return g0 + c;
        if ((it & 15) == 0) {
            if (__builtin_amdgcn_readfirstlane(ld_agent(tmo)) == tr.call ||
                __builtin_amdgcn_s_memrealtime() - t0 > DEADLINE_TICKS) {
                st_agent(tmo, tr.call);
                return NSTEP + 8;
            }
        }
        __builtin_amdgcn_s_sleep(PSLEEP);
    }
}

__device__ __forceinline__ void pusher(const Params& p, int piece, int phase, int lane,
                                       unsigned* cnt, char* ring) {
    const char* srcx = (const char*)(p.x + (size_t)piece * 64 * KSH) + lane * (KSH * 4);
    const char* srcw = (const char*)(p.w + piece * 64) + (lane >> 4) * (NN * 4) + (lane & 15) * 16;
    auto issue_load = [&](int j, int slot) {
        char* l = ring + slot * 2048;
        const bool isw = (j + phase) & 1;
        const char* g0 = isw ? srcw + (size_t)j * (8 * NN * 4) : srcx + j * 32;
        const char* g1 = g0 + (isw ? 4 * NN * 4 : 16);
        __builtin_amdgcn_global_load_lds((const gu32*)g0, (unsigned*)l, 16, 0, 0);
        __builtin_amdgcn_global_load_lds((const gu32*)g1, (unsigned*)(l + 1024), 16, 0, 0);
    };
    const unsigned cnt_addr = (unsigned)(size_t)(__attribute__((address_space(3))) unsigned*)cnt;
    const char* lc = uni(p.own + OFF_LC);
    auto signal = [&](int j) {
        if (lane == 0)
            asm volatile("global_atomic_add %0, %1, %2" :: "v"(j * 128), "v"(1u), "s"(lc) : "memory");
    };
    const char* base[NR - 1];
    const unsigned vbase = (unsigned)(p.rank * CH_BYTES + piece * 1024 + lane * 16);
    auto push = [&](int j, int slot) {
        const char* l = ring + slot * 2048;
        const bool isw = (j + phase) & 1;
        float s[8];
        if (isw) {
#pragma unroll
            for (int q = 0; q < 8; ++q) s[q] = *(const float*)(l + q * 256 + lane * 4);
        } else {
            *(f32x4*)&s[0] = *(const f32x4*)(l + lane * 16);
            *(f32x4*)&s[4] = *(const f32x4*)(l + 1024 + lane * 16);
        }
        u32x4 d;
        d.x = pk16(s[0], s[1]); d.y = pk16(s[2], s[3]); d.z = pk16(s[4], s[5]); d.w = pk16(s[6], s[7]);
        unsigned vo = vbase + (unsigned)j * (unsigned)(NR * CH_BYTES) + (isw ? (unsigned)OP_BYTES : 0u);
#pragma unroll
        for (int q = 0; q < NR - 1; ++q)
            asm volatile("global_store_dwordx4 %0, %1, %2 sc0 sc1\n\ts_nop 1" :: "v"(vo), "v"(d), "s"(base[q]) : "memory");
    };
#define PUSH_WAIT(N) asm volatile("s_waitcnt vmcnt(" #N ")" ::: "memory")
    auto throttle = [&](int j) {
        if (j < THROTTLE) return;
        const unsigned need = j - THROTTLE + 1;
        unsigned v;
        asm volatile("ds_read_b32 %0, %1\n\ts_waitcnt lgkmcnt(0)" : "=v"(v) : "v"(cnt_addr) : "memory");
        if (__builtin_amdgcn_readfirstlane(v) >= need) return;
        unsigned long long t0 = __builtin_amdgcn_s_memrealtime();
        do {
            __builtin_amdgcn_s_sleep(2);
            asm volatile("ds_read_b32 %0, %1\n\ts_waitcnt lgkmcnt(0)" : "=v"(v) : "v"(cnt_addr) : "memory");
        } while (__builtin_amdgcn_readfirstlane(v) < need &&
                 __builtin_amdgcn_s_memrealtime() - t0 < DEADLINE_TICKS);
    };
    issue_load(0, 0);
    issue_load(1, 1);
    issue_load(2, 2);
    unsigned long long wsv[NR];
#pragma unroll
    for (int k = 0; k < NR; ++k) {
        wsv[k] = (unsigned long long)p.ws[k];
        asm volatile("" : "+s"(wsv[k]));
    }
    char* peer[NR - 1];
#pragma unroll
    for (int i = 0; i < NR - 1; ++i) {
        const int q = (p.rank + 1 + (i + piece) % 7) & 7;
        unsigned long long r = wsv[0];
#pragma unroll
        for (int k = 1; k < NR; ++k) r = q == k ? wsv[k] : r;
        peer[i] = (char*)r;
    }
    const size_t stage_off = stage_of(get_call(p));
#pragma unroll
    for (int i = 0; i < NR - 1; ++i) {
        base[i] = uni(peer[i] + stage_off);
    }
    asm volatile("s_nop 4" ::: "memory");
    PUSH_WAIT(4); push(0, 0);
    issue_load(3, 0); PUSH_WAIT(2); signal(0); push(1, 1);
#pragma unroll 1
    for (int j0 = 2; j0 < 62; j0 += 3) {
        issue_load(j0 + 2, 1); PUSH_WAIT(2); signal(j0 - 1); throttle(j0); push(j0, 2);
        issue_load(j0 + 3, 2); PUSH_WAIT(2); signal(j0); throttle(j0 + 1); push(j0 + 1, 0);
        issue_load(j0 + 4, 0); PUSH_WAIT(2); signal(j0 + 1); throttle(j0 + 2); push(j0 + 2, 1);
    }
    PUSH_WAIT(0); signal(61); throttle(62); push(62, 2);
    PUSH_WAIT(0); signal(62); throttle(63); push(63, 0);
    PUSH_WAIT(0); signal(63);
#undef PUSH_WAIT
}

__device__ __forceinline__ void sender(const Params& p, int lane, unsigned call) {
    unsigned* tmo = (unsigned*)(p.own + OFF_TMO);
    const FlagTrack tr{p.own + OFF_LC, 128, call * (4 * NCOMM), call};
    char* dst = p.ws[lane & 7] + OFF_FLAG + p.rank * 128;
    int sent = 0;
    while (sent < NSTEP) {
        int have = wait_step(tr, sent, lane, tmo);
        if (have > NSTEP) break;
        for (; sent < have; ++sent)
            if (lane < NR) st_sys((unsigned*)(dst + sent * (NR * 128)), call);
    }
}
__device__ __forceinline__ void releaser(const Params& p, int lane, unsigned call, unsigned* cnt) {
    unsigned* tmo = (unsigned*)(p.own + OFF_TMO);
    const FlagTrack tr{p.own + OFF_FLAG + p.rank * 128, NR * 128, call, call};
    int have = 0;
    while (have < NSTEP - THROTTLE) {
        have = wait_step(tr, have, lane, tmo);
        if (lane == 0) __hip_atomic_store(cnt, (unsigned)have, __ATOMIC_RELAXED, __HIP_MEMORY_SCOPE_WORKGROUP);
    }
}

__device__ __forceinline__ unsigned stage_own(const Params& p, int g, int wave, int lane, char* l) {
    const int b = g & 7;
    const int v = (g >> 3) * 8 + wave;
    const int piece = v >> 1;
    const bool isw = v & 1;
    const int vo = p.rank * (int)CH_BYTES + piece * 1024 + lane * 16;
    const int r = lane >> 4, c = lane & 15;
    const char* src = isw ? (const char*)(p.w + (size_t)(b * 64 + r) * NN + piece * 64)
                          : (const char*)(p.x + (size_t)(piece * 64 + r) * KSH + b * 64);
    const int rstride = isw ? 4 * NN * 4 : 4 * KSH * 4;
#pragma unroll
    for (int i = 0; i < 16; ++i) {
        const int slot = isw ? c : (c ^ ((4 * i + r) & 15));
        __builtin_amdgcn_global_load_lds((const gu32*)(src + (size_t)i * rstride + slot * 16),
                                         (unsigned*)(l + i * 1024), 16, 0, 0);
    }
    const unsigned call = get_call(p);
    const __amdgpu_buffer_rsrc_t rd = __builtin_amdgcn_make_buffer_rsrc(
        p.own + stage_of(call) + (isw ? OP_BYTES : 0), 0, (int)OP_BYTES, 0x00020000);
    asm volatile("s_waitcnt vmcnt(0)" ::: "memory");
    float cur[64];
    if (isw) {
#pragma unroll
        for (int k = 0; k < 64; ++k) cur[k] = *(const float*)(l + k * 256 + lane * 4);
    } else {
#pragma unroll
        for (int k = 0; k < 16; ++k) *(f32x4*)&cur[4 * k] = *(const f32x4*)(l + lane * 256 + ((k ^ (lane & 15)) * 16));
    }
#pragma unroll
    for (int o = 0; o < 8; ++o) {
        u32x4 d;
        d.x = pk16(cur[8 * o], cur[8 * o + 1]); d.y = pk16(cur[8 * o + 2], cur[8 * o + 3]);
        d.z = pk16(cur[8 * o + 4], cur[8 * o + 5]); d.w = pk16(cur[8 * o + 6], cur[8 * o + 7]);
        __builtin_amdgcn_raw_buffer_store_b128(d, rd, vo, (b * 8 + o) * (int)(NR * CH_BYTES), 17);
    }
    asm volatile("s_waitcnt vmcnt(0)" ::: "memory");
    __syncthreads();
    if (threadIdx.x == 0)
        __hip_atomic_fetch_add((gu32*)(p.own + OFF_LCL + b * 128), 1u, __ATOMIC_RELAXED, __HIP_MEMORY_SCOPE_AGENT);
    return call;
}

__device__ __forceinline__ void gemm(const Params& p, int tm, int tn, char* smem, unsigned call, size_t stage_off) {
    const int tid = threadIdx.x, lane = tid & 63;
    const int wave = __builtin_amdgcn_readfirstlane(tid >> 6);
    const int wr = wave >> 2, wc = wave & 3;
    const int row0 = tm * TM, col0 = tn * TN;
    char* ws = uni(p.own);
    unsigned* tmo = (unsigned*)(ws + OFF_TMO);
    const StepTrack tr{ws, call, wave, p.rank};
    const __amdgpu_buffer_rsrc_t rs =
        __builtin_amdgcn_make_buffer_rsrc(ws + stage_off, 0, (int)(2 * OP_BYTES), 0x00020000);
    const int vx = row0 * 16 + lane * 16;
    const int vw = (int)OP_BYTES + col0 * 16 + lane * 16;
    const bool full = row0 + TM <= MM;

    auto plane = [&](int st) {
        const int q = 8 * (st - 8) + wave;
        const int jj = (q * 9363) >> 16;
        return st < 8 ? ((8 * st + wave) * NR + p.rank) * (int)CH_BYTES
                      : (jj * NR + ((p.rank + 1 + q - 7 * jj) & 7)) * (int)CH_BYTES;
    };
    auto stage_in = [&](int j, int so) {
        __attribute__((address_space(3))) char* la =
            (__attribute__((address_space(3))) char*)(smem + (j & 1) * STAGE + wave * (TM * 16));
        __attribute__((address_space(3))) char* lb =
            (__attribute__((address_space(3))) char*)(smem + (j & 1) * STAGE + A_STAGE + wave * (TN * 16));
        __builtin_amdgcn_raw_ptr_buffer_load_lds(rs, la, 16, vx, so, 0, LD_AUX);
        __builtin_amdgcn_raw_ptr_buffer_load_lds(rs, la, 16, vx, so, 1024, LD_AUX);
        __builtin_amdgcn_raw_ptr_buffer_load_lds(rs, la, 16, vx, so, 2048, LD_AUX);
        __builtin_amdgcn_raw_ptr_buffer_load_lds(rs, la, 16, vx, so, 3072, LD_AUX);
        if (full)
            __builtin_amdgcn_raw_ptr_buffer_load_lds(rs, la + 4096, 16, vx + 4096, so, 0, LD_AUX);
        __builtin_amdgcn_raw_ptr_buffer_load_lds(rs, lb, 16, vw, so, 0, LD_AUX);
        __builtin_amdgcn_raw_ptr_buffer_load_lds(rs, lb, 16, vw, so, 1024, LD_AUX);
        __builtin_amdgcn_raw_ptr_buffer_load_lds(rs, lb, 16, vw, so, 2048, LD_AUX);
        __builtin_amdgcn_raw_ptr_buffer_load_lds(rs, lb, 16, vw, so, 3072, LD_AUX);
    };

    auto stage_one = [&](int j, int i, int so) {
        __attribute__((address_space(3))) char* la =
            (__attribute__((address_space(3))) char*)(smem + (j & 1) * STAGE + wave * (TM * 16));
        __attribute__((address_space(3))) char* lb =
            (__attribute__((address_space(3))) char*)(smem + (j & 1) * STAGE + A_STAGE + wave * (TN * 16));
        if (i < 4) __builtin_amdgcn_raw_ptr_buffer_load_lds(rs, la + i * 1024, 16, vx + i * 1024, so, 0, LD_AUX);
        else if (i == 4) { if (full) __builtin_amdgcn_raw_ptr_buffer_load_lds(rs, la + 4096, 16, vx + 4096, so, 0, LD_AUX); }
        else __builtin_amdgcn_raw_ptr_buffer_load_lds(rs, lb + (i - 5) * 1024, 16, vw + (i - 5) * 1024, so, 0, LD_AUX);
    };

    f32x4 acc[10][4];
#pragma unroll
    for (int m = 0; m < 10; ++m)
#pragma unroll
        for (int n = 0; n < 4; ++n) acc[m][n] = f32x4{0.f, 0.f, 0.f, 0.f};

    const int fr = lane & 15, fq = lane >> 4;
    const int xoff = fq * (TM * 16) + (wr * 160 + fr) * 16;
    const int woff = A_STAGE + fq * (TN * 16) + (wc * 64 + fr) * 16;

    int ready;
    ready = wait_step(tr, 0, lane, tmo);
    stage_in(0, plane(0));
    asm volatile("s_waitcnt vmcnt(0)" ::: "memory");
    __builtin_amdgcn_s_barrier();

    for (int j = 0; j < NSTEP - 1; ++j) {
        constexpr bool more = true;
        const bool pre = more && j + 1 < ready;
        const bool peek = ready < NSTEP && ready < j + PEEKW;
        const int ready0 = ready;
        unsigned peekv = 0, peekt = 0, peek2;
        asm volatile("v_mov_b32 %0, 0" : "=v"(peek2));
        if (peek) peekv = tr.peek(ready0, lane, peekt);
        bool late = false;
        const int so1 = __builtin_amdgcn_readfirstlane(plane(j + 1));
        const bool preE = pre && wr == 0, preL = pre && wr != 0;
        if (ILV_L == 0 && preL) stage_in(j + 1, so1);
        const char* sx = smem + (j & 1) * STAGE + xoff;
        const char* sw = smem + (j & 1) * STAGE + woff;
        auto ldx = [&](int t) { return *(const h16x8*)(sx + (t / 10) * 4 * (TM * 16) + (t % 10) * 256); };
        auto ldw = [&](int s, int n) { return *(const h16x8*)(sw + s * 4 * (TN * 16) + n * 256); };
        h16x8 wf[2][4], xf[4];
        __builtin_amdgcn_sched_barrier(0);
        wf[0][0] = ldw(0, 0); xf[0] = ldx(0);
        __builtin_amdgcn_sched_barrier(0);
        wf[0][1] = ldw(0, 1); wf[0][2] = ldw(0, 2); wf[0][3] = ldw(0, 3);
        xf[1] = ldx(1); xf[2] = ldx(2);
#pragma unroll
        for (int t = 0; t < 20; ++t) {
            __builtin_amdgcn_sched_barrier(0);
            acc[t % 10][0] = __builtin_amdgcn_mfma_f32_16x16x32_f16(wf[t / 10][0], xf[t & 3], acc[t % 10][0], 0, 0, 0);
            __builtin_amdgcn_sched_barrier(0);
            if (t + 3 < 20) xf[(t + 3) & 3] = ldx(t + 3);
            if (t >= 2 && t < 6) wf[1][t - 2] = ldw(1, t - 2);
#pragma unroll
            for (int i = 0; i < 9; ++i) {
                if (i / ILV_E == t && preE) stage_one(j + 1, i, so1);
                if (ILV_L > 0 && i / (ILV_L > 0 ? ILV_L : 1) == t && preL) stage_one(j + 1, i, so1);
            }
            if (t == LATE_T && more && !pre) {
                asm volatile("" : "+v"(peekv));
                ready = ready0 + flag_count(peekv, peekt);
                late = j + 1 < ready;
                if (late) stage_in(j + 1, so1);
            }
            if (t == LATE_T + 1 && more && !pre && !late) {
                unsigned th2;
                peek2 = tr.peek(ready0, lane, th2);
            }
            if (t == LATE2_T && more && !pre && !late) {
                asm volatile("" : "+v"(peek2));
                ready = ready0 + flag_count(peek2, peekt);
                late = j + 1 < ready;
                if (late) stage_in(j + 1, so1);
            }
            __builtin_amdgcn_sched_barrier(0);
#pragma unroll
            for (int n = 1; n < 4; ++n)
                acc[t % 10][n] = __builtin_amdgcn_mfma_f32_16x16x32_f16(wf[t / 10][n], xf[t & 3], acc[t % 10][n], 0, 0, 0);
        }
        __builtin_amdgcn_sched_barrier(0);
        {
            const int c = flag_count(peekv, peekt);
            if (peek && ready0 + c > ready) ready = ready0 + c;
        }
        if (more && !pre && !late) {
            if (j + 1 >= ready) ready = wait_step(tr, j + 1, lane, tmo);
            stage_in(j + 1, so1);
        }
        asm volatile("s_waitcnt vmcnt(0) lgkmcnt(0)" ::: "memory");
        __builtin_amdgcn_s_barrier();
    }

    const unsigned tmo_v = ld_agent(tmo);
    const int mlim = full ? 10 : (MM - row0 - wr * 160 + 15) / 16;
    const __amdgpu_buffer_rsrc_t ro =
        __builtin_amdgcn_make_buffer_rsrc(p.out, 0, (int)((size_t)MM * NN * 4), 0x00020000);
    const int vo = ((row0 + wr * 160 + fr) * NN + col0 + wc * 64 + fq * 4) * 4;
    f32x2 kneg = {-1.44269504088896340736f, -1.44269504088896340736f};
    asm volatile("" : "+v"(kneg));
    bool bad = false;
    {
        const char* sx = smem + ((NSTEP - 1) & 1) * STAGE + xoff;
        const char* sw = smem + ((NSTEP - 1) & 1) * STAGE + woff;
        auto ldx = [&](int s, int m) { return *(const h16x8*)(sx + s * 4 * (TM * 16) + m * 256); };
        auto ldw = [&](int s, int n) { return *(const h16x8*)(sw + s * 4 * (TN * 16) + n * 256); };
        auto store_group = [&](int m) {
            if (!bad && m < mlim) {
#pragma unroll
                for (int n = 0; n < 4; ++n)
                    __builtin_amdgcn_raw_buffer_store_b128(__builtin_bit_cast(u32x4, silu4(acc[m][n], kneg)), ro,
                                                           vo + m * (16 * NN * 4) + n * 64, 0, ST_AUX);
            }
        };
        h16x8 wf[2][4], xf[2][2];
        __builtin_amdgcn_sched_barrier(0);
        wf[0][0] = ldw(0, 0); xf[0][0] = ldx(0, 0);
        __builtin_amdgcn_sched_barrier(0);
        wf[0][1] = ldw(0, 1); wf[0][2] = ldw(0, 2); wf[0][3] = ldw(0, 3);
        xf[0][1] = ldx(1, 0);
        wf[1][0] = ldw(1, 0); wf[1][1] = ldw(1, 1); wf[1][2] = ldw(1, 2); wf[1][3] = ldw(1, 3);
#pragma unroll
        for (int m = 0; m < 10; ++m) {
            __builtin_amdgcn_sched_barrier(0);
            if (m + 1 < 10) { xf[(m + 1) & 1][0] = ldx(0, m + 1); xf[(m + 1) & 1][1] = ldx(1, m + 1); }
            __builtin_amdgcn_sched_barrier(0);
#pragma unroll
            for (int s = 0; s < 2; ++s)
#pragma unroll
                for (int n = 0; n < 4; ++n)
                    acc[m][n] = __builtin_amdgcn_mfma_f32_16x16x32_f16(wf[s][n], xf[m & 1][s], acc[m][n], 0, 0, 0);
            __builtin_amdgcn_sched_barrier(0);
            if (m == 1) bad = __builtin_amdgcn_readfirstlane(tmo_v) == call;
            if (m >= 1) store_group(m - 1);
        }
        __builtin_amdgcn_sched_barrier(0);
        store_group(9);
    }
    if (bad) {
        const u32x4 nan4 = {0x7fc00000u, 0x7fc00000u, 0x7fc00000u, 0x7fc00000u};
        for (int m = 0; m < mlim; ++m)
            for (int n = 0; n < 4; ++n)
                __builtin_amdgcn_raw_buffer_store_b128(nan4, ro, vo + m * (16 * NN * 4) + n * 64, 0, 0);
    }
}

__global__ __launch_bounds__(512, 2) void fused_gemm_allgather_silu(Params p) {
    extern __shared__ __attribute__((aligned(16))) char smem[];
    const int tid = threadIdx.x, lane = tid & 63;
    const int wave = __builtin_amdgcn_readfirstlane(tid >> 6);
    unsigned* done = (unsigned*)(p.own + OFF_DONE);
    const int xcd = blockIdx.x & 7, slot = blockIdx.x >> 3;
    if (slot < 4) {
        const int cb = slot * 8 + xcd;
        unsigned* cnt = (unsigned*)smem;
        if (tid == 0) *cnt = 0;
        __syncthreads();
        if (wave < 4) pusher(p, cb * 2 + (wave & 1), wave >> 1, lane, cnt, smem + 1024 + wave * 6144);
        else if (wave == 4) releaser(p, lane, get_call(p), cnt);
    } else if (slot < 30) {
        const int i = slot - 4, h = xcd >> 2;
        const int tm = i < 24 ? 7 * h + i % 6 : 6;
        const int tn = (xcd & 3) * 4 + (i < 24 ? i / 6 : 2 * h + i - 24);
        const unsigned call = i < 16 ? stage_own(p, xcd * 16 + i, wave, lane, smem + wave * 16384) : get_call(p);
        gemm(p, tm, tn, smem, call, stage_of(call));
    }
    else if (blockIdx.x == 241) {
        if (wave == 0) sender(p, lane, get_call(p));
    }
    __syncthreads();
    if (tid == 0) __hip_atomic_fetch_add((gu32*)done, 1u, __ATOMIC_RELAXED, __HIP_MEMORY_SCOPE_AGENT);
}

}

extern "C" void kernel_launch(int rank, int nranks, ncclComm_t comm,
                              hipStream_t stream,
                              void* const* d_in, const int* in_sizes, int n_in,
                              void* d_out, int out_size,
                              const int* in_dtypes, int out_dtype,
                              void* const* d_out_all,
                              void* const* d_ws_all, size_t ws_bytes) {
    (void)comm; (void)in_dtypes; (void)out_dtype; (void)d_out_all;
    if (nranks != NR || (unsigned)rank >= (unsigned)NR || n_in < 2 || ws_bytes < WS_NEED ||
        in_sizes[0] != MM * KSH || in_sizes[1] != KSH * NN || out_size != MM * NN) {
        fprintf(stderr, "unexpected problem shape\n");
        return;
    }
    static bool attr_set = false;
    if (!attr_set) {
        HIP_CHECK(hipFuncSetAttribute((const void*)fused_gemm_allgather_silu,
                                      hipFuncAttributeMaxDynamicSharedMemorySize, LDS_BYTES));
        attr_set = true;
    }
    Params p;
    p.x = (const float*)d_in[0];
    p.w = (const float*)d_in[1];
    p.out = (float*)d_out;
    for (int i = 0; i < NR; ++i) p.ws[i] = (char*)d_ws_all[i];
    p.own = p.ws[rank];
    p.rank = rank;
    fused_gemm_allgather_silu<<<NWG, 512, LDS_BYTES, stream>>>(p);
    HIP_CHECK(hipGetLastError());
}
```

```cpp
#include <hip/hip_runtime.h>
#include <rccl/rccl.h>
#include <cstdio>

#define HIP_CHECK(expr) do { hipError_t _e = (expr); if (_e != hipSuccess) { \
    fprintf(stderr, "HIP error %s at %s:%d\n", hipGetErrorString(_e), __FILE__, __LINE__); \
    return; } } while (0)

#define LD_AUX 17
#define ST_AUX 16

namespace {

constexpr int NR = 8;
constexpr int MM = 4096, NN = 4096, KSH = 512;
constexpr int NSTEP = 64;
constexpr int NCOMM = 32;
constexpr int NSTAGE = 16;
constexpr int NWG = 256;
constexpr int TM = 320, TN = 256;
#define THROTTLE 4
#define ILV_E 1
#define ILV_L 0
#define LA 1
#define PSLEEP 4
#define PEEKW 3
#define LATE2_T 12
#define LATE_T 4

constexpr size_t OFF_DONE = 0;
constexpr size_t OFF_TMO = 128;
constexpr size_t OFF_LC = 4096;
constexpr size_t OFF_LCL = 12288;
constexpr size_t OFF_FLAG = 16384;
constexpr size_t CTRL_BYTES = 1u << 20;
constexpr size_t CH_BYTES = 65536;
constexpr size_t OP_BYTES = (size_t)NSTEP * NR * CH_BYTES;
#define SRC_MAJOR 1
constexpr unsigned J_STRIDE = SRC_MAJOR ? CH_BYTES : NR * CH_BYTES;
constexpr unsigned C_STRIDE = SRC_MAJOR ? NSTEP * CH_BYTES : CH_BYTES;
constexpr size_t WS_NEED = CTRL_BYTES + 4 * OP_BYTES;
static_assert(OFF_LC + NSTEP * 128 <= OFF_LCL && OFF_LCL + 8 * 128 <= OFF_FLAG, "layout");
static_assert(OFF_FLAG + NSTEP * NR * 128 <= CTRL_BYTES, "layout");

constexpr int A_STAGE = 8 * TM * 16;
constexpr int B_STAGE = 8 * TN * 16;
constexpr int STAGE = A_STAGE + B_STAGE;
constexpr int LDS_BYTES = 2 * STAGE;

constexpr unsigned long long DEADLINE_TICKS = 50000000ull;

typedef unsigned u32x4 __attribute__((ext_vector_type(4)));
typedef float f32x4 __attribute__((ext_vector_type(4)));
typedef _Float16 h16x8 __attribute__((ext_vector_type(8)));
typedef _Float16 h16x2 __attribute__((ext_vector_type(2)));
typedef __attribute__((address_space(1))) unsigned gu32;

struct Params {
    const float* x;
    const float* w;
    float* out;
    char* ws[NR];
    char* own;
    int rank;
};

template <class T>
__device__ __forceinline__ T* uni(T* p) {
    unsigned long long v = (unsigned long long)p;
    unsigned lo = __builtin_amdgcn_readfirstlane((unsigned)v);
    unsigned hi = __builtin_amdgcn_readfirstlane((unsigned)(v >> 32));
    return (T*)(((unsigned long long)hi << 32) | lo);
}
__device__ __forceinline__ unsigned pk16(float a, float b) {
    h16x2 v;
    v.x = (_Float16)a;
    v.y = (_Float16)b;
    return __builtin_bit_cast(unsigned, v);
}
typedef float f32x2 __attribute__((ext_vector_type(2)));
__device__ __forceinline__ f32x4 silu4(f32x4 v, f32x2 k) {
    const float lim = 60.0f * 1.44269504088896340736f;
    f32x2 a = f32x2{v.x, v.y} * k, b = f32x2{v.z, v.w} * k;
    a.x = __builtin_amdgcn_exp2f(__builtin_amdgcn_fmed3f(a.x, -lim, lim));
    a.y = __builtin_amdgcn_exp2f(__builtin_amdgcn_fmed3f(a.y, -lim, lim));
    b.x = __builtin_amdgcn_exp2f(__builtin_amdgcn_fmed3f(b.x, -lim, lim));
    b.y = __builtin_amdgcn_exp2f(__builtin_amdgcn_fmed3f(b.y, -lim, lim));
    a = a + 1.0f;
    b = b + 1.0f;
    a.x = __builtin_amdgcn_rcpf(a.x); a.y = __builtin_amdgcn_rcpf(a.y);
    b.x = __builtin_amdgcn_rcpf(b.x); b.y = __builtin_amdgcn_rcpf(b.y);
    a = a * f32x2{v.x, v.y};
    b = b * f32x2{v.z, v.w};
    return f32x4{a.x, a.y, b.x, b.y};
}
__device__ __forceinline__ unsigned ld_sys(const unsigned* p) {
    return __hip_atomic_load((gu32*)p, __ATOMIC_RELAXED, __HIP_MEMORY_SCOPE_SYSTEM);
}
__device__ __forceinline__ unsigned ld_agent(const unsigned* p) {
    return __hip_atomic_load((gu32*)p, __ATOMIC_RELAXED, __HIP_MEMORY_SCOPE_AGENT);
}
__device__ __forceinline__ void st_agent(unsigned* p, unsigned v) {
    __hip_atomic_store((gu32*)p, v, __ATOMIC_RELAXED, __HIP_MEMORY_SCOPE_AGENT);
}
__device__ __forceinline__ void st_sys(unsigned* p, unsigned v) {
    __hip_atomic_store((gu32*)p, v, __ATOMIC_RELAXED, __HIP_MEMORY_SCOPE_SYSTEM);
}
__device__ __forceinline__ unsigned get_call(const Params& p) {
    return __builtin_amdgcn_readfirstlane(ld_agent((const unsigned*)(p.own + OFF_DONE))) / NWG + 1;
}
__device__ __forceinline__ size_t stage_of(unsigned call) {
    return CTRL_BYTES + (size_t)(call & 1) * 2 * OP_BYTES;
}
struct FlagTrack {
    const char* f0;
    int stride;
    unsigned lim, call;
    __device__ __forceinline__ unsigned peek(int i0, int lane, unsigned& thr) const {
        int g = i0 + (lane & 7);
        g = g < NSTEP ? g : NSTEP - 1;
        thr = lim;
        unsigned v = lim - 1;
        if (lane < LA) v = ld_sys((const unsigned*)(f0 + g * stride));
        return v;
    }
};
struct StepTrack {
    const char* ws;
    unsigned call;
    int wave, rank;
    __device__ __forceinline__ unsigned peek(int i0, int lane, unsigned& thr) const {
        int s = i0 + (lane & 7);
        s = s < NSTEP ? s : NSTEP - 1;
        const int q = 8 * (s - 8) + wave;
        const int j = (q * 9363) >> 16;
        const int src = (rank + 1 + q - 7 * j) & 7;
        const int off = s < 8 ? (int)OFF_LCL + s * 128 : (int)OFF_FLAG + (j * NR + src) * 128;
        thr = s < 8 ? call * NSTAGE : call;
        unsigned v = 0;
        if (lane < LA) v = ld_sys((const unsigned*)(ws + off));
        return v;
    }
};
__device__ __forceinline__ int flag_count(unsigned v, unsigned thr) {
    unsigned m = (unsigned)__builtin_amdgcn_ballot_w64((int)(v - thr) >= 0) & ((1u << LA) - 1);
    return __builtin_ctz(~m);
}
template <class T>
__device__ __forceinline__ int wait_step(const T& tr, int g0, int lane, unsigned* tmo) {
    unsigned thr;
    unsigned v = tr.peek(g0, lane, thr);
    int c = flag_count(v, thr);
    if (c > 0) return g0 + c;
    unsigned long long t0 = __builtin_amdgcn_s_memrealtime();
    for (unsigned it = 1;; ++it) {
        v = tr.peek(g0, lane, thr);
        c = flag_count(v, thr);
        if (c > 0) return g0 + c;
        if ((it & 15) == 0) {
            if (__builtin_amdgcn_readfirstlane(ld_agent(tmo)) == tr.call ||
                __builtin_amdgcn_s_memrealtime() - t0 > DEADLINE_TICKS) {
                st_agent(tmo, tr.call);
                return NSTEP + 8;
            }
        }
        __builtin_amdgcn_s_sleep(PSLEEP);
    }
}

__device__ __forceinline__ void pusher(const Params& p, int piece, int phase, int lane,
                                       unsigned* cnt, char* ring) {
    const char* srcx = (const char*)(p.x + (size_t)piece * 64 * KSH) + lane * (KSH * 4);
    const char* srcw = (const char*)(p.w + piece * 64) + (lane >> 4) * (NN * 4) + (lane & 15) * 16;
    auto issue_load = [&](int j, int slot) {
        char* l = ring + slot * 2048;
        const bool isw = (j + phase) & 1;
        const char* g0 = isw ? srcw + (size_t)j * (8 * NN * 4) : srcx + j * 32;
        const char* g1 = g0 + (isw ? 4 * NN * 4 : 16);
        __builtin_amdgcn_global_load_lds((const gu32*)g0, (unsigned*)l, 16, 0, 0);
        __builtin_amdgcn_global_load_lds((const gu32*)g1, (unsigned*)(l + 1024), 16, 0, 0);
    };
    const unsigned cnt_addr = (unsigned)(size_t)(__attribute__((address_space(3))) unsigned*)cnt;
    const char* lc = uni(p.own + OFF_LC);
    auto signal = [&](int j) {
        if (lane == 0)
            asm volatile("global_atomic_add %0, %1, %2" :: "v"(j * 128), "v"(1u), "s"(lc) : "memory");
    };
    const char* base[NR - 1];
    const unsigned vbase = (unsigned)(p.rank * C_STRIDE + piece * 1024 + lane * 16);
    auto push = [&](int j, int slot) {
        const char* l = ring + slot * 2048;
        const bool isw = (j + phase) & 1;
        float s[8];
        if (isw) {
#pragma unroll
            for (int q = 0; q < 8; ++q) s[q] = *(const float*)(l + q * 256 + lane * 4);
        } else {
            *(f32x4*)&s[0] = *(const f32x4*)(l + lane * 16);
            *(f32x4*)&s[4] = *(const f32x4*)(l + 1024 + lane * 16);
        }
        u32x4 d;
        d.x = pk16(s[0], s[1]); d.y = pk16(s[2], s[3]); d.z = pk16(s[4], s[5]); d.w = pk16(s[6], s[7]);
        unsigned vo = vbase + (unsigned)j * J_STRIDE + (isw ? (unsigned)OP_BYTES : 0u);
#pragma unroll
        for (int q = 0; q < NR - 1; ++q)
            asm volatile("global_store_dwordx4 %0, %1, %2 sc0 sc1\n\ts_nop 1" :: "v"(vo), "v"(d), "s"(base[q]) : "memory");
    };
#define PUSH_WAIT(N) asm volatile("s_waitcnt vmcnt(" #N ")" ::: "memory")
    auto throttle = [&](int j) {
        if (j < THROTTLE) return;
        const unsigned need = j - THROTTLE + 1;
        unsigned v;
        asm volatile("ds_read_b32 %0, %1\n\ts_waitcnt lgkmcnt(0)" : "=v"(v) : "v"(cnt_addr) : "memory");
        if (__builtin_amdgcn_readfirstlane(v) >= need) return;
        unsigned long long t0 = __builtin_amdgcn_s_memrealtime();
        do {
            __builtin_amdgcn_s_sleep(2);
            asm volatile("ds_read_b32 %0, %1\n\ts_waitcnt lgkmcnt(0)" : "=v"(v) : "v"(cnt_addr) : "memory");
        } while (__builtin_amdgcn_readfirstlane(v) < need &&
                 __builtin_amdgcn_s_memrealtime() - t0 < DEADLINE_TICKS);
    };
    issue_load(0, 0);
    issue_load(1, 1);
    issue_load(2, 2);
    unsigned long long wsv[NR];
#pragma unroll
    for (int k = 0; k < NR; ++k) {
        wsv[k] = (unsigned long long)p.ws[k];
        asm volatile("" : "+s"(wsv[k]));
    }
    char* peer[NR - 1];
#pragma unroll
    for (int i = 0; i < NR - 1; ++i) {
        const int q = (p.rank + 1 + (i + piece) % 7) & 7;
        unsigned long long r = wsv[0];
#pragma unroll
        for (int k = 1; k < NR; ++k) r = q == k ? wsv[k] : r;
        peer[i] = (char*)r;
    }
    const size_t stage_off = stage_of(get_call(p));
#pragma unroll
    for (int i = 0; i < NR - 1; ++i) {
        base[i] = uni(peer[i] + stage_off);
    }
    asm volatile("s_nop 4" ::: "memory");
    PUSH_WAIT(4); push(0, 0);
    issue_load(3, 0); PUSH_WAIT(2); signal(0); push(1, 1);
#pragma unroll 1
    for (int j0 = 2; j0 < 62; j0 += 3) {
        issue_load(j0 + 2, 1); PUSH_WAIT(2); signal(j0 - 1); throttle(j0); push(j0, 2);
        issue_load(j0 + 3, 2); PUSH_WAIT(2); signal(j0); throttle(j0 + 1); push(j0 + 1, 0);
        issue_load(j0 + 4, 0); PUSH_WAIT(2); signal(j0 + 1); throttle(j0 + 2); push(j0 + 2, 1);
    }
    PUSH_WAIT(0); signal(61); throttle(62); push(62, 2);
    PUSH_WAIT(0); signal(62); throttle(63); push(63, 0);
    PUSH_WAIT(0); signal(63);
#undef PUSH_WAIT
}

__device__ __forceinline__ void sender(const Params& p, int lane, unsigned call) {
    unsigned* tmo = (unsigned*)(p.own + OFF_TMO);
    const FlagTrack tr{p.own + OFF_LC, 128, call * (4 * NCOMM), call};
    char* dst = p.ws[lane & 7] + OFF_FLAG + p.rank * 128;
    int sent = 0;
    while (sent < NSTEP) {
        int have = wait_step(tr, sent, lane, tmo);
        if (have > NSTEP) break;
        for (; sent < have; ++sent)
            if (lane < NR) st_sys((unsigned*)(dst + sent * (NR * 128)), call);
    }
}
__device__ __forceinline__ void releaser(const Params& p, int lane, unsigned call, unsigned* cnt) {
    unsigned* tmo = (unsigned*)(p.own + OFF_TMO);
    const FlagTrack tr{p.own + OFF_FLAG + p.rank * 128, NR * 128, call, call};
    int have = 0;
    while (have < NSTEP - THROTTLE) {
        have = wait_step(tr, have, lane, tmo);
        if (lane == 0) __hip_atomic_store(cnt, (unsigned)have, __ATOMIC_RELAXED, __HIP_MEMORY_SCOPE_WORKGROUP);
    }
}

__device__ __forceinline__ unsigned stage_own(const Params& p, int g, int wave, int lane, char* l) {
    const int b = g & 7;
    const int v = (g >> 3) * 8 + wave;
    const int piece = v >> 1;
    const bool isw = v & 1;
    const int vo = p.rank * (int)C_STRIDE + piece * 1024 + lane * 16;
    const int r = lane >> 4, c = lane & 15;
    const char* src = isw ? (const char*)(p.w + (size_t)(b * 64 + r) * NN + piece * 64)
                          : (const char*)(p.x + (size_t)(piece * 64 + r) * KSH + b * 64);
    const int rstride = isw ? 4 * NN * 4 : 4 * KSH * 4;
#pragma unroll
    for (int i = 0; i < 16; ++i) {
        const int slot = isw ? c : (c ^ ((4 * i + r) & 15));
        __builtin_amdgcn_global_load_lds((const gu32*)(src + (size_t)i * rstride + slot * 16),
                                         (unsigned*)(l + i * 1024), 16, 0, 0);
    }
    const unsigned call = get_call(p);
    const __amdgpu_buffer_rsrc_t rd = __builtin_amdgcn_make_buffer_rsrc(
        p.own + stage_of(call) + (isw ? OP_BYTES : 0), 0, (int)OP_BYTES, 0x00020000);
    asm volatile("s_waitcnt vmcnt(0)" ::: "memory");
    float cur[64];
    if (isw) {
#pragma unroll
        for (int k = 0; k < 64; ++k) cur[k] = *(const float*)(l + k * 256 + lane * 4);
    } else {
#pragma unroll
        for (int k = 0; k < 16; ++k) *(f32x4*)&cur[4 * k] = *(const f32x4*)(l + lane * 256 + ((k ^ (lane & 15)) * 16));
    }
#pragma unroll
    for (int o = 0; o < 8; ++o) {
        u32x4 d;
        d.x = pk16(cur[8 * o], cur[8 * o + 1]); d.y = pk16(cur[8 * o + 2], cur[8 * o + 3]);
        d.z = pk16(cur[8 * o + 4], cur[8 * o + 5]); d.w = pk16(cur[8 * o + 6], cur[8 * o + 7]);
        __builtin_amdgcn_raw_buffer_store_b128(d, rd, vo, (b * 8 + o) * (int)J_STRIDE, 17);
    }
    asm volatile("s_waitcnt vmcnt(0)" ::: "memory");
    __syncthreads();
    if (threadIdx.x == 0)
        __hip_atomic_fetch_add((gu32*)(p.own + OFF_LCL + b * 128), 1u, __ATOMIC_RELAXED, __HIP_MEMORY_SCOPE_AGENT);
    return call;
}

__device__ __forceinline__ void gemm(const Params& p, int tm, int tn, char* smem, unsigned call, size_t stage_off) {
    const int tid = threadIdx.x, lane = tid & 63;
    const int wave = __builtin_amdgcn_readfirstlane(tid >> 6);
    const int wr = wave >> 2, wc = wave & 3;
    const int row0 = tm * TM, col0 = tn * TN;
    char* ws = uni(p.own);
    unsigned* tmo = (unsigned*)(ws + OFF_TMO);
    const StepTrack tr{ws, call, wave, p.rank};
    const __amdgpu_buffer_rsrc_t rs =
        __builtin_amdgcn_make_buffer_rsrc(ws + stage_off, 0, (int)(2 * OP_BYTES), 0x00020000);
    const int vx = row0 * 16 + lane * 16;
    const int vw = (int)OP_BYTES + col0 * 16 + lane * 16;
    const bool full = row0 + TM <= MM;

    auto plane = [&](int st) {
        const int q = 8 * (st - 8) + wave;
        const int jj = (q * 9363) >> 16;
        return st < 8 ? (8 * st + wave) * (int)J_STRIDE + p.rank * (int)C_STRIDE
                      : jj * (int)J_STRIDE + ((p.rank + 1 + q - 7 * jj) & 7) * (int)C_STRIDE;
    };
    auto stage_in = [&](int j, int so) {
        __attribute__((address_space(3))) char* la =
            (__attribute__((address_space(3))) char*)(smem + (j & 1) * STAGE + wave * (TM * 16));
        __attribute__((address_space(3))) char* lb =
            (__attribute__((address_space(3))) char*)(smem + (j & 1) * STAGE + A_STAGE + wave * (TN * 16));
        __builtin_amdgcn_raw_ptr_buffer_load_lds(rs, la, 16, vx, so, 0, LD_AUX);
        __builtin_amdgcn_raw_ptr_buffer_load_lds(rs, la, 16, vx, so, 1024, LD_AUX);
        __builtin_amdgcn_raw_ptr_buffer_load_lds(rs, la, 16, vx, so, 2048, LD_AUX);
        __builtin_amdgcn_raw_ptr_buffer_load_lds(rs, la, 16, vx, so, 3072, LD_AUX);
        if (full)
            __builtin_amdgcn_raw_ptr_buffer_load_lds(rs, la + 4096, 16, vx + 4096, so, 0, LD_AUX);
        __builtin_amdgcn_raw_ptr_buffer_load_lds(rs, lb, 16, vw, so, 0, LD_AUX);
        __builtin_amdgcn_raw_ptr_buffer_load_lds(rs, lb, 16, vw, so, 1024, LD_AUX);
        __builtin_amdgcn_raw_ptr_buffer_load_lds(rs, lb, 16, vw, so, 2048, LD_AUX);
        __builtin_amdgcn_raw_ptr_buffer_load_lds(rs, lb, 16, vw, so, 3072, LD_AUX);
    };

    auto stage_one = [&](int j, int i, int so) {
        __attribute__((address_space(3))) char* la =
            (__attribute__((address_space(3))) char*)(smem + (j & 1) * STAGE + wave * (TM * 16));
        __attribute__((address_space(3))) char* lb =
            (__attribute__((address_space(3))) char*)(smem + (j & 1) * STAGE + A_STAGE + wave * (TN * 16));
        if (i < 4) __builtin_amdgcn_raw_ptr_buffer_load_lds(rs, la + i * 1024, 16, vx + i * 1024, so, 0, LD_AUX);
        else if (i == 4) { if (full) __builtin_amdgcn_raw_ptr_buffer_load_lds(rs, la + 4096, 16, vx + 4096, so, 0, LD_AUX); }
        else __builtin_amdgcn_raw_ptr_buffer_load_lds(rs, lb + (i - 5) * 1024, 16, vw + (i - 5) * 1024, so, 0, LD_AUX);
    };

    f32x4 acc[10][4];
#pragma unroll
    for (int m = 0; m < 10; ++m)
#pragma unroll
        for (int n = 0; n < 4; ++n) acc[m][n] = f32x4{0.f, 0.f, 0.f, 0.f};

    const int fr = lane & 15, fq = lane >> 4;
    const int xoff = fq * (TM * 16) + (wr * 160 + fr) * 16;
    const int woff = A_STAGE + fq * (TN * 16) + (wc * 64 + fr) * 16;

    int ready;
    ready = wait_step(tr, 0, lane, tmo);
    stage_in(0, plane(0));
    asm volatile("s_waitcnt vmcnt(0)" ::: "memory");
    __builtin_amdgcn_s_barrier();

    for (int j = 0; j < NSTEP - 1; ++j) {
        constexpr bool more = true;
        const bool pre = more && j + 1 < ready;
        const bool peek = ready < NSTEP && ready < j + PEEKW;
        const int ready0 = ready;
        unsigned peekv = 0, peekt = 0, peek2;
        asm volatile("v_mov_b32 %0, 0" : "=v"(peek2));
        if (peek) peekv = tr.peek(ready0, lane, peekt);
        bool late = false;
        const int so1 = __builtin_amdgcn_readfirstlane(plane(j + 1));
        const bool preE = pre && wr == 0, preL = pre && wr != 0;
        if (ILV_L == 0 && preL) stage_in(j + 1, so1);
        const char* sx = smem + (j & 1) * STAGE + xoff;
        const char* sw = smem + (j & 1) * STAGE + woff;
        auto ldx = [&](int t) { return *(const h16x8*)(sx + (t / 10) * 4 * (TM * 16) + (t % 10) * 256); };
        auto ldw = [&](int s, int n) { return *(const h16x8*)(sw + s * 4 * (TN * 16) + n * 256); };
        h16x8 wf[2][4], xf[4];
        __builtin_amdgcn_sched_barrier(0);
        wf[0][0] = ldw(0, 0); xf[0] = ldx(0);
        __builtin_amdgcn_sched_barrier(0);
        wf[0][1] = ldw(0, 1); wf[0][2] = ldw(0, 2); wf[0][3] = ldw(0, 3);
        xf[1] = ldx(1); xf[2] = ldx(2);
#pragma unroll
        for (int t = 0; t < 20; ++t) {
            __builtin_amdgcn_sched_barrier(0);
            acc[t % 10][0] = __builtin_amdgcn_mfma_f32_16x16x32_f16(wf[t / 10][0], xf[t & 3], acc[t % 10][0], 0, 0, 0);
            __builtin_amdgcn_sched_barrier(0);
            if (t + 3 < 20) xf[(t + 3) & 3] = ldx(t + 3);
            if (t >= 2 && t < 6) wf[1][t - 2] = ldw(1, t - 2);
#pragma unroll
            for (int i = 0; i < 9; ++i) {
                if (i / ILV_E == t && preE) stage_one(j + 1, i, so1);
                if (ILV_L > 0 && i / (ILV_L > 0 ? ILV_L : 1) == t && preL) stage_one(j + 1, i, so1);
            }
            if (t == LATE_T && more && !pre) {
                asm volatile("" : "+v"(peekv));
                ready = ready0 + flag_count(peekv, peekt);
                late = j + 1 < ready;
                if (late) stage_in(j + 1, so1);
            }
            if (t == LATE_T + 1 && more && !pre && !late) {
                unsigned th2;
                peek2 = tr.peek(ready0, lane, th2);
            }
            if (t == LATE2_T && more && !pre && !late) {
                asm volatile("" : "+v"(peek2));
                ready = ready0 + flag_count(peek2, peekt);
                late = j + 1 < ready;
                if (late) stage_in(j + 1, so1);
            }
            __builtin_amdgcn_sched_barrier(0);
#pragma unroll
            for (int n = 1; n < 4; ++n)
                acc[t % 10][n] = __builtin_amdgcn_mfma_f32_16x16x32_f16(wf[t / 10][n], xf[t & 3], acc[t % 10][n], 0, 0, 0);
        }
        __builtin_amdgcn_sched_barrier(0);
        {
            const int c = flag_count(peekv, peekt);
            if (peek && ready0 + c > ready) ready = ready0 + c;
        }
        if (more && !pre && !late) {
            if (j + 1 >= ready) ready = wait_step(tr, j + 1, lane, tmo);
            stage_in(j + 1, so1);
        }
        asm volatile("s_waitcnt vmcnt(0) lgkmcnt(0)" ::: "memory");
        __builtin_amdgcn_s_barrier();
    }

    const unsigned tmo_v = ld_agent(tmo);
    const int mlim = full ? 10 : (MM - row0 - wr * 160 + 15) / 16;
    const __amdgpu_buffer_rsrc_t ro =
        __builtin_amdgcn_make_buffer_rsrc(p.out, 0, (int)((size_t)MM * NN * 4), 0x00020000);
    const int vo = ((row0 + wr * 160 + fr) * NN + col0 + wc * 64 + fq * 4) * 4;
    f32x2 kneg = {-1.44269504088896340736f, -1.44269504088896340736f};
    asm volatile("" : "+v"(kneg));
    bool bad = false;
    {
        const char* sx = smem + ((NSTEP - 1) & 1) * STAGE + xoff;
        const char* sw = smem + ((NSTEP - 1) & 1) * STAGE + woff;
        auto ldx = [&](int s, int m) { return *(const h16x8*)(sx + s * 4 * (TM * 16) + m * 256); };
        auto ldw = [&](int s, int n) { return *(const h16x8*)(sw + s * 4 * (TN * 16) + n * 256); };
        auto store_group = [&](int m) {
            if (!bad && m < mlim) {
#pragma unroll
                for (int n = 0; n < 4; ++n)
                    __builtin_amdgcn_raw_buffer_store_b128(__builtin_bit_cast(u32x4, silu4(acc[m][n], kneg)), ro,
                                                           vo + m * (16 * NN * 4) + n * 64, 0, ST_AUX);
            }
        };
        h16x8 wf[2][4], xf[2][2];
        __builtin_amdgcn_sched_barrier(0);
        wf[0][0] = ldw(0, 0); xf[0][0] = ldx(0, 0);
        __builtin_amdgcn_sched_barrier(0);
        wf[0][1] = ldw(0, 1); wf[0][2] = ldw(0, 2); wf[0][3] = ldw(0, 3);
        xf[0][1] = ldx(1, 0);
        wf[1][0] = ldw(1, 0); wf[1][1] = ldw(1, 1); wf[1][2] = ldw(1, 2); wf[1][3] = ldw(1, 3);
#pragma unroll
        for (int m = 0; m < 10; ++m) {
            __builtin_amdgcn_sched_barrier(0);
            if (m + 1 < 10) { xf[(m + 1) & 1][0] = ldx(0, m + 1); xf[(m + 1) & 1][1] = ldx(1, m + 1); }
            __builtin_amdgcn_sched_barrier(0);
#pragma unroll
            for (int s = 0; s < 2; ++s)
#pragma unroll
                for (int n = 0; n < 4; ++n)
                    acc[m][n] = __builtin_amdgcn_mfma_f32_16x16x32_f16(wf[s][n], xf[m & 1][s], acc[m][n], 0, 0, 0);
            __builtin_amdgcn_sched_barrier(0);
            if (m == 1) bad = __builtin_amdgcn_readfirstlane(tmo_v) == call;
            if (m >= 1) store_group(m - 1);
        }
        __builtin_amdgcn_sched_barrier(0);
        store_group(9);
    }
    if (bad) {
        const u32x4 nan4 = {0x7fc00000u, 0x7fc00000u, 0x7fc00000u, 0x7fc00000u};
        for (int m = 0; m < mlim; ++m)
            for (int n = 0; n < 4; ++n)
                __builtin_amdgcn_raw_buffer_store_b128(nan4, ro, vo + m * (16 * NN * 4) + n * 64, 0, 0);
    }
}

__global__ __launch_bounds__(512, 2) void fused_gemm_allgather_silu(Params p) {
    extern __shared__ __attribute__((aligned(16))) char smem[];
    const int tid = threadIdx.x, lane = tid & 63;
    const int wave = __builtin_amdgcn_readfirstlane(tid >> 6);
    unsigned* done = (unsigned*)(p.own + OFF_DONE);
    const int xcd = blockIdx.x & 7, slot = blockIdx.x >> 3;
    if (slot < 4) {
        const int cb = slot * 8 + xcd;
        unsigned* cnt = (unsigned*)smem;
        if (tid == 0) *cnt = 0;
        __syncthreads();
        if (wave < 4) pusher(p, cb * 2 + (wave & 1), wave >> 1, lane, cnt, smem + 1024 + wave * 6144);
        else if (wave == 4) releaser(p, lane, get_call(p), cnt);
    } else if (slot < 30) {
        const int i = slot - 4, h = xcd >> 2;
        const int tm = i < 24 ? 7 * h + i % 6 : 6;
        const int tn = (xcd & 3) * 4 + (i < 24 ? i / 6 : 2 * h + i - 24);
        const unsigned call = i < 16 ? stage_own(p, xcd * 16 + i, wave, lane, smem + wave * 16384) : get_call(p);
        gemm(p, tm, tn, smem, call, stage_of(call));
    }
    else if (blockIdx.x == 241) {
        if (wave == 0) sender(p, lane, get_call(p));
    }
    __syncthreads();
    if (tid == 0) __hip_atomic_fetch_add((gu32*)done, 1u, __ATOMIC_RELAXED, __HIP_MEMORY_SCOPE_AGENT);
}

}

extern "C" void kernel_launch(int rank, int nranks, ncclComm_t comm,
                              hipStream_t stream,
                              void* const* d_in, const int* in_sizes, int n_in,
                              void* d_out, int out_size,
                              const int* in_dtypes, int out_dtype,
                              void* const* d_out_all,
                              void* const* d_ws_all, size_t ws_bytes) {
    (void)comm; (void)in_dtypes; (void)out_dtype; (void)d_out_all;
    if (nranks != NR || (unsigned)rank >= (unsigned)NR || n_in < 2 || ws_bytes < WS_NEED ||
        in_sizes[0] != MM * KSH || in_sizes[1] != KSH * NN || out_size != MM * NN) {
        fprintf(stderr, "unexpected problem shape\n");
        return;
    }
    static bool attr_set = false;
    if (!attr_set) {
        HIP_CHECK(hipFuncSetAttribute((const void*)fused_gemm_allgather_silu,
                                      hipFuncAttributeMaxDynamicSharedMemorySize, LDS_BYTES));
        attr_set = true;
    }
    Params p;
    p.x = (const float*)d_in[0];
    p.w = (const float*)d_in[1];
    p.out = (float*)d_out;
    for (int i = 0; i < NR; ++i) p.ws[i] = (char*)d_ws_all[i];
    p.own = p.ws[rank];
    p.rank = rank;
    fused_gemm_allgather_silu<<<NWG, 512, LDS_BYTES, stream>>>(p);
    HIP_CHECK(hipGetLastError());
}
```

```cpp
#include <hip/hip_runtime.h>
#include <rccl/rccl.h>
#include <cstdio>

#define HIP_CHECK(expr) do { hipError_t _e = (expr); if (_e != hipSuccess) { \
    fprintf(stderr, "HIP error %s at %s:%d\n", hipGetErrorString(_e), __FILE__, __LINE__); \
    return; } } while (0)

#define LD_AUX 17
#define ST_AUX 16

namespace {

constexpr int NR = 8;
constexpr int MM = 4096, NN = 4096, KSH = 512;
constexpr int NSTEP = 64;
constexpr int NCOMM = 32;
constexpr int NSTAGE = 16;
constexpr int NWG = 256;
constexpr int TM = 320, TN = 256;
#define THROTTLE 4
#define ILV_E 1
#define ILV_L 0
#define LA 1
#define PSLEEP 4
#define PEEKW 3
#define LATE2_T 12
#define LATE_T 4

constexpr size_t OFF_DONE = 0;
constexpr size_t OFF_TMO = 128;
constexpr size_t OFF_LC = 4096;
constexpr size_t OFF_LCL = 12288;
constexpr size_t OFF_FLAG = 16384;
constexpr size_t CTRL_BYTES = 1u << 20;
constexpr size_t CH_BYTES = 65536;
constexpr size_t OP_BYTES = (size_t)NSTEP * NR * CH_BYTES;
#define WARM_TLB 1
#define SRC_MAJOR 1
constexpr unsigned J_STRIDE = SRC_MAJOR ? CH_BYTES : NR * CH_BYTES;
constexpr unsigned C_STRIDE = SRC_MAJOR ? NSTEP * CH_BYTES : CH_BYTES;
static_assert((NSTEP - 1) * (size_t)J_STRIDE + (NR - 1) * (size_t)C_STRIDE + CH_BYTES <= OP_BYTES, "layout");
constexpr size_t WS_NEED = CTRL_BYTES + 4 * OP_BYTES;
static_assert(OFF_LC + NSTEP * 128 <= OFF_LCL && OFF_LCL + 8 * 128 <= OFF_FLAG, "layout");
static_assert(OFF_FLAG + NSTEP * NR * 128 <= CTRL_BYTES, "layout");

constexpr int A_STAGE = 8 * TM * 16;
constexpr int B_STAGE = 8 * TN * 16;
constexpr int STAGE = A_STAGE + B_STAGE;
constexpr int LDS_BYTES = 2 * STAGE;

constexpr unsigned long long DEADLINE_TICKS = 50000000ull;

typedef unsigned u32x4 __attribute__((ext_vector_type(4)));
typedef float f32x4 __attribute__((ext_vector_type(4)));
typedef _Float16 h16x8 __attribute__((ext_vector_type(8)));
typedef _Float16 h16x2 __attribute__((ext_vector_type(2)));
typedef __attribute__((address_space(1))) unsigned gu32;

struct Params {
    const float* x;
    const float* w;
    float* out;
    char* ws[NR];
    char* own;
    int rank;
};

template <class T>
__device__ __forceinline__ T* uni(T* p) {
    unsigned long long v = (unsigned long long)p;
    unsigned lo = __builtin_amdgcn_readfirstlane((unsigned)v);
    unsigned hi = __builtin_amdgcn_readfirstlane((unsigned)(v >> 32));
    return (T*)(((unsigned long long)hi << 32) | lo);
}
__device__ __forceinline__ unsigned pk16(float a, float b) {
    h16x2 v;
    v.x = (_Float16)a;
    v.y = (_Float16)b;
    return __builtin_bit_cast(unsigned, v);
}
typedef float f32x2 __attribute__((ext_vector_type(2)));
__device__ __forceinline__ f32x4 silu4(f32x4 v, f32x2 k) {
    const float lim = 60.0f * 1.44269504088896340736f;
    f32x2 a = f32x2{v.x, v.y} * k, b = f32x2{v.z, v.w} * k;
    a.x = __builtin_amdgcn_exp2f(__builtin_amdgcn_fmed3f(a.x, -lim, lim));
    a.y = __builtin_amdgcn_exp2f(__builtin_amdgcn_fmed3f(a.y, -lim, lim));
    b.x = __builtin_amdgcn_exp2f(__builtin_amdgcn_fmed3f(b.x, -lim, lim));
    b.y = __builtin_amdgcn_exp2f(__builtin_amdgcn_fmed3f(b.y, -lim, lim));
    a = a + 1.0f;
    b = b + 1.0f;
    a.x = __builtin_amdgcn_rcpf(a.x); a.y = __builtin_amdgcn_rcpf(a.y);
    b.x = __builtin_amdgcn_rcpf(b.x); b.y = __builtin_amdgcn_rcpf(b.y);
    a = a * f32x2{v.x, v.y};
    b = b * f32x2{v.z, v.w};
    return f32x4{a.x, a.y, b.x, b.y};
}
__device__ __forceinline__ unsigned ld_sys(const unsigned* p) {
    return __hip_atomic_load((gu32*)p, __ATOMIC_RELAXED, __HIP_MEMORY_SCOPE_SYSTEM);
}
__device__ __forceinline__ unsigned ld_agent(const unsigned* p) {
    return __hip_atomic_load((gu32*)p, __ATOMIC_RELAXED, __HIP_MEMORY_SCOPE_AGENT);
}
__device__ __forceinline__ void st_agent(unsigned* p, unsigned v) {
    __hip_atomic_store((gu32*)p, v, __ATOMIC_RELAXED, __HIP_MEMORY_SCOPE_AGENT);
}
__device__ __forceinline__ void st_sys(unsigned* p, unsigned v) {
    __hip_atomic_store((gu32*)p, v, __ATOMIC_RELAXED, __HIP_MEMORY_SCOPE_SYSTEM);
}
__device__ __forceinline__ unsigned get_call(const Params& p) {
    return __builtin_amdgcn_readfirstlane(ld_agent((const unsigned*)(p.own + OFF_DONE))) / NWG + 1;
}
__device__ __forceinline__ size_t stage_of(unsigned call) {
    return CTRL_BYTES + (size_t)(call & 1) * 2 * OP_BYTES;
}
struct FlagTrack {
    const char* f0;
    int stride;
    unsigned lim, call;
    __device__ __forceinline__ unsigned peek(int i0, int lane, unsigned& thr) const {
        int g = i0 + (lane & 7);
        g = g < NSTEP ? g : NSTEP - 1;
        thr = lim;
        unsigned v = lim - 1;
        if (lane < LA) v = ld_sys((const unsigned*)(f0 + g * stride));
        return v;
    }
};
struct StepTrack {
    const char* ws;
    unsigned call;
    int wave, rank;
    __device__ __forceinline__ unsigned peek(int i0, int lane, unsigned& thr) const {
        int s = i0 + (lane & 7);
        s = s < NSTEP ? s : NSTEP - 1;
        const int q = 8 * (s - 8) + wave;
        const int j = (q * 9363) >> 16;
        const int src = (rank + 1 + q - 7 * j) & 7;
        const int off = s < 8 ? (int)OFF_LCL + s * 128 : (int)OFF_FLAG + (j * NR + src) * 128;
        thr = s < 8 ? call * NSTAGE : call;
        unsigned v = 0;
        if (lane < LA) v = ld_sys((const unsigned*)(ws + off));
        return v;
    }
};
__device__ __forceinline__ int flag_count(unsigned v, unsigned thr) {
    unsigned m = (unsigned)__builtin_amdgcn_ballot_w64((int)(v - thr) >= 0) & ((1u << LA) - 1);
    return __builtin_ctz(~m);
}
template <class T>
__device__ __forceinline__ int wait_step(const T& tr, int g0, int lane, unsigned* tmo) {
    unsigned thr;
    unsigned v = tr.peek(g0, lane, thr);
    int c = flag_count(v, thr);
    if (c > 0) return g0 + c;
    unsigned long long t0 = __builtin_amdgcn_s_memrealtime();
    for (unsigned it = 1;; ++it) {
        v = tr.peek(g0, lane, thr);
        c = flag_count(v, thr);
        if (c > 0) return g0 + c;
        if ((it & 15) == 0) {
            if (__builtin_amdgcn_readfirstlane(ld_agent(tmo)) == tr.call ||
                __builtin_amdgcn_s_memrealtime() - t0 > DEADLINE_TICKS) {
                st_agent(tmo, tr.call);
                return NSTEP + 8;
            }
        }
        __builtin_amdgcn_s_sleep(PSLEEP);
    }
}

__device__ __forceinline__ void pusher(const Params& p, int piece, int phase, int lane,
                                       unsigned* cnt, char* ring) {
    const char* srcx = (const char*)(p.x + (size_t)piece * 64 * KSH) + lane * (KSH * 4);
    const char* srcw = (const char*)(p.w + piece * 64) + (lane >> 4) * (NN * 4) + (lane & 15) * 16;
    auto issue_load = [&](int j, int slot) {
        char* l = ring + slot * 2048;
        const bool isw = (j + phase) & 1;
        const char* g0 = isw ? srcw + (size_t)j * (8 * NN * 4) : srcx + j * 32;
        const char* g1 = g0 + (isw ? 4 * NN * 4 : 16);
        __builtin_amdgcn_global_load_lds((const gu32*)g0, (unsigned*)l, 16, 0, 0);
        __builtin_amdgcn_global_load_lds((const gu32*)g1, (unsigned*)(l + 1024), 16, 0, 0);
    };
    const unsigned cnt_addr = (unsigned)(size_t)(__attribute__((address_space(3))) unsigned*)cnt;
    const char* lc = uni(p.own + OFF_LC);
    auto signal = [&](int j) {
        if (lane == 0)
            asm volatile("global_atomic_add %0, %1, %2" :: "v"(j * 128), "v"(1u), "s"(lc) : "memory");
    };
    const char* base[NR - 1];
    const unsigned vbase = (unsigned)(p.rank * C_STRIDE + piece * 1024 + lane * 16);
    auto push = [&](int j, int slot) {
        const char* l = ring + slot * 2048;
        const bool isw = (j + phase) & 1;
        float s[8];
        if (isw) {
#pragma unroll
            for (int q = 0; q < 8; ++q) s[q] = *(const float*)(l + q * 256 + lane * 4);
        } else {
            *(f32x4*)&s[0] = *(const f32x4*)(l + lane * 16);
            *(f32x4*)&s[4] = *(const f32x4*)(l + 1024 + lane * 16);
        }
        u32x4 d;
        d.x = pk16(s[0], s[1]); d.y = pk16(s[2], s[3]); d.z = pk16(s[4], s[5]); d.w = pk16(s[6], s[7]);
        unsigned vo = vbase + (unsigned)j * J_STRIDE + (isw ? (unsigned)OP_BYTES : 0u);
#pragma unroll
        for (int q = 0; q < NR - 1; ++q)
            asm volatile("global_store_dwordx4 %0, %1, %2 sc0 sc1\n\ts_nop 1" :: "v"(vo), "v"(d), "s"(base[q]) : "memory");
    };
#define PUSH_WAIT(N) asm volatile("s_waitcnt vmcnt(" #N ")" ::: "memory")
    auto throttle = [&](int j) {
        if (j < THROTTLE) return;
        const unsigned need = j - THROTTLE + 1;
        unsigned v;
        asm volatile("ds_read_b32 %0, %1\n\ts_waitcnt lgkmcnt(0)" : "=v"(v) : "v"(cnt_addr) : "memory");
        if (__builtin_amdgcn_readfirstlane(v) >= need) return;
        unsigned long long t0 = __builtin_amdgcn_s_memrealtime();
        do {
            __builtin_amdgcn_s_sleep(2);
            asm volatile("ds_read_b32 %0, %1\n\ts_waitcnt lgkmcnt(0)" : "=v"(v) : "v"(cnt_addr) : "memory");
        } while (__builtin_amdgcn_readfirstlane(v) < need &&
                 __builtin_amdgcn_s_memrealtime() - t0 < DEADLINE_TICKS);
    };
    issue_load(0, 0);
    issue_load(1, 1);
    issue_load(2, 2);
    unsigned long long wsv[NR];
#pragma unroll
    for (int k = 0; k < NR; ++k) {
        wsv[k] = (unsigned long long)p.ws[k];
        asm volatile("" : "+s"(wsv[k]));
    }
    char* peer[NR - 1];
#pragma unroll
    for (int i = 0; i < NR - 1; ++i) {
        const int q = (p.rank + 1 + (i + piece) % 7) & 7;
        unsigned long long r = wsv[0];
#pragma unroll
        for (int k = 1; k < NR; ++k) r = q == k ? wsv[k] : r;
        peer[i] = (char*)r;
    }
    const size_t stage_off = stage_of(get_call(p));
#pragma unroll
    for (int i = 0; i < NR - 1; ++i) {
        base[i] = uni(peer[i] + stage_off);
    }
    asm volatile("s_nop 4" ::: "memory");
    PUSH_WAIT(4); push(0, 0);
    issue_load(3, 0); PUSH_WAIT(2); signal(0); push(1, 1);
#pragma unroll 1
    for (int j0 = 2; j0 < 62; j0 += 3) {
        issue_load(j0 + 2, 1); PUSH_WAIT(2); signal(j0 - 1); throttle(j0); push(j0, 2);
        issue_load(j0 + 3, 2); PUSH_WAIT(2); signal(j0); throttle(j0 + 1); push(j0 + 1, 0);
        issue_load(j0 + 4, 0); PUSH_WAIT(2); signal(j0 + 1); throttle(j0 + 2); push(j0 + 2, 1);
    }
    PUSH_WAIT(0); signal(61); throttle(62); push(62, 2);
    PUSH_WAIT(0); signal(62); throttle(63); push(63, 0);
    PUSH_WAIT(0); signal(63);
#undef PUSH_WAIT
}

__device__ __forceinline__ void sender(const Params& p, int lane, unsigned call) {
    unsigned* tmo = (unsigned*)(p.own + OFF_TMO);
    const FlagTrack tr{p.own + OFF_LC, 128, call * (4 * NCOMM), call};
    char* dst = p.ws[lane & 7] + OFF_FLAG + p.rank * 128;
    int sent = 0;
    while (sent < NSTEP) {
        int have = wait_step(tr, sent, lane, tmo);
        if (have > NSTEP) break;
        for (; sent < have; ++sent)
            if (lane < NR) st_sys((unsigned*)(dst + sent * (NR * 128)), call);
    }
}
__device__ __forceinline__ void releaser(const Params& p, int lane, unsigned call, unsigned* cnt) {
    unsigned* tmo = (unsigned*)(p.own + OFF_TMO);
    const FlagTrack tr{p.own + OFF_FLAG + p.rank * 128, NR * 128, call, call};
    int have = 0;
    while (have < NSTEP - THROTTLE) {
        have = wait_step(tr, have, lane, tmo);
        if (lane == 0) __hip_atomic_store(cnt, (unsigned)have, __ATOMIC_RELAXED, __HIP_MEMORY_SCOPE_WORKGROUP);
    }
}

__device__ __forceinline__ unsigned stage_own(const Params& p, int g, int wave, int lane, char* l) {
    const int b = g & 7;
    const int v = (g >> 3) * 8 + wave;
    const int piece = v >> 1;
    const bool isw = v & 1;
    const int vo = p.rank * (int)C_STRIDE + piece * 1024 + lane * 16;
    const int r = lane >> 4, c = lane & 15;
    const char* src = isw ? (const char*)(p.w + (size_t)(b * 64 + r) * NN + piece * 64)
                          : (const char*)(p.x + (size_t)(piece * 64 + r) * KSH + b * 64);
    const int rstride = isw ? 4 * NN * 4 : 4 * KSH * 4;
#pragma unroll
    for (int i = 0; i < 16; ++i) {
        const int slot = isw ? c : (c ^ ((4 * i + r) & 15));
        __builtin_amdgcn_global_load_lds((const gu32*)(src + (size_t)i * rstride + slot * 16),
                                         (unsigned*)(l + i * 1024), 16, 0, 0);
    }
    const unsigned call = get_call(p);
    const __amdgpu_buffer_rsrc_t rd = __builtin_amdgcn_make_buffer_rsrc(
        p.own + stage_of(call) + (isw ? OP_BYTES : 0), 0, (int)OP_BYTES, 0x00020000);
    asm volatile("s_waitcnt vmcnt(0)" ::: "memory");
    float cur[64];
    if (isw) {
#pragma unroll
        for (int k = 0; k < 64; ++k) cur[k] = *(const float*)(l + k * 256 + lane * 4);
    } else {
#pragma unroll
        for (int k = 0; k < 16; ++k) *(f32x4*)&cur[4 * k] = *(const f32x4*)(l + lane * 256 + ((k ^ (lane & 15)) * 16));
    }
#pragma unroll
    for (int o = 0; o < 8; ++o) {
        u32x4 d;
        d.x = pk16(cur[8 * o], cur[8 * o + 1]); d.y = pk16(cur[8 * o + 2], cur[8 * o + 3]);
        d.z = pk16(cur[8 * o + 4], cur[8 * o + 5]); d.w = pk16(cur[8 * o + 6], cur[8 * o + 7]);
        __builtin_amdgcn_raw_buffer_store_b128(d, rd, vo, (b * 8 + o) * (int)J_STRIDE, 17);
    }
    asm volatile("s_waitcnt vmcnt(0)" ::: "memory");
    __syncthreads();
    if (threadIdx.x == 0)
        __hip_atomic_fetch_add((gu32*)(p.own + OFF_LCL + b * 128), 1u, __ATOMIC_RELAXED, __HIP_MEMORY_SCOPE_AGENT);
    return call;
}

__device__ __forceinline__ void gemm(const Params& p, int tm, int tn, char* smem, unsigned call, size_t stage_off) {
    const int tid = threadIdx.x, lane = tid & 63;
    const int wave = __builtin_amdgcn_readfirstlane(tid >> 6);
    const int wr = wave >> 2, wc = wave & 3;
    const int row0 = tm * TM, col0 = tn * TN;
    char* ws = uni(p.own);
    unsigned* tmo = (unsigned*)(ws + OFF_TMO);
    const StepTrack tr{ws, call, wave, p.rank};
    const __amdgpu_buffer_rsrc_t rs =
        __builtin_amdgcn_make_buffer_rsrc(ws + stage_off, 0, (int)(2 * OP_BYTES), 0x00020000);
    const int vx = row0 * 16 + lane * 16;
    const int vw = (int)OP_BYTES + col0 * 16 + lane * 16;
    const bool full = row0 + TM <= MM;

    auto plane = [&](int st) {
        const int q = 8 * (st - 8) + wave;
        const int jj = (q * 9363) >> 16;
        return st < 8 ? (8 * st + wave) * (int)J_STRIDE + p.rank * (int)C_STRIDE
                      : jj * (int)J_STRIDE + ((p.rank + 1 + q - 7 * jj) & 7) * (int)C_STRIDE;
    };
    auto stage_in = [&](int j, int so) {
        __attribute__((address_space(3))) char* la =
            (__attribute__((address_space(3))) char*)(smem + (j & 1) * STAGE + wave * (TM * 16));
        __attribute__((address_space(3))) char* lb =
            (__attribute__((address_space(3))) char*)(smem + (j & 1) * STAGE + A_STAGE + wave * (TN * 16));
        __builtin_amdgcn_raw_ptr_buffer_load_lds(rs, la, 16, vx, so, 0, LD_AUX);
        __builtin_amdgcn_raw_ptr_buffer_load_lds(rs, la, 16, vx, so, 1024, LD_AUX);
        __builtin_amdgcn_raw_ptr_buffer_load_lds(rs, la, 16, vx, so, 2048, LD_AUX);
        __builtin_amdgcn_raw_ptr_buffer_load_lds(rs, la, 16, vx, so, 3072, LD_AUX);
        if (full)
            __builtin_amdgcn_raw_ptr_buffer_load_lds(rs, la + 4096, 16, vx + 4096, so, 0, LD_AUX);
        __builtin_amdgcn_raw_ptr_buffer_load_lds(rs, lb, 16, vw, so, 0, LD_AUX);
        __builtin_amdgcn_raw_ptr_buffer_load_lds(rs, lb, 16, vw, so, 1024, LD_AUX);
        __builtin_amdgcn_raw_ptr_buffer_load_lds(rs, lb, 16, vw, so, 2048, LD_AUX);
        __builtin_amdgcn_raw_ptr_buffer_load_lds(rs, lb, 16, vw, so, 3072, LD_AUX);
    };

    auto stage_one = [&](int j, int i, int so) {
        __attribute__((address_space(3))) char* la =
            (__attribute__((address_space(3))) char*)(smem + (j & 1) * STAGE + wave * (TM * 16));
        __attribute__((address_space(3))) char* lb =
            (__attribute__((address_space(3))) char*)(smem + (j & 1) * STAGE + A_STAGE + wave * (TN * 16));
        if (i < 4) __builtin_amdgcn_raw_ptr_buffer_load_lds(rs, la + i * 1024, 16, vx + i * 1024, so, 0, LD_AUX);
        else if (i == 4) { if (full) __builtin_amdgcn_raw_ptr_buffer_load_lds(rs, la + 4096, 16, vx + 4096, so, 0, LD_AUX); }
        else __builtin_amdgcn_raw_ptr_buffer_load_lds(rs, lb + (i - 5) * 1024, 16, vw + (i - 5) * 1024, so, 0, LD_AUX);
    };

    f32x4 acc[10][4];
#pragma unroll
    for (int m = 0; m < 10; ++m)
#pragma unroll
        for (int n = 0; n < 4; ++n) acc[m][n] = f32x4{0.f, 0.f, 0.f, 0.f};

    const int fr = lane & 15, fq = lane >> 4;
    const int xoff = fq * (TM * 16) + (wr * 160 + fr) * 16;
    const int woff = A_STAGE + fq * (TN * 16) + (wc * 64 + fr) * 16;

    int ready;
    ready = wait_step(tr, 0, lane, tmo);
    stage_in(0, plane(0));
    asm volatile("s_waitcnt vmcnt(0)" ::: "memory");
    __builtin_amdgcn_s_barrier();

    for (int j = 0; j < NSTEP - 1; ++j) {
        constexpr bool more = true;
        const bool pre = more && j + 1 < ready;
        const bool peek = ready < NSTEP && ready < j + PEEKW;
        const int ready0 = ready;
        unsigned peekv = 0, peekt = 0, peek2;
        asm volatile("v_mov_b32 %0, 0" : "=v"(peek2));
        if (peek) peekv = tr.peek(ready0, lane, peekt);
        bool late = false;
        const int so1 = __builtin_amdgcn_readfirstlane(plane(j + 1));
        const bool preE = pre && wr == 0, preL = pre && wr != 0;
        if (ILV_L == 0 && preL) stage_in(j + 1, so1);
        const char* sx = smem + (j & 1) * STAGE + xoff;
        const char* sw = smem + (j & 1) * STAGE + woff;
        auto ldx = [&](int t) { return *(const h16x8*)(sx + (t / 10) * 4 * (TM * 16) + (t % 10) * 256); };
        auto ldw = [&](int s, int n) { return *(const h16x8*)(sw + s * 4 * (TN * 16) + n * 256); };
        h16x8 wf[2][4], xf[4];
        __builtin_amdgcn_sched_barrier(0);
        wf[0][0] = ldw(0, 0); xf[0] = ldx(0);
        __builtin_amdgcn_sched_barrier(0);
        wf[0][1] = ldw(0, 1); wf[0][2] = ldw(0, 2); wf[0][3] = ldw(0, 3);
        xf[1] = ldx(1); xf[2] = ldx(2);
#pragma unroll
        for (int t = 0; t < 20; ++t) {
            __builtin_amdgcn_sched_barrier(0);
            acc[t % 10][0] = __builtin_amdgcn_mfma_f32_16x16x32_f16(wf[t / 10][0], xf[t & 3], acc[t % 10][0], 0, 0, 0);
            __builtin_amdgcn_sched_barrier(0);
            if (t + 3 < 20) xf[(t + 3) & 3] = ldx(t + 3);
            if (t >= 2 && t < 6) wf[1][t - 2] = ldw(1, t - 2);
#pragma unroll
            for (int i = 0; i < 9; ++i) {
                if (i / ILV_E == t && preE) stage_one(j + 1, i, so1);
                if (ILV_L > 0 && i / (ILV_L > 0 ? ILV_L : 1) == t && preL) stage_one(j + 1, i, so1);
            }
            if (t == LATE_T && more && !pre) {
                asm volatile("" : "+v"(peekv));
                ready = ready0 + flag_count(peekv, peekt);
                late = j + 1 < ready;
                if (late) stage_in(j + 1, so1);
            }
            if (t == LATE_T + 1 && more && !pre && !late) {
                unsigned th2;
                peek2 = tr.peek(ready0, lane, th2);
            }
            if (t == LATE2_T && more && !pre && !late) {
                asm volatile("" : "+v"(peek2));
                ready = ready0 + flag_count(peek2, peekt);
                late = j + 1 < ready;
                if (late) stage_in(j + 1, so1);
            }
            __builtin_amdgcn_sched_barrier(0);
#pragma unroll
            for (int n = 1; n < 4; ++n)
                acc[t % 10][n] = __builtin_amdgcn_mfma_f32_16x16x32_f16(wf[t / 10][n], xf[t & 3], acc[t % 10][n], 0, 0, 0);
        }
        __builtin_amdgcn_sched_barrier(0);
        {
            const int c = flag_count(peekv, peekt);
            if (peek && ready0 + c > ready) ready = ready0 + c;
        }
        if (more && !pre && !late) {
            if (j + 1 >= ready) ready = wait_step(tr, j + 1, lane, tmo);
            stage_in(j + 1, so1);
        }
        asm volatile("s_waitcnt vmcnt(0) lgkmcnt(0)" ::: "memory");
        __builtin_amdgcn_s_barrier();
    }

    const unsigned tmo_v = ld_agent(tmo);
    const int mlim = full ? 10 : (MM - row0 - wr * 160 + 15) / 16;
    const __amdgpu_buffer_rsrc_t ro =
        __builtin_amdgcn_make_buffer_rsrc(p.out, 0, (int)((size_t)MM * NN * 4), 0x00020000);
    const int vo = ((row0 + wr * 160 + fr) * NN + col0 + wc * 64 + fq * 4) * 4;
    f32x2 kneg = {-1.44269504088896340736f, -1.44269504088896340736f};
    asm volatile("" : "+v"(kneg));
    bool bad = false;
    {
        const char* sx = smem + ((NSTEP - 1) & 1) * STAGE + xoff;
        const char* sw = smem + ((NSTEP - 1) & 1) * STAGE + woff;
        auto ldx = [&](int s, int m) { return *(const h16x8*)(sx + s * 4 * (TM * 16) + m * 256); };
        auto ldw = [&](int s, int n) { return *(const h16x8*)(sw + s * 4 * (TN * 16) + n * 256); };
        auto store_group = [&](int m) {
            if (!bad && m < mlim) {
#pragma unroll
                for (int n = 0; n < 4; ++n)
                    __builtin_amdgcn_raw_buffer_store_b128(__builtin_bit_cast(u32x4, silu4(acc[m][n], kneg)), ro,
                                                           vo + m * (16 * NN * 4) + n * 64, 0, ST_AUX);
            }
        };
        h16x8 wf[2][4], xf[2][2];
        __builtin_amdgcn_sched_barrier(0);
        wf[0][0] = ldw(0, 0); xf[0][0] = ldx(0, 0);
        __builtin_amdgcn_sched_barrier(0);
        wf[0][1] = ldw(0, 1); wf[0][2] = ldw(0, 2); wf[0][3] = ldw(0, 3);
        xf[0][1] = ldx(1, 0);
        wf[1][0] = ldw(1, 0); wf[1][1] = ldw(1, 1); wf[1][2] = ldw(1, 2); wf[1][3] = ldw(1, 3);
#pragma unroll
        for (int m = 0; m < 10; ++m) {
            __builtin_amdgcn_sched_barrier(0);
            if (m + 1 < 10) { xf[(m + 1) & 1][0] = ldx(0, m + 1); xf[(m + 1) & 1][1] = ldx(1, m + 1); }
            __builtin_amdgcn_sched_barrier(0);
#pragma unroll
            for (int s = 0; s < 2; ++s)
#pragma unroll
                for (int n = 0; n < 4; ++n)
                    acc[m][n] = __builtin_amdgcn_mfma_f32_16x16x32_f16(wf[s][n], xf[m & 1][s], acc[m][n], 0, 0, 0);
            __builtin_amdgcn_sched_barrier(0);
            if (m == 1) bad = __builtin_amdgcn_readfirstlane(tmo_v) == call;
            if (m >= 1) store_group(m - 1);
        }
        __builtin_amdgcn_sched_barrier(0);
        store_group(9);
    }
    if (bad) {
        const u32x4 nan4 = {0x7fc00000u, 0x7fc00000u, 0x7fc00000u, 0x7fc00000u};
        for (int m = 0; m < mlim; ++m)
            for (int n = 0; n < 4; ++n)
                __builtin_amdgcn_raw_buffer_store_b128(nan4, ro, vo + m * (16 * NN * 4) + n * 64, 0, 0);
    }
}

__global__ __launch_bounds__(512, 2) void fused_gemm_allgather_silu(Params p) {
    extern __shared__ __attribute__((aligned(16))) char smem[];
    const int tid = threadIdx.x, lane = tid & 63;
    const int wave = __builtin_amdgcn_readfirstlane(tid >> 6);
    unsigned* done = (unsigned*)(p.own + OFF_DONE);
    const int xcd = blockIdx.x & 7, slot = blockIdx.x >> 3;
    if (slot < 4) {
        const int cb = slot * 8 + xcd;
        unsigned* cnt = (unsigned*)smem;
        if (tid == 0) *cnt = 0;
        __syncthreads();
        if (wave < 4) pusher(p, cb * 2 + (wave & 1), wave >> 1, lane, cnt, smem + 1024 + wave * 6144);
        else if (wave == 4) releaser(p, lane, get_call(p), cnt);
    } else if (slot < 30) {
        const int i = slot - 4, h = xcd >> 2;
        const int tm = i < 24 ? 7 * h + i % 6 : 6;
        const int tn = (xcd & 3) * 4 + (i < 24 ? i / 6 : 2 * h + i - 24);
        const unsigned call = i < 16 ? stage_own(p, xcd * 16 + i, wave, lane, smem + wave * 16384) : get_call(p);
        gemm(p, tm, tn, smem, call, stage_of(call));
    }
    else if (blockIdx.x == 241) {
        if (wave == 0) sender(p, lane, get_call(p));
    }
    else if (slot == 31 && wave == 0) {
        const unsigned q = lane & 7;
        if (q != (unsigned)p.rank) {
            const char* d = p.ws[q] + CTRL_BYTES + (size_t)((lane >> 3) & 3) * OP_BYTES +
                            (size_t)p.rank * C_STRIDE + (size_t)(lane >> 5) * (C_STRIDE / 2);
            const unsigned a = ld_sys((const unsigned*)d);
            const unsigned b = ld_sys((const unsigned*)(d + C_STRIDE / 4));
            asm volatile("" :: "v"(a), "v"(b));
        }
    }
    __syncthreads();
    if (tid == 0) __hip_atomic_fetch_add((gu32*)done, 1u, __ATOMIC_RELAXED, __HIP_MEMORY_SCOPE_AGENT);
}

}

extern "C" void kernel_launch(int rank, int nranks, ncclComm_t comm,
                              hipStream_t stream,
                              void* const* d_in, const int* in_sizes, int n_in,
                              void* d_out, int out_size,
                              const int* in_dtypes, int out_dtype,
                              void* const* d_out_all,
                              void* const* d_ws_all, size_t ws_bytes) {
    (void)comm; (void)in_dtypes; (void)out_dtype; (void)d_out_all;
    if (nranks != NR || (unsigned)rank >= (unsigned)NR || n_in < 2 || ws_bytes < WS_NEED ||
        in_sizes[0] != MM * KSH || in_sizes[1] != KSH * NN || out_size != MM * NN) {
        fprintf(stderr, "unexpected problem shape\n");
        return;
    }
    static bool attr_set = false;
    if (!attr_set) {
        HIP_CHECK(hipFuncSetAttribute((const void*)fused_gemm_allgather_silu,
                                      hipFuncAttributeMaxDynamicSharedMemorySize, LDS_BYTES));
        attr_set = true;
    }
    Params p;
    p.x = (const float*)d_in[0];
    p.w = (const float*)d_in[1];
    p.out = (float*)d_out;
    for (int i = 0; i < NR; ++i) p.ws[i] = (char*)d_ws_all[i];
    p.own = p.ws[rank];
    p.rank = rank;
    fused_gemm_allgather_silu<<<NWG, 512, LDS_BYTES, stream>>>(p);
    HIP_CHECK(hipGetLastError());
}
```

```cpp
#include <hip/hip_runtime.h>
#include <rccl/rccl.h>
#include <cstdio>

#define HIP_CHECK(expr) do { hipError_t _e = (expr); if (_e != hipSuccess) { \
    fprintf(stderr, "HIP error %s at %s:%d\n", hipGetErrorString(_e), __FILE__, __LINE__); \
    return; } } while (0)

#define LD_AUX 17
#define ST_AUX 16

namespace {

constexpr int NR = 8;
constexpr int MM = 4096, NN = 4096, KSH = 512;
constexpr int NSTEP = 64;
constexpr int NCOMM = 32;
constexpr int NSTAGE = 16;
constexpr int NWG = 256;
constexpr int TM = 320, TN = 256;
#define THROTTLE 4
#define ILV_E 1
#define ILV_L 0
#define LA 1
#define PSLEEP 4
#define PEEKW 3
#define LATE2_T 12
#define LATE_T 4

constexpr size_t OFF_DONE = 0;
constexpr size_t OFF_TMO = 128;
constexpr size_t OFF_LC = 4096;
constexpr size_t OFF_LCL = 12288;
constexpr size_t OFF_FLAG = 16384;
constexpr size_t CTRL_BYTES = 1u << 20;
constexpr size_t CH_BYTES = 65536;
constexpr size_t OP_BYTES = (size_t)NSTEP * NR * CH_BYTES;
#define WARM_TLB 2
#define SRC_MAJOR 1
constexpr unsigned J_STRIDE = SRC_MAJOR ? CH_BYTES : NR * CH_BYTES;
constexpr unsigned C_STRIDE = SRC_MAJOR ? NSTEP * CH_BYTES : CH_BYTES;
static_assert((size_t)MM * NN * 4 == (64u << 20) && 4 * OP_BYTES == (128u << 20), "page touches");
static_assert((NSTEP - 1) * (size_t)J_STRIDE + (NR - 1) * (size_t)C_STRIDE + CH_BYTES <= OP_BYTES, "layout");
constexpr size_t WS_NEED = CTRL_BYTES + 4 * OP_BYTES;
static_assert(OFF_LC + NSTEP * 128 <= OFF_LCL && OFF_LCL + 8 * 128 <= OFF_FLAG, "layout");
static_assert(OFF_FLAG + NSTEP * NR * 128 <= CTRL_BYTES, "layout");

constexpr int A_STAGE = 8 * TM * 16;
constexpr int B_STAGE = 8 * TN * 16;
constexpr int STAGE = A_STAGE + B_STAGE;
constexpr int LDS_BYTES = 2 * STAGE;

constexpr unsigned long long DEADLINE_TICKS = 50000000ull;

typedef unsigned u32x4 __attribute__((ext_vector_type(4)));
typedef float f32x4 __attribute__((ext_vector_type(4)));
typedef _Float16 h16x8 __attribute__((ext_vector_type(8)));
typedef _Float16 h16x2 __attribute__((ext_vector_type(2)));
typedef __attribute__((address_space(1))) unsigned gu32;

struct Params {
    const float* x;
    const float* w;
    float* out;
    char* ws[NR];
    char* own;
    int rank;
};

template <class T>
__device__ __forceinline__ T* uni(T* p) {
    unsigned long long v = (unsigned long long)p;
    unsigned lo = __builtin_amdgcn_readfirstlane((unsigned)v);
    unsigned hi = __builtin_amdgcn_readfirstlane((unsigned)(v >> 32));
    return (T*)(((unsigned long long)hi << 32) | lo);
}
__device__ __forceinline__ unsigned pk16(float a, float b) {
    h16x2 v;
    v.x = (_Float16)a;
    v.y = (_Float16)b;
    return __builtin_bit_cast(unsigned, v);
}
typedef float f32x2 __attribute__((ext_vector_type(2)));
__device__ __forceinline__ f32x4 silu4(f32x4 v, f32x2 k) {
    const float lim = 60.0f * 1.44269504088896340736f;
    f32x2 a = f32x2{v.x, v.y} * k, b = f32x2{v.z, v.w} * k;
    a.x = __builtin_amdgcn_exp2f(__builtin_amdgcn_fmed3f(a.x, -lim, lim));
    a.y = __builtin_amdgcn_exp2f(__builtin_amdgcn_fmed3f(a.y, -lim, lim));
    b.x = __builtin_amdgcn_exp2f(__builtin_amdgcn_fmed3f(b.x, -lim, lim));
    b.y = __builtin_amdgcn_exp2f(__builtin_amdgcn_fmed3f(b.y, -lim, lim));
    a = a + 1.0f;
    b = b + 1.0f;
    a.x = __builtin_amdgcn_rcpf(a.x); a.y = __builtin_amdgcn_rcpf(a.y);
    b.x = __builtin_amdgcn_rcpf(b.x); b.y = __builtin_amdgcn_rcpf(b.y);
    a = a * f32x2{v.x, v.y};
    b = b * f32x2{v.z, v.w};
    return f32x4{a.x, a.y, b.x, b.y};
}
__device__ __forceinline__ unsigned ld_sys(const unsigned* p) {
    return __hip_atomic_load((gu32*)p, __ATOMIC_RELAXED, __HIP_MEMORY_SCOPE_SYSTEM);
}
__device__ __forceinline__ unsigned ld_agent(const unsigned* p) {
    return __hip_atomic_load((gu32*)p, __ATOMIC_RELAXED, __HIP_MEMORY_SCOPE_AGENT);
}
__device__ __forceinline__ void st_agent(unsigned* p, unsigned v) {
    __hip_atomic_store((gu32*)p, v, __ATOMIC_RELAXED, __HIP_MEMORY_SCOPE_AGENT);
}
__device__ __forceinline__ void st_sys(unsigned* p, unsigned v) {
    __hip_atomic_store((gu32*)p, v, __ATOMIC_RELAXED, __HIP_MEMORY_SCOPE_SYSTEM);
}
__device__ __forceinline__ unsigned get_call(const Params& p) {
    return __builtin_amdgcn_readfirstlane(ld_agent((const unsigned*)(p.own + OFF_DONE))) / NWG + 1;
}
__device__ __forceinline__ size_t stage_of(unsigned call) {
    return CTRL_BYTES + (size_t)(call & 1) * 2 * OP_BYTES;
}
struct FlagTrack {
    const char* f0;
    int stride;
    unsigned lim, call;
    __device__ __forceinline__ unsigned peek(int i0, int lane, unsigned& thr) const {
        int g = i0 + (lane & 7);
        g = g < NSTEP ? g : NSTEP - 1;
        thr = lim;
        unsigned v = lim - 1;
        if (lane < LA) v = ld_sys((const unsigned*)(f0 + g * stride));
        return v;
    }
};
struct StepTrack {
    const char* ws;
    unsigned call;
    int wave, rank;
    __device__ __forceinline__ unsigned peek(int i0, int lane, unsigned& thr) const {
        int s = i0 + (lane & 7);
        s = s < NSTEP ? s : NSTEP - 1;
        const int q = 8 * (s - 8) + wave;
        const int j = (q * 9363) >> 16;
        const int src = (rank + 1 + q - 7 * j) & 7;
        const int off = s < 8 ? (int)OFF_LCL + s * 128 : (int)OFF_FLAG + (j * NR + src) * 128;
        thr = s < 8 ? call * NSTAGE : call;
        unsigned v = 0;
        if (lane < LA) v = ld_sys((const unsigned*)(ws + off));
        return v;
    }
};
__device__ __forceinline__ int flag_count(unsigned v, unsigned thr) {
    unsigned m = (unsigned)__builtin_amdgcn_ballot_w64((int)(v - thr) >= 0) & ((1u << LA) - 1);
    return __builtin_ctz(~m);
}
template <class T>
__device__ __forceinline__ int wait_step(const T& tr, int g0, int lane, unsigned* tmo) {
    unsigned thr;
    unsigned v = tr.peek(g0, lane, thr);
    int c = flag_count(v, thr);
    if (c > 0) return g0 + c;
    unsigned long long t0 = __builtin_amdgcn_s_memrealtime();
    for (unsigned it = 1;; ++it) {
        v = tr.peek(g0, lane, thr);
        c = flag_count(v, thr);
        if (c > 0) return g0 + c;
        if ((it & 15) == 0) {
            if (__builtin_amdgcn_readfirstlane(ld_agent(tmo)) == tr.call ||
                __builtin_amdgcn_s_memrealtime() - t0 > DEADLINE_TICKS) {
                st_agent(tmo, tr.call);
                return NSTEP + 8;
            }
        }
        __builtin_amdgcn_s_sleep(PSLEEP);
    }
}

__device__ __forceinline__ void pusher(const Params& p, int piece, int phase, int lane,
                                       unsigned* cnt, char* ring) {
    const char* srcx = (const char*)(p.x + (size_t)piece * 64 * KSH) + lane * (KSH * 4);
    const char* srcw = (const char*)(p.w + piece * 64) + (lane >> 4) * (NN * 4) + (lane & 15) * 16;
    auto issue_load = [&](int j, int slot) {
        char* l = ring + slot * 2048;
        const bool isw = (j + phase) & 1;
        const char* g0 = isw ? srcw + (size_t)j * (8 * NN * 4) : srcx + j * 32;
        const char* g1 = g0 + (isw ? 4 * NN * 4 : 16);
        __builtin_amdgcn_global_load_lds((const gu32*)g0, (unsigned*)l, 16, 0, 0);
        __builtin_amdgcn_global_load_lds((const gu32*)g1, (unsigned*)(l + 1024), 16, 0, 0);
    };
    const unsigned cnt_addr = (unsigned)(size_t)(__attribute__((address_space(3))) unsigned*)cnt;
    const char* lc = uni(p.own + OFF_LC);
    auto signal = [&](int j) {
        if (lane == 0)
            asm volatile("global_atomic_add %0, %1, %2" :: "v"(j * 128), "v"(1u), "s"(lc) : "memory");
    };
    const char* base[NR - 1];
    const unsigned vbase = (unsigned)(p.rank * C_STRIDE + piece * 1024 + lane * 16);
    auto push = [&](int j, int slot) {
        const char* l = ring + slot * 2048;
        const bool isw = (j + phase) & 1;
        float s[8];
        if (isw) {
#pragma unroll
            for (int q = 0; q < 8; ++q) s[q] = *(const float*)(l + q * 256 + lane * 4);
        } else {
            *(f32x4*)&s[0] = *(const f32x4*)(l + lane * 16);
            *(f32x4*)&s[4] = *(const f32x4*)(l + 1024 + lane * 16);
        }
        u32x4 d;
        d.x = pk16(s[0], s[1]); d.y = pk16(s[2], s[3]); d.z = pk16(s[4], s[5]); d.w = pk16(s[6], s[7]);
        unsigned vo = vbase + (unsigned)j * J_STRIDE + (isw ? (unsigned)OP_BYTES : 0u);
#pragma unroll
        for (int q = 0; q < NR - 1; ++q)
            asm volatile("global_store_dwordx4 %0, %1, %2 sc0 sc1\n\ts_nop 1" :: "v"(vo), "v"(d), "s"(base[q]) : "memory");
    };
#define PUSH_WAIT(N) asm volatile("s_waitcnt vmcnt(" #N ")" ::: "memory")
    auto throttle = [&](int j) {
        if (j < THROTTLE) return;
        const unsigned need = j - THROTTLE + 1;
        unsigned v;
        asm volatile("ds_read_b32 %0, %1\n\ts_waitcnt lgkmcnt(0)" : "=v"(v) : "v"(cnt_addr) : "memory");
        if (__builtin_amdgcn_readfirstlane(v) >= need) return;
        unsigned long long t0 = __builtin_amdgcn_s_memrealtime();
        do {
            __builtin_amdgcn_s_sleep(2);
            asm volatile("ds_read_b32 %0, %1\n\ts_waitcnt lgkmcnt(0)" : "=v"(v) : "v"(cnt_addr) : "memory");
        } while (__builtin_amdgcn_readfirstlane(v) < need &&
                 __builtin_amdgcn_s_memrealtime() - t0 < DEADLINE_TICKS);
    };
    issue_load(0, 0);
    issue_load(1, 1);
    issue_load(2, 2);
    unsigned long long wsv[NR];
#pragma unroll
    for (int k = 0; k < NR; ++k) {
        wsv[k] = (unsigned long long)p.ws[k];
        asm volatile("" : "+s"(wsv[k]));
    }
    char* peer[NR - 1];
#pragma unroll
    for (int i = 0; i < NR - 1; ++i) {
        const int q = (p.rank + 1 + (i + piece) % 7) & 7;
        unsigned long long r = wsv[0];
#pragma unroll
        for (int k = 1; k < NR; ++k) r = q == k ? wsv[k] : r;
        peer[i] = (char*)r;
    }
    const size_t stage_off = stage_of(get_call(p));
#pragma unroll
    for (int i = 0; i < NR - 1; ++i) {
        base[i] = uni(peer[i] + stage_off);
    }
    asm volatile("s_nop 4" ::: "memory");
    PUSH_WAIT(4); push(0, 0);
    issue_load(3, 0); PUSH_WAIT(2); signal(0); push(1, 1);
#pragma unroll 1
    for (int j0 = 2; j0 < 62; j0 += 3) {
        issue_load(j0 + 2, 1); PUSH_WAIT(2); signal(j0 - 1); throttle(j0); push(j0, 2);
        issue_load(j0 + 3, 2); PUSH_WAIT(2); signal(j0); throttle(j0 + 1); push(j0 + 1, 0);
        issue_load(j0 + 4, 0); PUSH_WAIT(2); signal(j0 + 1); throttle(j0 + 2); push(j0 + 2, 1);
    }
    PUSH_WAIT(0); signal(61); throttle(62); push(62, 2);
    PUSH_WAIT(0); signal(62); throttle(63); push(63, 0);
    PUSH_WAIT(0); signal(63);
#undef PUSH_WAIT
}

__device__ __forceinline__ void sender(const Params& p, int lane, unsigned call) {
    unsigned* tmo = (unsigned*)(p.own + OFF_TMO);
    const FlagTrack tr{p.own + OFF_LC, 128, call * (4 * NCOMM), call};
    char* dst = p.ws[lane & 7] + OFF_FLAG + p.rank * 128;
    int sent = 0;
    while (sent < NSTEP) {
        int have = wait_step(tr, sent, lane, tmo);
        if (have > NSTEP) break;
        for (; sent < have; ++sent)
            if (lane < NR) st_sys((unsigned*)(dst + sent * (NR * 128)), call);
    }
}
__device__ __forceinline__ void releaser(const Params& p, int lane, unsigned call, unsigned* cnt) {
    unsigned* tmo = (unsigned*)(p.own + OFF_TMO);
    const FlagTrack tr{p.own + OFF_FLAG + p.rank * 128, NR * 128, call, call};
    int have = 0;
    while (have < NSTEP - THROTTLE) {
        have = wait_step(tr, have, lane, tmo);
        if (lane == 0) __hip_atomic_store(cnt, (unsigned)have, __ATOMIC_RELAXED, __HIP_MEMORY_SCOPE_WORKGROUP);
    }
}

__device__ __forceinline__ unsigned stage_own(const Params& p, int g, int wave, int lane, char* l) {
    const int b = g & 7;
    const int v = (g >> 3) * 8 + wave;
    const int piece = v >> 1;
    const bool isw = v & 1;
    const int vo = p.rank * (int)C_STRIDE + piece * 1024 + lane * 16;
    const int r = lane >> 4, c = lane & 15;
    const char* src = isw ? (const char*)(p.w + (size_t)(b * 64 + r) * NN + piece * 64)
                          : (const char*)(p.x + (size_t)(piece * 64 + r) * KSH + b * 64);
    const int rstride = isw ? 4 * NN * 4 : 4 * KSH * 4;
#pragma unroll
    for (int i = 0; i < 16; ++i) {
        const int slot = isw ? c : (c ^ ((4 * i + r) & 15));
        __builtin_amdgcn_global_load_lds((const gu32*)(src + (size_t)i * rstride + slot * 16),
                                         (unsigned*)(l + i * 1024), 16, 0, 0);
    }
    const unsigned call = get_call(p);
    const __amdgpu_buffer_rsrc_t rd = __builtin_amdgcn_make_buffer_rsrc(
        p.own + stage_of(call) + (isw ? OP_BYTES : 0), 0, (int)OP_BYTES, 0x00020000);
    asm volatile("s_waitcnt vmcnt(0)" ::: "memory");
    float cur[64];
    if (isw) {
#pragma unroll
        for (int k = 0; k < 64; ++k) cur[k] = *(const float*)(l + k * 256 + lane * 4);
    } else {
#pragma unroll
        for (int k = 0; k < 16; ++k) *(f32x4*)&cur[4 * k] = *(const f32x4*)(l + lane * 256 + ((k ^ (lane & 15)) * 16));
    }
#pragma unroll
    for (int o = 0; o < 8; ++o) {
        u32x4 d;
        d.x = pk16(cur[8 * o], cur[8 * o + 1]); d.y = pk16(cur[8 * o + 2], cur[8 * o + 3]);
        d.z = pk16(cur[8 * o + 4], cur[8 * o + 5]); d.w = pk16(cur[8 * o + 6], cur[8 * o + 7]);
        __builtin_amdgcn_raw_buffer_store_b128(d, rd, vo, (b * 8 + o) * (int)J_STRIDE, 17);
    }
    asm volatile("s_waitcnt vmcnt(0)" ::: "memory");
    __syncthreads();
    if (threadIdx.x == 0)
        __hip_atomic_fetch_add((gu32*)(p.own + OFF_LCL + b * 128), 1u, __ATOMIC_RELAXED, __HIP_MEMORY_SCOPE_AGENT);
    return call;
}

__device__ __forceinline__ void gemm(const Params& p, int tm, int tn, char* smem, unsigned call, size_t stage_off) {
    const int tid = threadIdx.x, lane = tid & 63;
    const int wave = __builtin_amdgcn_readfirstlane(tid >> 6);
    const int wr = wave >> 2, wc = wave & 3;
    const int row0 = tm * TM, col0 = tn * TN;
    char* ws = uni(p.own);
    unsigned* tmo = (unsigned*)(ws + OFF_TMO);
    const StepTrack tr{ws, call, wave, p.rank};
    const __amdgpu_buffer_rsrc_t rs =
        __builtin_amdgcn_make_buffer_rsrc(ws + stage_off, 0, (int)(2 * OP_BYTES), 0x00020000);
    const int vx = row0 * 16 + lane * 16;
    const int vw = (int)OP_BYTES + col0 * 16 + lane * 16;
    const bool full = row0 + TM <= MM;

    auto plane = [&](int st) {
        const int q = 8 * (st - 8) + wave;
        const int jj = (q * 9363) >> 16;
        return st < 8 ? (8 * st + wave) * (int)J_STRIDE + p.rank * (int)C_STRIDE
                      : jj * (int)J_STRIDE + ((p.rank + 1 + q - 7 * jj) & 7) * (int)C_STRIDE;
    };
    auto stage_in = [&](int j, int so) {
        __attribute__((address_space(3))) char* la =
            (__attribute__((address_space(3))) char*)(smem + (j & 1) * STAGE + wave * (TM * 16));
        __attribute__((address_space(3))) char* lb =
            (__attribute__((address_space(3))) char*)(smem + (j & 1) * STAGE + A_STAGE + wave * (TN * 16));
        __builtin_amdgcn_raw_ptr_buffer_load_lds(rs, la, 16, vx, so, 0, LD_AUX);
        __builtin_amdgcn_raw_ptr_buffer_load_lds(rs, la, 16, vx, so, 1024, LD_AUX);
        __builtin_amdgcn_raw_ptr_buffer_load_lds(rs, la, 16, vx, so, 2048, LD_AUX);
        __builtin_amdgcn_raw_ptr_buffer_load_lds(rs, la, 16, vx, so, 3072, LD_AUX);
        if (full)
            __builtin_amdgcn_raw_ptr_buffer_load_lds(rs, la + 4096, 16, vx + 4096, so, 0, LD_AUX);
        __builtin_amdgcn_raw_ptr_buffer_load_lds(rs, lb, 16, vw, so, 0, LD_AUX);
        __builtin_amdgcn_raw_ptr_buffer_load_lds(rs, lb, 16, vw, so, 1024, LD_AUX);
        __builtin_amdgcn_raw_ptr_buffer_load_lds(rs, lb, 16, vw, so, 2048, LD_AUX);
        __builtin_amdgcn_raw_ptr_buffer_load_lds(rs, lb, 16, vw, so, 3072, LD_AUX);
    };

    auto stage_one = [&](int j, int i, int so) {
        __attribute__((address_space(3))) char* la =
            (__attribute__((address_space(3))) char*)(smem + (j & 1) * STAGE + wave * (TM * 16));
        __attribute__((address_space(3))) char* lb =
            (__attribute__((address_space(3))) char*)(smem + (j & 1) * STAGE + A_STAGE + wave * (TN * 16));
        if (i < 4) __builtin_amdgcn_raw_ptr_buffer_load_lds(rs, la + i * 1024, 16, vx + i * 1024, so, 0, LD_AUX);
        else if (i == 4) { if (full) __builtin_amdgcn_raw_ptr_buffer_load_lds(rs, la + 4096, 16, vx + 4096, so, 0, LD_AUX); }
        else __builtin_amdgcn_raw_ptr_buffer_load_lds(rs, lb + (i - 5) * 1024, 16, vw + (i - 5) * 1024, so, 0, LD_AUX);
    };

    f32x4 acc[10][4];
#pragma unroll
    for (int m = 0; m < 10; ++m)
#pragma unroll
        for (int n = 0; n < 4; ++n) acc[m][n] = f32x4{0.f, 0.f, 0.f, 0.f};

    const int fr = lane & 15, fq = lane >> 4;
    const int xoff = fq * (TM * 16) + (wr * 160 + fr) * 16;
    const int woff = A_STAGE + fq * (TN * 16) + (wc * 64 + fr) * 16;

    int ready;
    ready = wait_step(tr, 0, lane, tmo);
    stage_in(0, plane(0));
    asm volatile("s_waitcnt vmcnt(0)" ::: "memory");
    __builtin_amdgcn_s_barrier();

    for (int j = 0; j < NSTEP - 1; ++j) {
        constexpr bool more = true;
        const bool pre = more && j + 1 < ready;
        const bool peek = ready < NSTEP && ready < j + PEEKW;
        const int ready0 = ready;
        unsigned peekv = 0, peekt = 0, peek2;
        asm volatile("v_mov_b32 %0, 0" : "=v"(peek2));
        if (peek) peekv = tr.peek(ready0, lane, peekt);
        bool late = false;
        const int so1 = __builtin_amdgcn_readfirstlane(plane(j + 1));
        const bool preE = pre && wr == 0, preL = pre && wr != 0;
        if (ILV_L == 0 && preL) stage_in(j + 1, so1);
        const char* sx = smem + (j & 1) * STAGE + xoff;
        const char* sw = smem + (j & 1) * STAGE + woff;
        auto ldx = [&](int t) { return *(const h16x8*)(sx + (t / 10) * 4 * (TM * 16) + (t % 10) * 256); };
        auto ldw = [&](int s, int n) { return *(const h16x8*)(sw + s * 4 * (TN * 16) + n * 256); };
        h16x8 wf[2][4], xf[4];
        __builtin_amdgcn_sched_barrier(0);
        wf[0][0] = ldw(0, 0); xf[0] = ldx(0);
        __builtin_amdgcn_sched_barrier(0);
        wf[0][1] = ldw(0, 1); wf[0][2] = ldw(0, 2); wf[0][3] = ldw(0, 3);
        xf[1] = ldx(1); xf[2] = ldx(2);
#pragma unroll
        for (int t = 0; t < 20; ++t) {
            __builtin_amdgcn_sched_barrier(0);
            acc[t % 10][0] = __builtin_amdgcn_mfma_f32_16x16x32_f16(wf[t / 10][0], xf[t & 3], acc[t % 10][0], 0, 0, 0);
            __builtin_amdgcn_sched_barrier(0);
            if (t + 3 < 20) xf[(t + 3) & 3] = ldx(t + 3);
            if (t >= 2 && t < 6) wf[1][t - 2] = ldw(1, t - 2);
#pragma unroll
            for (int i = 0; i < 9; ++i) {
                if (i / ILV_E == t && preE) stage_one(j + 1, i, so1);
                if (ILV_L > 0 && i / (ILV_L > 0 ? ILV_L : 1) == t && preL) stage_one(j + 1, i, so1);
            }
            if (t == LATE_T && more && !pre) {
                asm volatile("" : "+v"(peekv));
                ready = ready0 + flag_count(peekv, peekt);
                late = j + 1 < ready;
                if (late) stage_in(j + 1, so1);
            }
            if (t == LATE_T + 1 && more && !pre && !late) {
                unsigned th2;
                peek2 = tr.peek(ready0, lane, th2);
            }
            if (t == LATE2_T && more && !pre && !late) {
                asm volatile("" : "+v"(peek2));
                ready = ready0 + flag_count(peek2, peekt);
                late = j + 1 < ready;
                if (late) stage_in(j + 1, so1);
            }
            __builtin_amdgcn_sched_barrier(0);
#pragma unroll
            for (int n = 1; n < 4; ++n)
                acc[t % 10][n] = __builtin_amdgcn_mfma_f32_16x16x32_f16(wf[t / 10][n], xf[t & 3], acc[t % 10][n], 0, 0, 0);
        }
        __builtin_amdgcn_sched_barrier(0);
        {
            const int c = flag_count(peekv, peekt);
            if (peek && ready0 + c > ready) ready = ready0 + c;
        }
        if (more && !pre && !late) {
            if (j + 1 >= ready) ready = wait_step(tr, j + 1, lane, tmo);
            stage_in(j + 1, so1);
        }
        asm volatile("s_waitcnt vmcnt(0) lgkmcnt(0)" ::: "memory");
        __builtin_amdgcn_s_barrier();
    }

    const unsigned tmo_v = ld_agent(tmo);
    const int mlim = full ? 10 : (MM - row0 - wr * 160 + 15) / 16;
    const __amdgpu_buffer_rsrc_t ro =
        __builtin_amdgcn_make_buffer_rsrc(p.out, 0, (int)((size_t)MM * NN * 4), 0x00020000);
    const int vo = ((row0 + wr * 160 + fr) * NN + col0 + wc * 64 + fq * 4) * 4;
    f32x2 kneg = {-1.44269504088896340736f, -1.44269504088896340736f};
    asm volatile("" : "+v"(kneg));
    bool bad = false;
    {
        const char* sx = smem + ((NSTEP - 1) & 1) * STAGE + xoff;
        const char* sw = smem + ((NSTEP - 1) & 1) * STAGE + woff;
        auto ldx = [&](int s, int m) { return *(const h16x8*)(sx + s * 4 * (TM * 16) + m * 256); };
        auto ldw = [&](int s, int n) { return *(const h16x8*)(sw + s * 4 * (TN * 16) + n * 256); };
        auto store_group = [&](int m) {
            if (!bad && m < mlim) {
#pragma unroll
                for (int n = 0; n < 4; ++n)
                    __builtin_amdgcn_raw_buffer_store_b128(__builtin_bit_cast(u32x4, silu4(acc[m][n], kneg)), ro,
                                                           vo + m * (16 * NN * 4) + n * 64, 0, ST_AUX);
            }
        };
        h16x8 wf[2][4], xf[2][2];
        __builtin_amdgcn_sched_barrier(0);
        wf[0][0] = ldw(0, 0); xf[0][0] = ldx(0, 0);
        __builtin_amdgcn_sched_barrier(0);
        wf[0][1] = ldw(0, 1); wf[0][2] = ldw(0, 2); wf[0][3] = ldw(0, 3);
        xf[0][1] = ldx(1, 0);
        wf[1][0] = ldw(1, 0); wf[1][1] = ldw(1, 1); wf[1][2] = ldw(1, 2); wf[1][3] = ldw(1, 3);
#pragma unroll
        for (int m = 0; m < 10; ++m) {
            __builtin_amdgcn_sched_barrier(0);
            if (m + 1 < 10) { xf[(m + 1) & 1][0] = ldx(0, m + 1); xf[(m + 1) & 1][1] = ldx(1, m + 1); }
            __builtin_amdgcn_sched_barrier(0);
#pragma unroll
            for (int s = 0; s < 2; ++s)
#pragma unroll
                for (int n = 0; n < 4; ++n)
                    acc[m][n] = __builtin_amdgcn_mfma_f32_16x16x32_f16(wf[s][n], xf[m & 1][s], acc[m][n], 0, 0, 0);
            __builtin_amdgcn_sched_barrier(0);
            if (m == 1) bad = __builtin_amdgcn_readfirstlane(tmo_v) == call;
            if (m >= 1) store_group(m - 1);
        }
        __builtin_amdgcn_sched_barrier(0);
        store_group(9);
    }
    if (bad) {
        const u32x4 nan4 = {0x7fc00000u, 0x7fc00000u, 0x7fc00000u, 0x7fc00000u};
        for (int m = 0; m < mlim; ++m)
            for (int n = 0; n < 4; ++n)
                __builtin_amdgcn_raw_buffer_store_b128(nan4, ro, vo + m * (16 * NN * 4) + n * 64, 0, 0);
    }
}

__global__ __launch_bounds__(512, 2) void fused_gemm_allgather_silu(Params p) {
    extern __shared__ __attribute__((aligned(16))) char smem[];
    const int tid = threadIdx.x, lane = tid & 63;
    const int wave = __builtin_amdgcn_readfirstlane(tid >> 6);
    unsigned* done = (unsigned*)(p.own + OFF_DONE);
    const int xcd = blockIdx.x & 7, slot = blockIdx.x >> 3;
    if (slot < 4) {
        const int cb = slot * 8 + xcd;
        unsigned* cnt = (unsigned*)smem;
        if (tid == 0) *cnt = 0;
        __syncthreads();
        if (wave < 4) pusher(p, cb * 2 + (wave & 1), wave >> 1, lane, cnt, smem + 1024 + wave * 6144);
        else if (wave == 4) releaser(p, lane, get_call(p), cnt);
    } else if (slot < 30) {
        const int i = slot - 4, h = xcd >> 2;
        const int tm = i < 24 ? 7 * h + i % 6 : 6;
        const int tn = (xcd & 3) * 4 + (i < 24 ? i / 6 : 2 * h + i - 24);
        const unsigned call = i < 16 ? stage_own(p, xcd * 16 + i, wave, lane, smem + wave * 16384) : get_call(p);
        gemm(p, tm, tn, smem, call, stage_of(call));
    }
    else if (blockIdx.x == 241) {
        if (wave == 0) sender(p, lane, get_call(p));
    }
    else if (slot == 31 && wave == 0) {
        const unsigned q = lane & 7;
        if (q != (unsigned)p.rank) {
            const char* d = p.ws[q] + CTRL_BYTES + (size_t)((lane >> 3) & 3) * OP_BYTES +
                            (size_t)p.rank * C_STRIDE + (size_t)(lane >> 5) * (C_STRIDE / 2);
            const unsigned a = ld_sys((const unsigned*)d);
            const unsigned b = ld_sys((const unsigned*)(d + C_STRIDE / 4));
            asm volatile("" :: "v"(a), "v"(b));
        }
    }
    else if (slot == 31 && wave == 1) {
        const unsigned a = ld_sys((const unsigned*)((const char*)p.out + ((size_t)lane << 20)));
        asm volatile("" :: "v"(a));
    } else if (slot == 31 && wave == 2) {
        const char* d = p.own + CTRL_BYTES + ((size_t)lane << 21);
        const unsigned a = ld_sys((const unsigned*)d);
        const unsigned b = ld_sys((const unsigned*)(d + (1 << 20)));
        asm volatile("" :: "v"(a), "v"(b));
    }
    __syncthreads();
    if (tid == 0) __hip_atomic_fetch_add((gu32*)done, 1u, __ATOMIC_RELAXED, __HIP_MEMORY_SCOPE_AGENT);
}

}

extern "C" void kernel_launch(int rank, int nranks, ncclComm_t comm,
                              hipStream_t stream,
                              void* const* d_in, const int* in_sizes, int n_in,
                              void* d_out, int out_size,
                              const int* in_dtypes, int out_dtype,
                              void* const* d_out_all,
                              void* const* d_ws_all, size_t ws_bytes) {
    (void)comm; (void)in_dtypes; (void)out_dtype; (void)d_out_all;
    if (nranks != NR || (unsigned)rank >= (unsigned)NR || n_in < 2 || ws_bytes < WS_NEED ||
        in_sizes[0] != MM * KSH || in_sizes[1] != KSH * NN || out_size != MM * NN) {
        fprintf(stderr, "unexpected problem shape\n");
        return;
    }
    static bool attr_set = false;
    if (!attr_set) {
        HIP_CHECK(hipFuncSetAttribute((const void*)fused_gemm_allgather_silu,
                                      hipFuncAttributeMaxDynamicSharedMemorySize, LDS_BYTES));
        attr_set = true;
    }
    Params p;
    p.x = (const float*)d_in[0];
    p.w = (const float*)d_in[1];
    p.out = (float*)d_out;
    for (int i = 0; i < NR; ++i) p.ws[i] = (char*)d_ws_all[i];
    p.own = p.ws[rank];
    p.rank = rank;
    fused_gemm_allgather_silu<<<NWG, 512, LDS_BYTES, stream>>>(p);
    HIP_CHECK(hipGetLastError());
}
```
